```python
import numpy as np

kernel_name = "gpu_gemm_ar_m4096_k4096_n16384_bf16_none"


ABSMAX_RTOL = 0.008

COMPUTE_DTYPE = "bfloat16"
EPILOGUE = "none"
IN_QUANT = None
OUT_QUANT = None
IN_DTYPES = {"x": "bfloat16", "w_mat": "bfloat16"}
WORLD_SIZE = 8

def _to_bf16(x):
    u = np.ascontiguousarray(x, dtype=np.float32).view(np.uint32)
    return ((u + 0x7FFF + ((u >> 16) & 1)) & 0xFFFF0000).view(np.float32)

def setup_inputs():

    x = _to_bf16(np.random.randn(4096, 4096).astype(np.float32))
    w_mat = _to_bf16((np.random.randn(4096, 16384) / np.sqrt(4096)).astype(np.float32))
    return {"x": x, "w_mat": w_mat}

def _epilogue(y):
    return y

def reference(x, w_mat):
    y = (x @ w_mat).astype(np.float32)
    return _epilogue(y)
```

```cpp
#include <hip/hip_runtime.h>
#include <rccl/rccl.h>
#include <cstdio>
#include <cstdint>

#define HIP_CHECK(expr) do { hipError_t _e = (expr); if (_e != hipSuccess) { \
    fprintf(stderr, "HIP error %s at %s:%d\n", hipGetErrorString(_e), __FILE__, __LINE__); \
    return; } } while (0)

#define CFG_NA 4
#define CFG_QN 4
#define CFG_UC 4
#define CFG_NWV 6
#define CFG_BAT 8

namespace gar {

#define LAS __attribute__((address_space(3)))
typedef short bf16x8 __attribute__((ext_vector_type(8)));
typedef float f32x4 __attribute__((ext_vector_type(4)));
typedef unsigned u32x4 __attribute__((ext_vector_type(4)));
typedef unsigned long long u64;

constexpr int NR = 8;
constexpr int MM = 4096, NN = 16384, KK = 4096, KS = KK / NR;
constexpr int NXP = MM / 256;
constexpr int NWP = NN / 256;
constexpr int NA = CFG_NA;
constexpr int NB = NWP - NR * NA;
constexpr int NW = NA + NB;
constexpr int NP = NXP + NW;
constexpr int NYT = NXP * NA;
constexpr int QN = CFG_QN;
constexpr int NWG = 256, NC = 7 * QN, NG = NWG - NC;
constexpr int NTILES = NXP * NW;
constexpr int MAXT = 4;
constexpr int NUNITS = (NXP + NWP) * 4;
constexpr int UC = CFG_UC;
constexpr int NWV = CFG_NWV;
static_assert(NP <= 64 && NYT <= 128 && NA >= 1 && NA <= 8, "flag masks");
constexpr int BAT = CFG_BAT;
static_assert(QN >= 1 && QN <= 8 && 64 % UC == 0 && NWV >= 1 && NWV <= 7 && BAT >= NWV && (128 / UC) % BAT == 0, "copy split");
static_assert(NTILES <= 3 * NG + NG / 2, "tile table");

constexpr unsigned KiB = 1024u, MiB = 1024u * 1024u;
constexpr unsigned SLICE_B = 256 * KiB;
constexpr unsigned PANEL_B = 2 * MiB;
constexpr unsigned OFF_CALL = 0;
constexpr unsigned OFF_TMO = 16 * KiB;
constexpr unsigned OFF_TK = 20 * KiB;
constexpr unsigned OFF_INF = 32 * KiB;
constexpr unsigned OFF_YF = 48 * KiB;
constexpr unsigned OFF_DONE = 56 * KiB;
constexpr unsigned OFF_GOT = 64 * KiB;
constexpr unsigned OFF_XG = 1 * MiB;
constexpr unsigned OFF_WG = OFF_XG + NXP * PANEL_B;
constexpr unsigned WS_USED = OFF_WG + NWP * PANEL_B;
constexpr unsigned OUT_B = (unsigned)MM * NN * 2;
constexpr int PUN = 256 / UC, YUN = 128 / UC;
constexpr int UPF = NP * PUN, UTF = UPF + NYT * YUN;
constexpr unsigned TK_PER_CALL = UTF + QN * BAT;
constexpr u64 DEADLINE = 100000000ull;

constexpr int HTB = 128 * 64 * 2;
constexpr int STAGE_BYTES = 8 * HTB;
constexpr int CTL_BYTES = 256;
constexpr int LDS_BYTES = STAGE_BYTES + CTL_BYTES;
constexpr int CTL_HOOK = 0;
constexpr int CTL_ACNT = 32;
constexpr int CTL_L = 0;
constexpr int CTL_GB = 64;
constexpr int CTL_CNT = 128;

__host__ __device__ constexpr int imin(int a, int b) { return a < b ? a : b; }
__host__ __device__ constexpr int imax(int a, int b) { return a > b ? a : b; }
static_assert(NA <= 7 && NW >= 14, "pull order");
__host__ __device__ constexpr int posX(int m) { return m < 4 ? m : (m < 8 ? m + 7 : m + 14); }
__host__ __device__ constexpr int posW(int i) { return i < 7 ? i + 4 : (i < 14 ? i + 8 : i + 16); }
constexpr int NBS = 8 - NA;
__host__ __device__ inline int wpanel(int r, int i) { if (i < NA) return 8 * r + i; const int b = i - NA; return (b / NBS) * 8 + NA + b % NBS; }
__host__ __device__ inline void pos_item(int pos, bool& isw, int& idx) {
    if (pos < 4) { isw = false; idx = pos; }
    else if (pos < 11) { isw = true; idx = pos - 4; }
    else if (pos < 15) { isw = false; idx = pos - 7; }
    else if (pos < 22) { isw = true; idx = pos - 8; }
    else if (pos < 30) { isw = false; idx = pos - 14; }
    else { isw = true; idx = pos - 16; }
}

#define T_M(t) ((int)((t) >> 12))
#define T_I(t) ((int)(((t) >> 6) & 63u))
#define T_S(t) ((int)((t) & 63u))
struct Params {
    const unsigned short* x;
    const unsigned short* w;
    char* ws[NR];
    char* out[NR];
    int rank;
    u64 tiles[NG];
};

#define STAMP(wv, idx) do { } while (0)
#define WAIT_V(n) asm volatile("s_waitcnt vmcnt(" #n ")" ::: "memory")
#define WAIT_L(n) asm volatile("s_waitcnt lgkmcnt(" #n ")" ::: "memory")
#define BAR __builtin_amdgcn_s_barrier()
#define SCHED __builtin_amdgcn_sched_barrier(0)
#define RFL(x) __builtin_amdgcn_readfirstlane(x)

__device__ __forceinline__ unsigned lds_rd32(unsigned a) { unsigned v; asm volatile("ds_read_b32 %0, %1\n\ts_waitcnt lgkmcnt(0)" : "=&v"(v) : "v"(a) : "memory"); return v; }
__device__ __forceinline__ u64 lds_rd64(unsigned a) { u64 v; asm volatile("ds_read_b64 %0, %1\n\ts_waitcnt lgkmcnt(0)" : "=&v"(v) : "v"(a) : "memory"); return v; }
__device__ __forceinline__ void lds_wr32(unsigned a, unsigned v) { asm volatile("ds_write_b32 %0, %1\n\ts_waitcnt lgkmcnt(0)" :: "v"(a), "v"(v) : "memory"); }
__device__ __forceinline__ void lds_wr64(unsigned a, u64 v) { asm volatile("ds_write_b64 %0, %1\n\ts_waitcnt lgkmcnt(0)" :: "v"(a), "v"(v) : "memory"); }
__device__ __forceinline__ unsigned lds_add_rtn(unsigned a, unsigned v) { unsigned o; asm volatile("ds_add_rtn_u32 %0, %1, %2\n\ts_waitcnt lgkmcnt(0)" : "=&v"(o) : "v"(a), "v"(v) : "memory"); return o; }
__device__ __forceinline__ u64 rfl64(u64 v) { return ((u64)(unsigned)RFL((unsigned)(v >> 32)) << 32) | (unsigned)RFL((unsigned)v); }
__device__ __forceinline__ unsigned ld_sys(const char* p) { return __hip_atomic_load((const unsigned*)p, __ATOMIC_RELAXED, __HIP_MEMORY_SCOPE_SYSTEM); }
__device__ __forceinline__ unsigned ld_agent(const char* p) { return __hip_atomic_load((const unsigned*)p, __ATOMIC_RELAXED, __HIP_MEMORY_SCOPE_AGENT); }
__device__ __forceinline__ void st_sys(char* p, unsigned v) { __hip_atomic_store((unsigned*)p, v, __ATOMIC_RELAXED, __HIP_MEMORY_SCOPE_SYSTEM); }
__device__ __forceinline__ void st_agent(char* p, unsigned v) { __hip_atomic_store((unsigned*)p, v, __ATOMIC_RELAXED, __HIP_MEMORY_SCOPE_AGENT); }
__device__ __forceinline__ void add_agent(char* p, unsigned v) { (void)__hip_atomic_fetch_add((unsigned*)p, v, __ATOMIC_RELAXED, __HIP_MEMORY_SCOPE_AGENT); }
__device__ __forceinline__ unsigned cvt_pk_bf16(float lo, float hi) { unsigned r; asm volatile("v_cvt_pk_bf16_f32 %0, %1, %2" : "=v"(r) : "v"(lo), "v"(hi)); return r; }
__device__ __forceinline__ __amdgpu_buffer_rsrc_t mkrsrc(const void* p, unsigned bytes) {
    const u64 a = (u64)p;
    const u64 ua = ((u64)(unsigned)RFL((unsigned)(a >> 32)) << 32) | (unsigned)RFL((unsigned)a);
    return __builtin_amdgcn_make_buffer_rsrc((void*)ua, 0, bytes, 0x00020000);
}
constexpr int AUX_SYS = 17;
constexpr int AUX_AGT = 16;

__device__ __forceinline__ int lds_byte(int r, int c) { const int st = (r >> 4) * 2 + (c >> 5), rr = r & 15, cc = c & 31, ob = rr * 64 + cc * 2; return st * 1024 + (ob ^ (((ob >> 9) & 1) << 5)); }

__device__ __forceinline__ void stage_unit(const Params& P, int u, unsigned callv) {
    const int r = P.rank, tid = threadIdx.x, lane = tid & 63, wv = RFL(tid >> 6);
    bool isw; int panel, quarter, li = 0, dstrank = -1;
    if (u < 16) { isw = false; panel = u >> 2; quarter = u & 3; }
    else if (u < 16 + 32 * NA) {
        const int v = u - 16; isw = true; li = v >> 5; dstrank = (v >> 2) & 7; panel = 8 * dstrank + li; quarter = v & 3;
    } else {
        const int u2 = u - 16 - 32 * NA; int idx; pos_item(4 + NA + (u2 >> 2), isw, idx); quarter = u2 & 3;
        if (isw) { li = idx; panel = wpanel(r, idx); } else panel = idx;
    }
    const __amdgpu_buffer_rsrc_t rd = mkrsrc(P.ws[r], WS_USED);
    if (!isw) {
        const unsigned dst0 = OFF_XG + (unsigned)panel * PANEL_B + (unsigned)r * SLICE_B + (unsigned)quarter * 64 * KiB + (unsigned)wv * 8 * KiB;
        const int rr = lane >> 2, cc = (lane & 3) ^ (((lane >> 5) & 1) << 1);
        u32x4 v[8];
#pragma unroll
        for (int e = 0; e < 8; ++e) {
            const int q = wv * 8 + e, bidx = q >> 4, st = q & 15;
            const int kt = 2 * quarter + (bidx >> 1), h = bidx & 1;
            const int row = 256 * panel + 128 * h + (st >> 1) * 16 + rr, k = kt * 64 + (st & 1) * 32 + cc * 8;
            v[e] = *(const u32x4*)(P.x + (size_t)row * KS + k);
        }
#pragma unroll
        for (int e = 0; e < 8; ++e) __builtin_amdgcn_raw_buffer_store_b128(v[e], rd, dst0 + lane * 16 + e * 1024, 0, AUX_SYS);
    } else {
        const int cc = lane & 3, ng = lane >> 2;
        const int ktl = wv >> 2, chalf = (wv >> 1) & 1, nhalf = wv & 1;
        const int k0 = 128 * quarter + 64 * ktl + 32 * chalf + 8 * cc, c0 = 128 * nhalf + 8 * ng;
        u32x4 in[8];
#pragma unroll
        for (int j = 0; j < 8; ++j) in[j] = *(const u32x4*)(P.w + (size_t)(k0 + j) * NN + 256 * panel + c0);
        const int wc = c0 >> 6, bj = (c0 >> 5) & 1, a = (c0 >> 3) & 3;
        const unsigned dst0 = OFF_WG + (unsigned)panel * PANEL_B + (unsigned)r * SLICE_B + (unsigned)(2 * quarter + ktl) * 32 * KiB;
#pragma unroll
        for (int e = 0; e < 8; ++e) {
            u32x4 o;
#pragma unroll
            for (int d = 0; d < 4; ++d) {
                const unsigned lo = in[2 * d][e >> 1], hi = in[2 * d + 1][e >> 1];
                o[d] = (e & 1) ? __builtin_amdgcn_perm(hi, lo, 0x07060302u) : __builtin_amdgcn_perm(hi, lo, 0x05040100u);
            }
            const int R = 32 * wc + 16 * (e >> 2) + 4 * a + (e & 3), st = (R >> 4) * 2 + chalf, rr = R & 15;
            const unsigned off = (unsigned)(bj * 16 * KiB + st * 1024 + rr * 64 + ((cc ^ ((rr >> 3) << 1)) * 16));
            __builtin_amdgcn_raw_buffer_store_b128(o, rd, dst0 + off, 0, AUX_SYS);
        }
    }
    WAIT_V(0);
    __syncthreads();
    if (wv == 0) {
        const unsigned call = (unsigned)RFL(callv) + 1u;
        const int pos = isw ? posW(li) : posX(panel);
        if (lane < NR) {
            const int q = lane;
            if (q != r && (dstrank < 0 || dstrank == q)) st_sys(P.ws[q] + OFF_INF + ((r * 64 + pos) * 4 + quarter) * 4, call);
            if (q == r && (dstrank < 0 || dstrank == r)) add_agent(P.ws[r] + OFF_GOT + (pos * 8 + r) * 128, (unsigned)(PUN / 4));
        }
    }
}

__device__ __forceinline__ void gemm_wg(const Params& P, LAS unsigned char* lds, int g, unsigned callv) {
    const int r = P.rank;
    if (threadIdx.x < 64) STAMP(0, 0);
    for (int u = g; u < NUNITS; u += NG) stage_unit(P, u, callv);
    if (threadIdx.x < 64) STAMP(0, 1);
    const unsigned call = (unsigned)RFL(callv) + 1u;
    __syncthreads();
    if (threadIdx.x == 0) st_agent(P.ws[r] + OFF_CALL + blockIdx.x * 64, call);

    const int tid = threadIdx.x, wid = RFL(tid >> 6), lane = tid & 63, wr = wid >> 2, wc = wid & 3, fr = lane & 15, fq = lane >> 4;
    const unsigned ctl = (unsigned)(size_t)lds + STAGE_BYTES;
    char* const wsr = P.ws[r];
    const unsigned target = (unsigned)PUN * call;
    const unsigned voff0 = (unsigned)tid * 16u;
    constexpr size_t kstep = 32 * KiB, hstep = 16 * KiB;
    const unsigned ldsw = (unsigned)wid * 1024u;
    const int aoff = lds_byte(wr * 64 + fr, fq * 8), boff = lds_byte(wc * 32 + fr, fq * 8);
#define G_SA(b, h) (((b) * 2 + (h)) * HTB)
#define G_SB(b, h) ((4 + (b) * 2 + (h)) * HTB)
#define G_STAGE(bufoff, gbase) do { _Pragma("unroll") for (int _i = 0; _i < 2; ++_i) \
        __builtin_amdgcn_global_load_lds((const unsigned*)((const char*)(gbase) + voff0 + _i * 8192), (LAS unsigned*)(lds + (bufoff) + ldsw + _i * 8192), 16, 0, 0); } while (0)
#define G_LDA(dst, b, h) do { _Pragma("unroll") for (int m = 0; m < 4; ++m) _Pragma("unroll") for (int k = 0; k < 2; ++k) dst[m][k] = *(const LAS bf16x8*)(lds + G_SA(b, h) + aoff + m * 2048 + k * 1024); } while (0)
#define G_LDB(dst, b, h) do { _Pragma("unroll") for (int n = 0; n < 2; ++n) _Pragma("unroll") for (int k = 0; k < 2; ++k) dst[n][k] = *(const LAS bf16x8*)(lds + G_SB(b, h) + boff + n * 2048 + k * 1024); } while (0)
#define G_MMA(ai, bj, At, Bt) do { __builtin_amdgcn_s_setprio(1); _Pragma("unroll") for (int m = 0; m < 4; ++m) _Pragma("unroll") for (int n = 0; n < 2; ++n) _Pragma("unroll") for (int k = 0; k < 2; ++k) \
        acc[ai][bj][m][n] = __builtin_amdgcn_mfma_f32_16x16x32_bf16(Bt[n][k], At[m][k], acc[ai][bj][m][n], 0, 0, 0); __builtin_amdgcn_s_setprio(0); } while (0)

    int ui = 0, nhook = 0;
    const u64 mytiles = P.tiles[g];
    unsigned cur = (unsigned)(mytiles & 0xffffu);
    if (cur == 0xffffu) return;
    bool bad = false;
    f32x4 acc[2][2][4][2];
    bf16x8 At[4][2], B0[2][2], B1[2][2];
    constexpr int nt = KK / 64;
    for (;;) {
        if (wid == 0) {
            const int pos = (lane & 16) ? T_S(cur) : ((lane & 8) ? posX(T_M(cur)) : posW(T_I(cur)));
            const u64 t0 = __builtin_amdgcn_s_memrealtime();
            unsigned res = 1;
            for (;;) {
                const unsigned v = ld_agent(wsr + OFF_GOT + (pos * 8 + (lane & 7)) * 128);
                if (__builtin_amdgcn_ballot_w64(v >= target) == ~0ull) break;
                if (RFL(ld_agent(wsr + OFF_TMO)) == call || __builtin_amdgcn_s_memrealtime() - t0 > DEADLINE) {
                    if (lane == 0) st_agent(wsr + OFF_TMO, call);
                    res = 2; break;
                }
                __builtin_amdgcn_s_sleep(4);
            }
            __builtin_amdgcn_fence(__ATOMIC_ACQUIRE, "agent");
            WAIT_V(0);
            lds_wr32(ctl + CTL_HOOK + (nhook & 7) * 4, res);
        }
        asm volatile("" ::: "memory"); BAR; asm volatile("" ::: "memory");
        if (RFL(lds_rd32(ctl + CTL_HOOK + (nhook & 7) * 4)) != 1u) bad = true;
        ++nhook;
        const char* cA = wsr + OFF_XG + (size_t)T_M(cur) * PANEL_B;
        const char* cB = wsr + OFF_WG + (size_t)wpanel(r, T_I(cur)) * PANEL_B;
        G_STAGE(G_SB(0, 0), cB); G_STAGE(G_SA(0, 0), cA); G_STAGE(G_SB(0, 1), cB + hstep); G_STAGE(G_SA(0, 1), cA + hstep);
        if (wr == 1) BAR;
        WAIT_V(4); BAR;
        G_STAGE(G_SB(1, 0), cB + kstep); G_STAGE(G_SA(1, 0), cA + kstep); G_STAGE(G_SB(1, 1), cB + hstep + kstep);
        WAIT_V(6); BAR;
        unsigned nxt;
        for (;;) {
#pragma unroll
            for (int a = 0; a < 2; ++a)
#pragma unroll
                for (int b = 0; b < 2; ++b)
#pragma unroll
                    for (int m = 0; m < 4; ++m)
#pragma unroll
                        for (int n = 0; n < 2; ++n) acc[a][b][m][n] = (f32x4){0.f, 0.f, 0.f, 0.f};
            if (wid == 0) STAMP(0, 2 + 2 * ui);
            nxt = (ui + 1 < 4) ? (unsigned)((mytiles >> (16 * (ui + 1))) & 0xffffu) : 0xffffu;
            bool chain = false;
            const char* nA = cA; const char* nB = cB;
            for (int t = 0; t < nt; t += 2) {
                const bool last = (t == nt - 2);
                if (last && nxt != 0xffffu) {
                    if (wid == 0) {
                        const int pos = (lane & 16) ? T_S(nxt) : ((lane & 8) ? posX(T_M(nxt)) : posW(T_I(nxt)));
                        const unsigned v = ld_agent(wsr + OFF_GOT + (pos * 8 + (lane & 7)) * 128);
                        const bool ok = __builtin_amdgcn_ballot_w64(v >= target) == ~0ull;
                        __builtin_amdgcn_fence(__ATOMIC_ACQUIRE, "agent");
                        WAIT_V(0);
                        lds_wr32(ctl + CTL_HOOK + (nhook & 7) * 4, ok ? 1u : 3u);
                    }
                    asm volatile("" ::: "memory"); BAR; asm volatile("" ::: "memory");
                    chain = RFL(lds_rd32(ctl + CTL_HOOK + (nhook & 7) * 4)) == 1u;
                    ++nhook;
                    if (chain) { nA = wsr + OFF_XG + (size_t)T_M(nxt) * PANEL_B; nB = wsr + OFF_WG + (size_t)wpanel(r, T_I(nxt)) * PANEL_B; }
                }
                const char* a1 = cA + (size_t)(t + 1) * kstep;
                const char* a2 = last ? nA : cA + (size_t)(t + 2) * kstep; const char* b2 = last ? nB : cB + (size_t)(t + 2) * kstep;
                const char* a3 = a2 + kstep; const char* b3 = b2 + kstep;
                G_LDB(B0, 0, 0); SCHED; G_LDA(At, 0, 0); G_STAGE(G_SA(1, 1), a1 + hstep);
                WAIT_L(8); BAR; WAIT_L(0); G_MMA(0, 0, At, B0); BAR; SCHED;
                G_LDB(B1, 0, 1); G_STAGE(G_SB(0, 0), b2);
                BAR; WAIT_L(0); G_MMA(0, 1, At, B1); BAR;
                G_LDA(At, 0, 1); G_STAGE(G_SA(0, 0), a2);
                BAR; WAIT_L(0); G_MMA(1, 0, At, B0); BAR; SCHED;
                G_STAGE(G_SB(0, 1), b2 + hstep);
                WAIT_V(6); BAR; G_MMA(1, 1, At, B1); BAR;
                G_LDB(B0, 1, 0); SCHED; G_LDA(At, 1, 0); G_STAGE(G_SA(0, 1), a2 + hstep);
                WAIT_L(8); BAR; WAIT_L(0); G_MMA(0, 0, At, B0); BAR; SCHED;
                G_LDB(B1, 1, 1); G_STAGE(G_SB(1, 0), b3);
                BAR; WAIT_L(0); G_MMA(0, 1, At, B1); BAR;
                G_LDA(At, 1, 1); G_STAGE(G_SA(1, 0), a3);
                BAR; WAIT_L(0); G_MMA(1, 0, At, B0); BAR; SCHED;
                G_STAGE(G_SB(1, 1), b3 + hstep);
                WAIT_V(6); BAR; G_MMA(1, 1, At, B1); BAR;
            }
            {
                int tz = threadIdx.x; asm volatile("" : "+v"(tz));
                const int l2 = tz & 63, fr2 = l2 & 15, fq2 = l2 >> 4;
                const int m0 = T_M(cur), li = T_I(cur), pn = wpanel(r, li);
                const __amdgpu_buffer_rsrc_t ro = mkrsrc(P.out[r], OUT_B);
                const unsigned voff = (unsigned)(fr2 * (NN * 2) + fq2 * 16) + (unsigned)((m0 * 256 + wr * 64) * (NN * 2) + (pn * 256 + wc * 64) * 2);
#pragma unroll
                for (int ai = 0; ai < 2; ++ai)
#pragma unroll
                    for (int m = 0; m < 4; ++m)
#pragma unroll
                        for (int bj = 0; bj < 2; ++bj) {
                            const f32x4 v0 = acc[ai][bj][m][0], v1 = acc[ai][bj][m][1];
                            u32x4 o; o.x = cvt_pk_bf16(v0[0], v0[1]); o.y = cvt_pk_bf16(v0[2], v0[3]); o.z = cvt_pk_bf16(v1[0], v1[1]); o.w = cvt_pk_bf16(v1[2], v1[3]);
                            if (bad) o = (u32x4){0x7fc07fc0u, 0x7fc07fc0u, 0x7fc07fc0u, 0x7fc07fc0u};
                            __builtin_amdgcn_raw_buffer_store_b128(o, ro, voff + (unsigned)((ai * 128 + m * 16) * (NN * 2) + bj * 64), 0, AUX_SYS);
                        }
                if (li < NA) {
                    WAIT_V(0);
                    unsigned old = 0;
                    if (l2 == 0) old = lds_add_rtn(ctl + CTL_ACNT + (ui & 7) * 4, 1u);
                    if (RFL(old) == 7u && l2 < NR && l2 != r) st_sys(P.ws[l2] + OFF_YF + (r * 128 + m0 * NA + li) * 4, call);
                }
            }
            if (wid == 0) STAMP(0, 3 + 2 * ui);
            ++ui;
            if (!chain) break;
            cur = nxt; cA = nA; cB = nB;
        }
        WAIT_V(0);
        if (wr == 0) BAR;
        BAR;
        if (nxt == 0xffffu) break;
        cur = nxt;
    }
}

__device__ __forceinline__ void dispatch_wave(const Params& P, unsigned ctl, int cwg, unsigned call) {
    const int r = P.rank, lane = threadIdx.x & 63;
    const int p = (r + 1 + cwg % 7) & 7;
    unsigned* const tkp = (unsigned*)(P.ws[r] + OFF_TK + p * 128);
    const unsigned tk0 = (call - 1u) * TK_PER_CALL;
    for (unsigned b = 0;; ++b) {
        while ((unsigned)RFL(lds_rd32(ctl + CTL_L)) / BAT + 2u < b) __builtin_amdgcn_s_sleep(2);
        unsigned o = 0;
        if (lane == 0) o = __hip_atomic_fetch_add(tkp, (unsigned)BAT, __ATOMIC_RELAXED, __HIP_MEMORY_SCOPE_AGENT);
        const unsigned base = (unsigned)RFL(o) - tk0;
        if (lane == 0) lds_wr64(ctl + CTL_GB + (b & 7) * 8, ((u64)(b + 1u) << 32) | base);
        if (base >= (unsigned)UTF) break;
    }
}

__device__ __forceinline__ void copy_wave(const Params& P, unsigned ctl, int cwg, int w, unsigned call) {
    const int r = P.rank, lane = threadIdx.x & 63;
    const int p = (r + 1 + cwg % 7) & 7, sub = cwg / 7;
    char* const wsr = P.ws[r];
    const __amdgpu_buffer_rsrc_t rs_ws = mkrsrc(P.ws[p], WS_USED), rd_ws = mkrsrc(wsr, WS_USED);
    const __amdgpu_buffer_rsrc_t rs_out = mkrsrc(P.out[p], OUT_B), rd_out = mkrsrc(P.out[r], OUT_B);
    const unsigned vo_p = (unsigned)lane * 16u, vo_y = (unsigned)(lane >> 5) * (NN * 2) + (unsigned)(lane & 31) * 16u;
    constexpr unsigned CS_P = KiB, CS_Y = 2u * NN * 2u;
    unsigned tabP, tabY0, tabY1;
    {
        bool isw; int idx; pos_item(lane < NP ? lane : 0, isw, idx);
        tabP = (isw ? OFF_WG + (unsigned)wpanel(r, idx) * PANEL_B : OFF_XG + (unsigned)idx * PANEL_B) + (unsigned)p * SLICE_B;
        const int t0 = lane, t1 = lane + 64;
        tabY0 = (unsigned)(256 * (t0 / NA)) * (unsigned)(NN * 2) + (unsigned)(8 * p + t0 % NA) * 512u;
        tabY1 = (unsigned)(256 * (t1 / NA)) * (unsigned)(NN * 2) + (unsigned)(8 * p + t1 % NA) * 512u;
    }
    u64 rin = 0, ry0 = 0, ry1 = 0;
    const u64 t0 = __builtin_amdgcn_s_memrealtime();
    int prev = -1, prevb = 0;
    auto retire = [&](int pos, int lb) {
        if (pos >= 0) {
            unsigned old = 0;
            if (lane == 0) old = lds_add_rtn(ctl + CTL_CNT + (lb & 15) * 4, 1u);
            if (((unsigned)RFL(old) + 1u) % BAT == 0u && lane == 0) add_agent(wsr + OFF_GOT + (pos * 8 + p) * 128, (unsigned)BAT);
        }
    };
    auto give_up = [&]() -> bool {
        if (RFL(ld_agent(wsr + OFF_TMO)) == call || __builtin_amdgcn_s_memrealtime() - t0 > DEADLINE) { if (lane == 0) st_agent(wsr + OFF_TMO, call); return true; }
        return false;
    };
    bool bad = false;
    STAMP(w, 0);
    for (;;) {
        unsigned o = 0;
        if (lane == 0) o = lds_add_rtn(ctl + CTL_L, 1u);
        const unsigned l = (unsigned)RFL(o), lb = l / BAT;
        u64 gb;
        for (;;) {
            gb = rfl64(lds_rd64(ctl + CTL_GB + (lb & 7) * 8));
            if ((unsigned)(gb >> 32) == lb + 1u) break;
            __builtin_amdgcn_s_sleep(1);
        }
        const unsigned tku = (unsigned)gb + l % BAT;
        if (tku >= (unsigned)UTF) break;
        const int tk = (int)tku;
        u32x4 v[UC];
        if (tk < UPF) {
            const int pos = tk / PUN;
            while (!bad && !((rin >> pos) & 1)) {
                const u32x4 f = __builtin_amdgcn_raw_buffer_load_b128(rd_ws, (unsigned)lane * 16u, OFF_INF + (unsigned)p * 1024u, AUX_SYS);
                rin = __builtin_amdgcn_ballot_w64(f.x >= call && f.y >= call && f.z >= call && f.w >= call);
                if ((rin >> pos) & 1) break;
                WAIT_V(0); retire(prev, prevb); prev = -1;
                bad = give_up();
                __builtin_amdgcn_s_sleep(1);
            }
            if (bad) continue;
            const unsigned off = (unsigned)__builtin_amdgcn_readlane(tabP, pos) + (unsigned)(tk % PUN) * (UC * CS_P);
#pragma unroll
            for (int u = 0; u < UC; ++u) v[u] = __builtin_amdgcn_raw_buffer_load_b128(rs_ws, vo_p + u * CS_P, off, AUX_SYS);
            WAIT_V(0);
#pragma unroll
            for (int u = 0; u < UC; ++u) __builtin_amdgcn_raw_buffer_store_b128(v[u], rd_ws, vo_p + off + u * CS_P, 0, AUX_AGT);
            retire(prev, prevb);
            prev = pos; prevb = (int)lb;
        } else {
            const int t = (tk - UPF) / YUN;
            for (;;) {
                const u64 mk = (NYT <= 64 || t < 64) ? ry0 : ry1;
                if (bad || ((mk >> (t & 63)) & 1)) break;
                const unsigned f0 = ld_sys(wsr + OFF_YF + (p * 128 + lane) * 4), f1 = NYT > 64 ? ld_sys(wsr + OFF_YF + (p * 128 + 64 + lane) * 4) : 0u;
                ry0 = __builtin_amdgcn_ballot_w64(f0 >= call); ry1 = __builtin_amdgcn_ballot_w64(f1 >= call);
                const u64 mk2 = (NYT <= 64 || t < 64) ? ry0 : ry1;
                if ((mk2 >> (t & 63)) & 1) break;
                WAIT_V(0); retire(prev, prevb); prev = -1;
                bad = give_up();
                __builtin_amdgcn_s_sleep(2);
            }
            const unsigned ib = (NYT <= 64 || t < 64) ? (unsigned)__builtin_amdgcn_readlane(tabY0, t & 63) : (unsigned)__builtin_amdgcn_readlane(tabY1, t & 63);
            const unsigned off = ib + (unsigned)((tk - UPF) % YUN) * (UC * CS_Y);
            if (bad) {
#pragma unroll
                for (int u = 0; u < UC; ++u) v[u] = (u32x4){0x7fc07fc0u, 0x7fc07fc0u, 0x7fc07fc0u, 0x7fc07fc0u};
            } else {
#pragma unroll
                for (int u = 0; u < UC; ++u) v[u] = __builtin_amdgcn_raw_buffer_load_b128(rs_out, vo_y + u * CS_Y, off, AUX_SYS);
            }
            WAIT_V(0);
#pragma unroll
            for (int u = 0; u < UC; ++u) __builtin_amdgcn_raw_buffer_store_b128(v[u], rd_out, vo_y + off + u * CS_Y, 0, AUX_AGT);
            retire(prev, prevb);
            prev = -1;
        }
    }
    if (lane == 0) st_sys(P.ws[p] + OFF_DONE + (r * 64 + sub * 8 + w) * 4, call);
    WAIT_V(0);
    retire(prev, prevb);
    STAMP(w, 255);
    if (cwg != 0 || w != 0) return;
    while (!bad) {
        bool ok = true;
#pragma unroll
        for (int k = 0; k < 7; ++k) {
            const int q = (r + 1 + k) & 7;
            const unsigned f = ld_sys(wsr + OFF_DONE + (q * 64 + lane) * 4);
            ok = ok && (lane >= QN * 8 || (lane & 7) >= NWV || f >= call);
        }
        if (__builtin_amdgcn_ballot_w64(ok) == ~0ull) break;
        bad = give_up();
        __builtin_amdgcn_s_sleep(1);
    }
    if (bad && lane == 0) st_sys(P.out[r] + (size_t)(8 * r * 256) * 2, 0x7fc07fc0u);
    STAMP(7, 1);
}

__global__ __launch_bounds__(512, 2) void gemm_ar_kernel(const Params P) {
    extern __shared__ __attribute__((aligned(16))) unsigned char shm[];
    LAS unsigned char* lds = (LAS unsigned char*)shm;
    const unsigned ctl = (unsigned)(size_t)lds + STAGE_BYTES;
    const int bid = blockIdx.x;
    char* const cptr = P.ws[P.rank] + OFF_CALL + bid * 64;
    const unsigned callv = ld_agent(cptr);
    if (threadIdx.x < CTL_BYTES / 4) lds_wr32(ctl + threadIdx.x * 4, 0u);
    if (bid < NC) {
        const unsigned call = (unsigned)RFL(callv) + 1u;
        __syncthreads();
        if (threadIdx.x == 0) st_agent(cptr, call);
        const int w = RFL(threadIdx.x >> 6);
        if (w < NWV) copy_wave(P, ctl, bid, w, call);
        else if (w == 7) dispatch_wave(P, ctl, bid, call);
    } else {
        gemm_wg(P, lds, bid - NC, callv);
    }
}

static void build_tiles(Params& P) {
    static_assert(NG / 8 >= 28, "block size");
    struct Blk { int r0, r1, c0, c1, av; } blk[64]; int nb = 0;
    for (int c0 = 0; c0 < NW; c0 += 7) {
        const int c1 = imin(c0 + 7, NW), rows = imin(16, 4 * (7 / (c1 - c0)));
        for (int r0 = 0; r0 < NXP; r0 += rows) { const int r1 = imin(r0 + rows, NXP); blk[nb++] = Blk{r0, r1, c0, c1, imax(posX(r1 - 1), posW(c1 - 1))}; }
    }
    for (int i = 1; i < nb; ++i) for (int j = i; j > 0 && blk[j].av < blk[j - 1].av; --j) { const Blk x = blk[j]; blk[j] = blk[j - 1]; blk[j - 1] = x; }
    unsigned t[NG][4]; int cnt[NG];
    for (int g = 0; g < NG; ++g) { for (int k = 0; k < 4; ++k) t[g][k] = 0xffffu; cnt[g] = 0; }
    double freeat[8] = {0, 0, 0, 0, 0, 0, 0, 0};
    for (int k = 0; k < nb; ++k) {
        int xg = 0;
        for (int x = 1; x < 8; ++x) if (freeat[x] < freeat[xg]) xg = x;
        const double av = 5.0 + 4.3 * (blk[k].av + 1);
        freeat[xg] = (freeat[xg] > av ? freeat[xg] : av) + 84.0;
        int wgs[NG / 8 + 1], G = 0;
        for (int pass = 3; pass >= 0; --pass)
            for (int g = 0; g < NG; ++g) if ((g + NC) % 8 == xg && cnt[g] == pass) wgs[G++] = g;
        int j = 0;
        for (int i = blk[k].c0; i < blk[k].c1; ++i)
            for (int m = blk[k].r0; m < blk[k].r1; ++m) { const int g = wgs[j++]; t[g][cnt[g]++] = (unsigned)((m << 12) | (i << 6) | blk[k].av); }
    }
    for (int g = 0; g < NG; ++g) P.tiles[g] = (u64)t[g][0] | ((u64)t[g][1] << 16) | ((u64)t[g][2] << 32) | ((u64)t[g][3] << 48);
}

}

extern "C" void kernel_launch(int rank, int nranks, ncclComm_t comm,
                              hipStream_t stream,
                              void* const* d_in, const int* in_sizes, int n_in,
                              void* d_out, int out_size,
                              const int* in_dtypes, int out_dtype,
                              void* const* d_out_all,
                              void* const* d_ws_all, size_t ws_bytes) {
    using namespace gar;
    (void)comm; (void)in_sizes; (void)in_dtypes; (void)out_dtype; (void)d_out;
    if (nranks != NR || n_in < 2 || (unsigned)rank >= (unsigned)NR || ws_bytes < WS_USED || (size_t)out_size != (size_t)MM * NN) {
        fprintf(stderr, "gemm_ar: unexpected configuration\n");
        return;
    }
    static bool attr_set[64] = {};
    int dev = 0;
    HIP_CHECK(hipGetDevice(&dev));
    if (dev >= 0 && dev < 64 && !attr_set[dev]) {
        HIP_CHECK(hipFuncSetAttribute((const void*)gemm_ar_kernel, hipFuncAttributeMaxDynamicSharedMemorySize, LDS_BYTES));
        attr_set[dev] = true;
    }
    Params P;
    P.x = (const unsigned short*)d_in[0];
    P.w = (const unsigned short*)d_in[1];
    for (int p = 0; p < NR; ++p) { P.ws[p] = (char*)d_ws_all[p]; P.out[p] = (char*)d_out_all[p]; }
    P.rank = rank;
    build_tiles(P);
    hipLaunchKernelGGL(gemm_ar_kernel, dim3(NWG), dim3(512), LDS_BYTES, stream, P);
    HIP_CHECK(hipGetLastError());
}
```

```cpp
#include <hip/hip_runtime.h>
#include <rccl/rccl.h>
#include <cstdio>
#include <cstdint>

#define HIP_CHECK(expr) do { hipError_t _e = (expr); if (_e != hipSuccess) { \
    fprintf(stderr, "HIP error %s at %s:%d\n", hipGetErrorString(_e), __FILE__, __LINE__); \
    return; } } while (0)

#define CFG_NA 4
#define CFG_PUSH 1
#define CFG_UC 4
#define CFG_NWV 6
#define CFG_BAT 8

namespace gar {

#define LAS __attribute__((address_space(3)))
typedef short bf16x8 __attribute__((ext_vector_type(8)));
typedef float f32x4 __attribute__((ext_vector_type(4)));
typedef unsigned u32x4 __attribute__((ext_vector_type(4)));
typedef unsigned long long u64;

constexpr int NR = 8;
constexpr int MM = 4096, NN = 16384, KK = 4096, KS = KK / NR;
constexpr int NXP = MM / 256;
constexpr int NWP = NN / 256;
constexpr int NA = CFG_NA;
constexpr int NB = NWP - NR * NA;
constexpr int NW = NA + NB;
constexpr int NP = NXP + NW;
constexpr int NYT = NXP * NA;
constexpr int QN = 4;
constexpr int NPUSH = CFG_PUSH;
constexpr int NWG = 256, NC = 7 * QN, NG = NWG - NC;
constexpr int NTILES = NXP * NW;
constexpr int MAXT = 4;
constexpr int NUNITS = (NXP + NWP) * 4;
constexpr int UC = CFG_UC;
constexpr int NWV = CFG_NWV;
static_assert(NP <= 64 && NYT <= 128 && NA >= 1 && NA <= 8, "flag masks");
constexpr int BAT = CFG_BAT;
static_assert(QN >= 1 && QN <= 8 && 64 % UC == 0 && NWV >= 1 && NWV <= 7 && BAT >= NWV && (128 / UC) % BAT == 0, "copy split");
static_assert(NTILES <= 3 * NG + NG / 2, "tile table");

constexpr unsigned KiB = 1024u, MiB = 1024u * 1024u;
constexpr unsigned SLICE_B = 256 * KiB;
constexpr unsigned PANEL_B = 2 * MiB;
constexpr unsigned OFF_CALL = 0;
constexpr unsigned OFF_TMO = 16 * KiB;
constexpr unsigned OFF_TK = 20 * KiB;
constexpr unsigned OFF_INF = 32 * KiB;
constexpr unsigned OFF_YF = 48 * KiB;
constexpr unsigned OFF_DONE = 56 * KiB;
constexpr unsigned OFF_GOT = 64 * KiB;
constexpr unsigned OFF_XG = 1 * MiB;
constexpr unsigned OFF_WG = OFF_XG + (NXP + 1) * PANEL_B;
constexpr unsigned WS_USED = OFF_WG + NWP * PANEL_B;
constexpr unsigned OUT_B = (unsigned)MM * NN * 2;
constexpr int PUN = 256 / UC, YUN = 128 / UC;
constexpr int UPF = NP * PUN, UTF = UPF + NYT * YUN;
constexpr int UT0 = NPUSH * PUN;
constexpr unsigned TK_PER_CALL = UTF - UT0 + QN * BAT;
constexpr u64 DEADLINE = 100000000ull;

constexpr int HTB = 128 * 64 * 2;
constexpr int STAGE_BYTES = 8 * HTB;
constexpr int CTL_BYTES = 256;
constexpr int LDS_BYTES = STAGE_BYTES + CTL_BYTES;
constexpr int CTL_HOOK = 0;
constexpr int CTL_ACNT = 32;
constexpr int CTL_L = 0;
constexpr int CTL_GB = 64;
constexpr int CTL_CNT = 128;

__host__ __device__ constexpr int imin(int a, int b) { return a < b ? a : b; }
__host__ __device__ constexpr int imax(int a, int b) { return a > b ? a : b; }
static_assert(NA <= 7 && NW >= 14, "pull order");
__host__ __device__ constexpr int posX(int m) { return m < 4 ? m : (m < 8 ? m + 7 : m + 14); }
__host__ __device__ constexpr int posW(int i) { return i < 7 ? i + 4 : (i < 14 ? i + 8 : i + 16); }
constexpr int NBS = 8 - NA;
__host__ __device__ inline int wpanel(int r, int i) { if (i < NA) return 8 * r + i; const int b = i - NA; return (b / NBS) * 8 + NA + b % NBS; }
__host__ __device__ inline void pos_item(int pos, bool& isw, int& idx) {
    if (pos < 4) { isw = false; idx = pos; }
    else if (pos < 11) { isw = true; idx = pos - 4; }
    else if (pos < 15) { isw = false; idx = pos - 7; }
    else if (pos < 22) { isw = true; idx = pos - 8; }
    else if (pos < 30) { isw = false; idx = pos - 14; }
    else { isw = true; idx = pos - 16; }
}

__device__ __forceinline__ unsigned xpanel_off(int m, unsigned call) { return OFF_XG + (unsigned)((NPUSH && m == 0 && (call & 1u)) ? NXP : m) * PANEL_B; }
#define T_M(t) ((int)((t) >> 12))
#define T_I(t) ((int)(((t) >> 6) & 63u))
#define T_S(t) ((int)((t) & 63u))
struct Params {
    const unsigned short* x;
    const unsigned short* w;
    char* ws[NR];
    char* out[NR];
    int rank;
    int role[32];
    u64 tiles[NG];
};

#define STAMP(wv, idx) do { } while (0)
#define WAIT_V(n) asm volatile("s_waitcnt vmcnt(" #n ")" ::: "memory")
#define WAIT_L(n) asm volatile("s_waitcnt lgkmcnt(" #n ")" ::: "memory")
#define BAR __builtin_amdgcn_s_barrier()
#define SCHED __builtin_amdgcn_sched_barrier(0)
#define RFL(x) __builtin_amdgcn_readfirstlane(x)

__device__ __forceinline__ unsigned lds_rd32(unsigned a) { unsigned v; asm volatile("ds_read_b32 %0, %1\n\ts_waitcnt lgkmcnt(0)" : "=&v"(v) : "v"(a) : "memory"); return v; }
__device__ __forceinline__ u64 lds_rd64(unsigned a) { u64 v; asm volatile("ds_read_b64 %0, %1\n\ts_waitcnt lgkmcnt(0)" : "=&v"(v) : "v"(a) : "memory"); return v; }
__device__ __forceinline__ void lds_wr32(unsigned a, unsigned v) { asm volatile("ds_write_b32 %0, %1\n\ts_waitcnt lgkmcnt(0)" :: "v"(a), "v"(v) : "memory"); }
__device__ __forceinline__ void lds_wr64(unsigned a, u64 v) { asm volatile("ds_write_b64 %0, %1\n\ts_waitcnt lgkmcnt(0)" :: "v"(a), "v"(v) : "memory"); }
__device__ __forceinline__ unsigned lds_add_rtn(unsigned a, unsigned v) { unsigned o; asm volatile("ds_add_rtn_u32 %0, %1, %2\n\ts_waitcnt lgkmcnt(0)" : "=&v"(o) : "v"(a), "v"(v) : "memory"); return o; }
__device__ __forceinline__ u64 rfl64(u64 v) { return ((u64)(unsigned)RFL((unsigned)(v >> 32)) << 32) | (unsigned)RFL((unsigned)v); }
__device__ __forceinline__ unsigned ld_sys(const char* p) { return __hip_atomic_load((const unsigned*)p, __ATOMIC_RELAXED, __HIP_MEMORY_SCOPE_SYSTEM); }
__device__ __forceinline__ unsigned ld_agent(const char* p) { return __hip_atomic_load((const unsigned*)p, __ATOMIC_RELAXED, __HIP_MEMORY_SCOPE_AGENT); }
__device__ __forceinline__ void st_sys(char* p, unsigned v) { __hip_atomic_store((unsigned*)p, v, __ATOMIC_RELAXED, __HIP_MEMORY_SCOPE_SYSTEM); }
__device__ __forceinline__ void st_agent(char* p, unsigned v) { __hip_atomic_store((unsigned*)p, v, __ATOMIC_RELAXED, __HIP_MEMORY_SCOPE_AGENT); }
__device__ __forceinline__ void add_agent(char* p, unsigned v) { (void)__hip_atomic_fetch_add((unsigned*)p, v, __ATOMIC_RELAXED, __HIP_MEMORY_SCOPE_AGENT); }
__device__ __forceinline__ unsigned cvt_pk_bf16(float lo, float hi) { unsigned r; asm volatile("v_cvt_pk_bf16_f32 %0, %1, %2" : "=v"(r) : "v"(lo), "v"(hi)); return r; }
__device__ __forceinline__ char* uptr(char* p) { return (char*)rfl64((u64)p); }
__device__ __forceinline__ __amdgpu_buffer_rsrc_t mkrsrc(const void* p, unsigned bytes) {
    const u64 a = (u64)p;
    const u64 ua = ((u64)(unsigned)RFL((unsigned)(a >> 32)) << 32) | (unsigned)RFL((unsigned)a);
    return __builtin_amdgcn_make_buffer_rsrc((void*)ua, 0, bytes, 0x00020000);
}
constexpr int AUX_SYS = 17;
constexpr int AUX_AGT = 16;

__device__ __forceinline__ int lds_byte(int r, int c) { const int st = (r >> 4) * 2 + (c >> 5), rr = r & 15, cc = c & 31, ob = rr * 64 + cc * 2; return st * 1024 + (ob ^ (((ob >> 9) & 1) << 5)); }

__device__ __forceinline__ void stage_x(const Params& P, __amdgpu_buffer_rsrc_t rd, int panel, int quarter, unsigned callv) {
    const int r = P.rank, tid = threadIdx.x, lane = tid & 63, wv = RFL(tid >> 6);
    const int rr = lane >> 2, cc = (lane & 3) ^ (((lane >> 5) & 1) << 1);
    u32x4 v[8];
#pragma unroll
    for (int e = 0; e < 8; ++e) {
        const int q = wv * 8 + e, bidx = q >> 4, st = q & 15;
        const int kt = 2 * quarter + (bidx >> 1), h = bidx & 1;
        const int row = 256 * panel + 128 * h + (st >> 1) * 16 + rr, k = kt * 64 + (st & 1) * 32 + cc * 8;
        v[e] = *(const u32x4*)(P.x + (size_t)row * KS + k);
    }
    const unsigned dst0 = xpanel_off(panel, (unsigned)RFL(callv) + 1u) + (unsigned)r * SLICE_B + (unsigned)quarter * 64 * KiB + (unsigned)wv * 8 * KiB;
#pragma unroll
    for (int e = 0; e < 8; ++e) __builtin_amdgcn_raw_buffer_store_b128(v[e], rd, dst0 + lane * 16 + e * 1024, 0, AUX_SYS);
}

__device__ __forceinline__ void stage_unit(const Params& P, int u, unsigned callv) {
    const int r = P.rank, tid = threadIdx.x, lane = tid & 63, wv = RFL(tid >> 6);
    bool isw; int panel, quarter, li = 0, dstrank = -1;
    if (u < 16) { isw = false; panel = u >> 2; quarter = u & 3; if (panel < NPUSH) dstrank = r; }
    else if (u < 16 + 32 * NA) {
        const int v = u - 16; isw = true; li = v >> 5; dstrank = (v >> 2) & 7; panel = 8 * dstrank + li; quarter = v & 3;
    } else {
        const int u2 = u - 16 - 32 * NA; int idx; pos_item(4 + NA + (u2 >> 2), isw, idx); quarter = u2 & 3;
        if (isw) { li = idx; panel = wpanel(r, idx); } else panel = idx;
    }
    const __amdgpu_buffer_rsrc_t rd = mkrsrc(P.ws[r], WS_USED);
    if (!isw) {
        stage_x(P, rd, panel, quarter, callv);
    } else {
        const int cc = lane & 3, ng = lane >> 2;
        const int ktl = wv >> 2, chalf = (wv >> 1) & 1, nhalf = wv & 1;
        const int k0 = 128 * quarter + 64 * ktl + 32 * chalf + 8 * cc, c0 = 128 * nhalf + 8 * ng;
        u32x4 in[8];
#pragma unroll
        for (int j = 0; j < 8; ++j) in[j] = *(const u32x4*)(P.w + (size_t)(k0 + j) * NN + 256 * panel + c0);
        const int wc = c0 >> 6, bj = (c0 >> 5) & 1, a = (c0 >> 3) & 3;
        const unsigned dst0 = OFF_WG + (unsigned)panel * PANEL_B + (unsigned)r * SLICE_B + (unsigned)(2 * quarter + ktl) * 32 * KiB;
#pragma unroll
        for (int e = 0; e < 8; ++e) {
            u32x4 o;
#pragma unroll
            for (int d = 0; d < 4; ++d) {
                const unsigned lo = in[2 * d][e >> 1], hi = in[2 * d + 1][e >> 1];
                o[d] = (e & 1) ? __builtin_amdgcn_perm(hi, lo, 0x07060302u) : __builtin_amdgcn_perm(hi, lo, 0x05040100u);
            }
            const int R = 32 * wc + 16 * (e >> 2) + 4 * a + (e & 3), st = (R >> 4) * 2 + chalf, rr = R & 15;
            const unsigned off = (unsigned)(bj * 16 * KiB + st * 1024 + rr * 64 + ((cc ^ ((rr >> 3) << 1)) * 16));
            __builtin_amdgcn_raw_buffer_store_b128(o, rd, dst0 + off, 0, AUX_SYS);
        }
    }
    WAIT_V(0);
    __syncthreads();
    if (wv == 0) {
        const unsigned call = (unsigned)RFL(callv) + 1u;
        const int pos = isw ? posW(li) : posX(panel);
        if (lane < NR) {
            const int q = lane;
            if (q != r && (dstrank < 0 || dstrank == q)) st_sys(P.ws[q] + OFF_INF + ((r * 64 + pos) * 4 + quarter) * 4, call);
            if (q == r && (dstrank < 0 || dstrank == r)) add_agent(P.ws[r] + OFF_GOT + (pos * 8 + r) * 128, (unsigned)(PUN / 4));
        }
    }
}

__device__ __forceinline__ void gemm_wg(const Params& P, LAS unsigned char* lds, int g, unsigned callv) {
    const int r = P.rank;
    if (threadIdx.x < 64) STAMP(0, 0);
    for (int u = g; u < NUNITS; u += NG) stage_unit(P, u, callv);
    if (threadIdx.x < 64) STAMP(0, 1);
    const unsigned call = (unsigned)RFL(callv) + 1u;
    __syncthreads();
    if (threadIdx.x == 0) st_agent(P.ws[r] + OFF_CALL + blockIdx.x * 64, call);

    const int tid = threadIdx.x, wid = RFL(tid >> 6), lane = tid & 63, wr = wid >> 2, wc = wid & 3, fr = lane & 15, fq = lane >> 4;
    const unsigned ctl = (unsigned)(size_t)lds + STAGE_BYTES;
    char* const wsr = uptr(P.ws[r]);
    const unsigned target = (unsigned)PUN * call;
    const unsigned voff0 = (unsigned)tid * 16u;
    constexpr size_t kstep = 32 * KiB, hstep = 16 * KiB;
    const unsigned ldsw = (unsigned)wid * 1024u;
    const int aoff = lds_byte(wr * 64 + fr, fq * 8), boff = lds_byte(wc * 32 + fr, fq * 8);
#define G_SA(b, h) (((b) * 2 + (h)) * HTB)
#define G_SB(b, h) ((4 + (b) * 2 + (h)) * HTB)
#define G_STAGE(bufoff, gbase) do { _Pragma("unroll") for (int _i = 0; _i < 2; ++_i) \
        __builtin_amdgcn_global_load_lds((const unsigned*)((const char*)(gbase) + voff0 + _i * 8192), (LAS unsigned*)(lds + (bufoff) + ldsw + _i * 8192), 16, 0, 0); } while (0)
#define G_LDA(dst, b, h) do { _Pragma("unroll") for (int m = 0; m < 4; ++m) _Pragma("unroll") for (int k = 0; k < 2; ++k) dst[m][k] = *(const LAS bf16x8*)(lds + G_SA(b, h) + aoff + m * 2048 + k * 1024); } while (0)
#define G_LDB(dst, b, h) do { _Pragma("unroll") for (int n = 0; n < 2; ++n) _Pragma("unroll") for (int k = 0; k < 2; ++k) dst[n][k] = *(const LAS bf16x8*)(lds + G_SB(b, h) + boff + n * 2048 + k * 1024); } while (0)
#define G_MMA(ai, bj, At, Bt) do { __builtin_amdgcn_s_setprio(1); _Pragma("unroll") for (int m = 0; m < 4; ++m) _Pragma("unroll") for (int n = 0; n < 2; ++n) _Pragma("unroll") for (int k = 0; k < 2; ++k) \
        acc[ai][bj][m][n] = __builtin_amdgcn_mfma_f32_16x16x32_bf16(Bt[n][k], At[m][k], acc[ai][bj][m][n], 0, 0, 0); __builtin_amdgcn_s_setprio(0); } while (0)

    int ui = 0, nhook = 0;
    const u64 mytiles = rfl64(P.tiles[g]);
    unsigned cur = (unsigned)(mytiles & 0xffffu);
    if (cur == 0xffffu) return;
    bool bad = false;
    f32x4 acc[2][2][4][2];
    bf16x8 At[4][2], B0[2][2], B1[2][2];
    constexpr int nt = KK / 64;
    for (;;) {
        if (wid == 0) {
            const int pos = (lane & 16) ? T_S(cur) : ((lane & 8) ? posX(T_M(cur)) : posW(T_I(cur)));
            const u64 t0 = __builtin_amdgcn_s_memrealtime();
            unsigned res = 1;
            for (;;) {
                const unsigned v = ld_agent(wsr + OFF_GOT + (pos * 8 + (lane & 7)) * 128);
                if (__builtin_amdgcn_ballot_w64(v >= target) == ~0ull) break;
                if (RFL(ld_agent(wsr + OFF_TMO)) == call || __builtin_amdgcn_s_memrealtime() - t0 > DEADLINE) {
                    if (lane == 0) st_agent(wsr + OFF_TMO, call);
                    res = 2; break;
                }
                __builtin_amdgcn_s_sleep(4);
            }
            __builtin_amdgcn_fence(__ATOMIC_ACQUIRE, "agent");
            WAIT_V(0);
            lds_wr32(ctl + CTL_HOOK + (nhook & 7) * 4, res);
        }
        asm volatile("" ::: "memory"); BAR; asm volatile("" ::: "memory");
        if (RFL(lds_rd32(ctl + CTL_HOOK + (nhook & 7) * 4)) != 1u) bad = true;
        ++nhook;
        const char* cA = wsr + xpanel_off(T_M(cur), call);
        const char* cB = wsr + OFF_WG + (size_t)wpanel(r, T_I(cur)) * PANEL_B;
        G_STAGE(G_SB(0, 0), cB); G_STAGE(G_SA(0, 0), cA); G_STAGE(G_SB(0, 1), cB + hstep); G_STAGE(G_SA(0, 1), cA + hstep);
        if (wr == 1) BAR;
        WAIT_V(4); BAR;
        G_STAGE(G_SB(1, 0), cB + kstep); G_STAGE(G_SA(1, 0), cA + kstep); G_STAGE(G_SB(1, 1), cB + hstep + kstep);
        WAIT_V(6); BAR;
        unsigned nxt;
        for (;;) {
#pragma unroll
            for (int a = 0; a < 2; ++a)
#pragma unroll
                for (int b = 0; b < 2; ++b)
#pragma unroll
                    for (int m = 0; m < 4; ++m)
#pragma unroll
                        for (int n = 0; n < 2; ++n) acc[a][b][m][n] = (f32x4){0.f, 0.f, 0.f, 0.f};
            if (wid == 0) STAMP(0, 2 + 2 * ui);
            nxt = (ui + 1 < 4) ? (unsigned)((mytiles >> (16 * (ui + 1))) & 0xffffu) : 0xffffu;
            bool chain = false;
            const char* nA = cA; const char* nB = cB;
            for (int t = 0; t < nt; t += 2) {
                const bool last = (t == nt - 2);
                if (last && nxt != 0xffffu) {
                    if (wid == 0) {
                        const int pos = (lane & 16) ? T_S(nxt) : ((lane & 8) ? posX(T_M(nxt)) : posW(T_I(nxt)));
                        const unsigned v = ld_agent(wsr + OFF_GOT + (pos * 8 + (lane & 7)) * 128);
                        const bool ok = __builtin_amdgcn_ballot_w64(v >= target) == ~0ull;
                        __builtin_amdgcn_fence(__ATOMIC_ACQUIRE, "agent");
                        WAIT_V(0);
                        lds_wr32(ctl + CTL_HOOK + (nhook & 7) * 4, ok ? 1u : 3u);
                    }
                    asm volatile("" ::: "memory"); BAR; asm volatile("" ::: "memory");
                    chain = RFL(lds_rd32(ctl + CTL_HOOK + (nhook & 7) * 4)) == 1u;
                    ++nhook;
                    if (chain) { nA = wsr + xpanel_off(T_M(nxt), call); nB = wsr + OFF_WG + (size_t)wpanel(r, T_I(nxt)) * PANEL_B; }
                }
                const char* a1 = cA + (size_t)(t + 1) * kstep;
                const char* a2 = last ? nA : cA + (size_t)(t + 2) * kstep; const char* b2 = last ? nB : cB + (size_t)(t + 2) * kstep;
                const char* a3 = a2 + kstep; const char* b3 = b2 + kstep;
                G_LDB(B0, 0, 0); SCHED; G_LDA(At, 0, 0); G_STAGE(G_SA(1, 1), a1 + hstep);
                WAIT_L(8); BAR; WAIT_L(0); G_MMA(0, 0, At, B0); BAR; SCHED;
                G_LDB(B1, 0, 1); G_STAGE(G_SB(0, 0), b2);
                BAR; WAIT_L(0); G_MMA(0, 1, At, B1); BAR;
                G_LDA(At, 0, 1); G_STAGE(G_SA(0, 0), a2);
                BAR; WAIT_L(0); G_MMA(1, 0, At, B0); BAR; SCHED;
                G_STAGE(G_SB(0, 1), b2 + hstep);
                WAIT_V(6); BAR; G_MMA(1, 1, At, B1); BAR;
                G_LDB(B0, 1, 0); SCHED; G_LDA(At, 1, 0); G_STAGE(G_SA(0, 1), a2 + hstep);
                WAIT_L(8); BAR; WAIT_L(0); G_MMA(0, 0, At, B0); BAR; SCHED;
                G_LDB(B1, 1, 1); G_STAGE(G_SB(1, 0), b3);
                BAR; WAIT_L(0); G_MMA(0, 1, At, B1); BAR;
                G_LDA(At, 1, 1); G_STAGE(G_SA(1, 0), a3);
                BAR; WAIT_L(0); G_MMA(1, 0, At, B0); BAR; SCHED;
                G_STAGE(G_SB(1, 1), b3 + hstep);
                WAIT_V(6); BAR; G_MMA(1, 1, At, B1); BAR;
            }
            {
                int tz = threadIdx.x; asm volatile("" : "+v"(tz));
                const int l2 = tz & 63, fr2 = l2 & 15, fq2 = l2 >> 4;
                const int m0 = T_M(cur), li = T_I(cur), pn = wpanel(r, li);
                const __amdgpu_buffer_rsrc_t ro = mkrsrc(P.out[r], OUT_B);
                const unsigned voff = (unsigned)(fr2 * (NN * 2) + fq2 * 16) + (unsigned)((m0 * 256 + wr * 64) * (NN * 2) + (pn * 256 + wc * 64) * 2);
#pragma unroll
                for (int ai = 0; ai < 2; ++ai)
#pragma unroll
                    for (int m = 0; m < 4; ++m)
#pragma unroll
                        for (int bj = 0; bj < 2; ++bj) {
                            const f32x4 v0 = acc[ai][bj][m][0], v1 = acc[ai][bj][m][1];
                            u32x4 o; o.x = cvt_pk_bf16(v0[0], v0[1]); o.y = cvt_pk_bf16(v0[2], v0[3]); o.z = cvt_pk_bf16(v1[0], v1[1]); o.w = cvt_pk_bf16(v1[2], v1[3]);
                            if (bad) o = (u32x4){0x7fc07fc0u, 0x7fc07fc0u, 0x7fc07fc0u, 0x7fc07fc0u};
                            __builtin_amdgcn_raw_buffer_store_b128(o, ro, voff + (unsigned)((ai * 128 + m * 16) * (NN * 2) + bj * 64), 0, AUX_SYS);
                        }
                if (li < NA) {
                    WAIT_V(0);
                    unsigned old = 0;
                    if (l2 == 0) old = lds_add_rtn(ctl + CTL_ACNT + (ui & 7) * 4, 1u);
                    if (RFL(old) == 7u && l2 < NR && l2 != r) st_sys(P.ws[l2] + OFF_YF + (r * 128 + m0 * NA + li) * 4, call);
                }
            }
            if (wid == 0) STAMP(0, 3 + 2 * ui);
            ++ui;
            if (!chain) break;
            cur = nxt; cA = nA; cB = nB;
        }
        WAIT_V(0);
        if (wr == 0) BAR;
        BAR;
        if (nxt == 0xffffu) break;
        cur = nxt;
    }
}

__device__ __forceinline__ void dispatch_wave(const Params& P, unsigned ctl, int p, unsigned call) {
    const int r = P.rank, lane = threadIdx.x & 63;
    unsigned* const tkp = (unsigned*)(P.ws[r] + OFF_TK + p * 128);
    const unsigned tk0 = (call - 1u) * TK_PER_CALL;
    for (unsigned b = 0;; ++b) {
        while ((unsigned)RFL(lds_rd32(ctl + CTL_L)) / BAT + 2u < b) __builtin_amdgcn_s_sleep(2);
        unsigned o = 0;
        if (lane == 0) o = __hip_atomic_fetch_add(tkp, (unsigned)BAT, __ATOMIC_RELAXED, __HIP_MEMORY_SCOPE_AGENT);
        const unsigned base = (unsigned)RFL(o) - tk0 + (unsigned)UT0;
        if (lane == 0) lds_wr64(ctl + CTL_GB + (b & 7) * 8, ((u64)(b + 1u) << 32) | base);
        if (base >= (unsigned)UTF) break;
    }
}

__device__ __forceinline__ void copy_wave(const Params& P, unsigned ctl, int p, int sub, int w, unsigned call) {
    const int r = P.rank, lane = threadIdx.x & 63;
    char* const wsr = P.ws[r];
    const __amdgpu_buffer_rsrc_t rs_ws = mkrsrc(P.ws[p], WS_USED), rd_ws = mkrsrc(wsr, WS_USED);
    const __amdgpu_buffer_rsrc_t rs_out = mkrsrc(P.out[p], OUT_B), rd_out = mkrsrc(P.out[r], OUT_B);
    const unsigned vo_p = (unsigned)lane * 16u, vo_y = (unsigned)(lane >> 5) * (NN * 2) + (unsigned)(lane & 31) * 16u;
    constexpr unsigned CS_P = KiB, CS_Y = 2u * NN * 2u;
    unsigned tabP, tabY0, tabY1;
    {
        bool isw; int idx; pos_item(lane < NP ? lane : 0, isw, idx);
        tabP = (isw ? OFF_WG + (unsigned)wpanel(r, idx) * PANEL_B : OFF_XG + (unsigned)idx * PANEL_B) + (unsigned)p * SLICE_B;
        const int t0 = lane, t1 = lane + 64;
        tabY0 = (unsigned)(256 * (t0 / NA)) * (unsigned)(NN * 2) + (unsigned)(8 * p + t0 % NA) * 512u;
        tabY1 = (unsigned)(256 * (t1 / NA)) * (unsigned)(NN * 2) + (unsigned)(8 * p + t1 % NA) * 512u;
    }
    u64 rin = 0, ry0 = 0, ry1 = 0;
    const u64 t0 = __builtin_amdgcn_s_memrealtime();
    int prev = -1, prevb = 0;
    auto retire = [&](int pos, int lb) {
        if (pos >= 0) {
            unsigned old = 0;
            if (lane == 0) old = lds_add_rtn(ctl + CTL_CNT + (lb & 15) * 4, 1u);
            if (((unsigned)RFL(old) + 1u) % BAT == 0u && lane == 0) add_agent(wsr + OFF_GOT + (pos * 8 + p) * 128, (unsigned)BAT);
        }
    };
    auto give_up = [&]() -> bool {
        if (RFL(ld_agent(wsr + OFF_TMO)) == call || __builtin_amdgcn_s_memrealtime() - t0 > DEADLINE) { if (lane == 0) st_agent(wsr + OFF_TMO, call); return true; }
        return false;
    };
    bool bad = false;
    STAMP(w, 0);
    for (;;) {
        unsigned o = 0;
        if (lane == 0) o = lds_add_rtn(ctl + CTL_L, 1u);
        const unsigned l = (unsigned)RFL(o), lb = l / BAT;
        u64 gb;
        for (;;) {
            gb = rfl64(lds_rd64(ctl + CTL_GB + (lb & 7) * 8));
            if ((unsigned)(gb >> 32) == lb + 1u) break;
            __builtin_amdgcn_s_sleep(1);
        }
        const unsigned tku = (unsigned)gb + l % BAT;
        if (tku >= (unsigned)UTF) break;
        const int tk = (int)tku;
        u32x4 v[UC];
        if (tk < UPF) {
            const int pos = tk / PUN;
            while (!bad && !((rin >> pos) & 1)) {
                const u32x4 f = __builtin_amdgcn_raw_buffer_load_b128(rd_ws, (unsigned)lane * 16u, OFF_INF + (unsigned)p * 1024u, AUX_SYS);
                rin = __builtin_amdgcn_ballot_w64(f.x >= call && f.y >= call && f.z >= call && f.w >= call);
                if ((rin >> pos) & 1) break;
                WAIT_V(0); retire(prev, prevb); prev = -1;
                bad = give_up();
                __builtin_amdgcn_s_sleep(1);
            }
            if (bad) continue;
            const unsigned off = (unsigned)__builtin_amdgcn_readlane(tabP, pos) + (unsigned)(tk % PUN) * (UC * CS_P);
#pragma unroll
            for (int u = 0; u < UC; ++u) v[u] = __builtin_amdgcn_raw_buffer_load_b128(rs_ws, vo_p + u * CS_P, off, AUX_SYS);
            WAIT_V(0);
#pragma unroll
            for (int u = 0; u < UC; ++u) __builtin_amdgcn_raw_buffer_store_b128(v[u], rd_ws, vo_p + off + u * CS_P, 0, AUX_AGT);
            retire(prev, prevb);
            prev = pos; prevb = (int)lb;
        } else {
            const int t = (tk - UPF) / YUN;
            for (;;) {
                const u64 mk = (NYT <= 64 || t < 64) ? ry0 : ry1;
                if (bad || ((mk >> (t & 63)) & 1)) break;
                const unsigned f0 = ld_sys(wsr + OFF_YF + (p * 128 + lane) * 4), f1 = NYT > 64 ? ld_sys(wsr + OFF_YF + (p * 128 + 64 + lane) * 4) : 0u;
                ry0 = __builtin_amdgcn_ballot_w64(f0 >= call); ry1 = __builtin_amdgcn_ballot_w64(f1 >= call);
                const u64 mk2 = (NYT <= 64 || t < 64) ? ry0 : ry1;
                if ((mk2 >> (t & 63)) & 1) break;
                WAIT_V(0); retire(prev, prevb); prev = -1;
                bad = give_up();
                __builtin_amdgcn_s_sleep(2);
            }
            const unsigned ib = (NYT <= 64 || t < 64) ? (unsigned)__builtin_amdgcn_readlane(tabY0, t & 63) : (unsigned)__builtin_amdgcn_readlane(tabY1, t & 63);
            const unsigned off = ib + (unsigned)((tk - UPF) % YUN) * (UC * CS_Y);
            if (bad) {
#pragma unroll
                for (int u = 0; u < UC; ++u) v[u] = (u32x4){0x7fc07fc0u, 0x7fc07fc0u, 0x7fc07fc0u, 0x7fc07fc0u};
            } else {
#pragma unroll
                for (int u = 0; u < UC; ++u) v[u] = __builtin_amdgcn_raw_buffer_load_b128(rs_out, vo_y + u * CS_Y, off, AUX_SYS);
            }
            WAIT_V(0);
#pragma unroll
            for (int u = 0; u < UC; ++u) __builtin_amdgcn_raw_buffer_store_b128(v[u], rd_out, vo_y + off + u * CS_Y, 0, AUX_AGT);
            retire(prev, prevb);
            prev = -1;
        }
    }
    if (lane == 0) st_sys(P.ws[p] + OFF_DONE + (r * 64 + sub * 8 + w) * 4, call);
    WAIT_V(0);
    retire(prev, prevb);
    STAMP(w, 255);
    if (blockIdx.x != 31 || w != 0) return;
    while (!bad) {
        bool ok = true;
#pragma unroll
        for (int k = 0; k < 7; ++k) {
            const int q = (r + 1 + k) & 7;
            const unsigned f = ld_sys(wsr + OFF_DONE + (q * 64 + lane) * 4);
            ok = ok && (lane >= QN * 8 || (lane & 7) >= NWV || f >= call);
        }
        if (__builtin_amdgcn_ballot_w64(ok) == ~0ull) break;
        bad = give_up();
        __builtin_amdgcn_s_sleep(1);
    }
    if (bad && lane == 0) st_sys(P.out[r] + (size_t)(8 * r * 256) * 2, 0x7fc07fc0u);
    STAMP(7, 1);
}

__global__ __launch_bounds__(512, 2) void gemm_ar_kernel(const Params P) {
    extern __shared__ __attribute__((aligned(16))) unsigned char shm[];
    LAS unsigned char* lds = (LAS unsigned char*)shm;
    const unsigned ctl = (unsigned)(size_t)lds + STAGE_BYTES;
    const int bid = blockIdx.x;
    char* const cptr = P.ws[P.rank] + OFF_CALL + bid * 64;
    const unsigned callv = ld_agent(cptr);
    if (threadIdx.x < CTL_BYTES / 4) lds_wr32(ctl + threadIdx.x * 4, 0u);
    const int role = bid < 32 ? P.role[bid] : -1;
    if (role >= 0) {
        const int p = role & 7, sub = role >> 4;
        if (NPUSH) {
            stage_x(P, mkrsrc(P.ws[p], WS_USED), 0, sub, callv);
            WAIT_V(0);
        }
        const unsigned call = (unsigned)RFL(callv) + 1u;
        __syncthreads();
        if (threadIdx.x == 0) {
            if (NPUSH) (void)__hip_atomic_fetch_add((unsigned*)(P.ws[p] + OFF_GOT + (posX(0) * 8 + P.rank) * 128), (unsigned)(PUN / 4), __ATOMIC_RELAXED, __HIP_MEMORY_SCOPE_SYSTEM);
            st_agent(cptr, call);
        }
        const int w = RFL(threadIdx.x >> 6);
        if (w < NWV) copy_wave(P, ctl, p, sub, w, call);
        else if (w == 7) dispatch_wave(P, ctl, p, call);
    } else {
        gemm_wg(P, lds, bid < 4 ? bid : bid - NC, callv);
    }
}

static inline int gemm_xcd(int g) { return (g < 4 ? g : g + NC) % 8; }

static void build_roles(Params& P) {
    const unsigned quad = 0x4Du;
    const int r = P.rank; int near[3], far[4], nn = 0, nf = 0;
    for (int k = 1; k < NR; ++k) { const int q = (r + k) & 7; if (((quad >> q) & 1) == ((quad >> r) & 1)) near[nn++] = q; else far[nf++] = q; }
    for (int b = 0; b < 32; ++b) {
        const int x = b & 7, k = b >> 3;
        if (b < 4) P.role[b] = -1;
        else if (x < 4) { const int idx = (k - 1) * 4 + x; P.role[b] = near[idx % 3] | ((idx / 3) << 4); }
        else P.role[b] = far[(k + x) & 3] | (k << 4);
    }
}

static void build_tiles(Params& P) {
    static_assert(NG / 8 >= 28, "block size");
    struct Blk { int r0, r1, c0, c1, av; } blk[64]; int nb = 0;
    for (int c0 = 0; c0 < NW; c0 += 7) {
        const int c1 = imin(c0 + 7, NW), rows = imin(16, 4 * (7 / (c1 - c0)));
        for (int r0 = 0; r0 < NXP; r0 += rows) { const int r1 = imin(r0 + rows, NXP); blk[nb++] = Blk{r0, r1, c0, c1, imax(posX(r1 - 1), posW(c1 - 1))}; }
    }
    for (int i = 1; i < nb; ++i) for (int j = i; j > 0 && blk[j].av < blk[j - 1].av; --j) { const Blk x = blk[j]; blk[j] = blk[j - 1]; blk[j - 1] = x; }
    unsigned t[NG][4]; int cnt[NG];
    for (int g = 0; g < NG; ++g) { for (int k = 0; k < 4; ++k) t[g][k] = 0xffffu; cnt[g] = 0; }
    double freeat[8] = {0, 0, 0, 0, 0, 0, 0, 0};
    for (int k = 0; k < nb; ++k) {
        int xg = 0;
        for (int x = 1; x < 8; ++x) if (freeat[x] < freeat[xg]) xg = x;
        const double av = 5.0 + 4.3 * (blk[k].av + 1);
        freeat[xg] = (freeat[xg] > av ? freeat[xg] : av) + 84.0;
        int wgs[NG / 8 + 1], G = 0;
        for (int pass = 3; pass >= 0; --pass)
            for (int g = 0; g < NG; ++g) if (gemm_xcd(g) == xg && cnt[g] == pass) wgs[G++] = g;
        int j = 0;
        for (int i = blk[k].c0; i < blk[k].c1; ++i)
            for (int m = blk[k].r0; m < blk[k].r1; ++m) { const int g = wgs[j++]; t[g][cnt[g]++] = (unsigned)((m << 12) | (i << 6) | blk[k].av); }
    }
    for (int g = 0; g < NG; ++g) P.tiles[g] = (u64)t[g][0] | ((u64)t[g][1] << 16) | ((u64)t[g][2] << 32) | ((u64)t[g][3] << 48);
}

}

extern "C" void kernel_launch(int rank, int nranks, ncclComm_t comm,
                              hipStream_t stream,
                              void* const* d_in, const int* in_sizes, int n_in,
                              void* d_out, int out_size,
                              const int* in_dtypes, int out_dtype,
                              void* const* d_out_all,
                              void* const* d_ws_all, size_t ws_bytes) {
    using namespace gar;
    (void)comm; (void)in_sizes; (void)in_dtypes; (void)out_dtype; (void)d_out;
    if (nranks != NR || n_in < 2 || (unsigned)rank >= (unsigned)NR || ws_bytes < WS_USED || (size_t)out_size != (size_t)MM * NN) {
        fprintf(stderr, "gemm_ar: unexpected configuration\n");
        return;
    }
    static bool attr_set[64] = {};
    int dev = 0;
    HIP_CHECK(hipGetDevice(&dev));
    if (dev >= 0 && dev < 64 && !attr_set[dev]) {
        HIP_CHECK(hipFuncSetAttribute((const void*)gemm_ar_kernel, hipFuncAttributeMaxDynamicSharedMemorySize, LDS_BYTES));
        attr_set[dev] = true;
    }
    Params P;
    P.x = (const unsigned short*)d_in[0];
    P.w = (const unsigned short*)d_in[1];
    for (int p = 0; p < NR; ++p) { P.ws[p] = (char*)d_ws_all[p]; P.out[p] = (char*)d_out_all[p]; }
    P.rank = rank;
    build_roles(P);
    build_tiles(P);
    hipLaunchKernelGGL(gemm_ar_kernel, dim3(NWG), dim3(512), LDS_BYTES, stream, P);
    HIP_CHECK(hipGetLastError());
}
```

```cpp
#include <hip/hip_runtime.h>
#include <rccl/rccl.h>
#include <cstdio>
#include <cstdint>

#define HIP_CHECK(expr) do { hipError_t _e = (expr); if (_e != hipSuccess) { \
    fprintf(stderr, "HIP error %s at %s:%d\n", hipGetErrorString(_e), __FILE__, __LINE__); \
    return; } } while (0)

#define CFG_NA 4
#define CFG_PUSH 1
#define CFG_UC 4
#define CFG_NWV 6
#define POLL_SLEEP 32
#define STAGE_DELAY 100
#define PUSH_DELAY 32
#define CFG_BAT 8

namespace gar {

#define LAS __attribute__((address_space(3)))
typedef short bf16x8 __attribute__((ext_vector_type(8)));
typedef float f32x4 __attribute__((ext_vector_type(4)));
typedef unsigned u32x4 __attribute__((ext_vector_type(4)));
typedef unsigned long long u64;

constexpr int NR = 8;
constexpr int MM = 4096, NN = 16384, KK = 4096, KS = KK / NR;
constexpr int NXP = MM / 256;
constexpr int NWP = NN / 256;
constexpr int NA = CFG_NA;
constexpr int NB = NWP - NR * NA;
constexpr int NW = NA + NB;
constexpr int NP = NXP + NW;
constexpr int NYT = NXP * NA;
constexpr int QN = 4;
constexpr int NPUSH = CFG_PUSH;
static_assert(NPUSH == 0 || NPUSH == 1, "push");
constexpr int NWG = 256, NC = 7 * QN, NG = NWG - NC;
constexpr int NTILES = NXP * NW;
constexpr int MAXT = 4;
constexpr int NUNITS = (NXP + NWP) * 4;
constexpr int UC = CFG_UC;
constexpr int NWV = CFG_NWV;
static_assert(NP <= 64 && NYT <= 128 && NA >= 1 && NA <= 8, "flag masks");
constexpr int BAT = CFG_BAT;
static_assert(QN >= 1 && QN <= 8 && 64 % UC == 0 && NWV >= 1 && NWV <= 7 && BAT >= NWV && (128 / UC) % BAT == 0, "copy split");
static_assert(NTILES <= 3 * NG + NG / 2, "tile table");

constexpr unsigned KiB = 1024u, MiB = 1024u * 1024u;
constexpr unsigned SLICE_B = 256 * KiB;
constexpr unsigned PANEL_B = 2 * MiB;
constexpr unsigned OFF_CALL = 0;
constexpr unsigned OFF_TMO = 16 * KiB;
constexpr unsigned OFF_TK = 20 * KiB;
constexpr unsigned OFF_INF = 32 * KiB;
constexpr unsigned OFF_YF = 48 * KiB;
constexpr unsigned OFF_DONE = 56 * KiB;
constexpr unsigned OFF_GOT = 64 * KiB;
constexpr unsigned OFF_XG = 1 * MiB;
constexpr unsigned OFF_WG = OFF_XG + (NXP + 1) * PANEL_B;
constexpr unsigned WS_USED = OFF_WG + NWP * PANEL_B;
constexpr unsigned OUT_B = (unsigned)MM * NN * 2;
constexpr int PUN = 256 / UC, YUN = 128 / UC;
constexpr int UPF = NP * PUN, UTF = UPF + NYT * YUN;
constexpr int UT0 = NPUSH * PUN;
constexpr unsigned TK_PER_CALL = UTF - UT0 + QN * BAT;
constexpr u64 DEADLINE = 100000000ull;

constexpr int HTB = 128 * 64 * 2;
constexpr int STAGE_BYTES = 8 * HTB;
constexpr int CTL_BYTES = 256;
constexpr int LDS_BYTES = STAGE_BYTES + CTL_BYTES;
constexpr int CTL_HOOK = 0;
constexpr int CTL_ACNT = 32;
constexpr int CTL_L = 0;
constexpr int CTL_GB = 64;
constexpr int CTL_CNT = 128;

__host__ __device__ constexpr int imin(int a, int b) { return a < b ? a : b; }
__host__ __device__ constexpr int imax(int a, int b) { return a > b ? a : b; }
static_assert(NA <= 7 && NW >= 14, "pull order");
__host__ __device__ constexpr int posX(int m) { return m < 4 ? m : (m < 8 ? m + 7 : m + 14); }
__host__ __device__ constexpr int posW(int i) { return i < 7 ? i + 4 : (i < 14 ? i + 8 : i + 16); }
constexpr int NBS = 8 - NA;
__host__ __device__ inline int wpanel(int r, int i) { if (i < NA) return 8 * r + i; const int b = i - NA; return (b / NBS) * 8 + NA + b % NBS; }
__host__ __device__ inline void pos_item(int pos, bool& isw, int& idx) {
    if (pos < 4) { isw = false; idx = pos; }
    else if (pos < 11) { isw = true; idx = pos - 4; }
    else if (pos < 15) { isw = false; idx = pos - 7; }
    else if (pos < 22) { isw = true; idx = pos - 8; }
    else if (pos < 30) { isw = false; idx = pos - 14; }
    else { isw = true; idx = pos - 16; }
}

__device__ __forceinline__ unsigned xpanel_off(int m, unsigned call) { return OFF_XG + (unsigned)((m < NPUSH && (call & 1u)) ? NXP + m : m) * PANEL_B; }
#define T_M(t) ((int)((t) >> 12))
#define T_I(t) ((int)(((t) >> 6) & 63u))
#define T_S(t) ((int)((t) & 63u))
struct Params {
    const unsigned short* x;
    const unsigned short* w;
    char* wsr;
    char* outr;
    int rank, pad;
    char* ws[NR];
    char* out[NR];
    char* cws[32];
    char* cout[32];
    int role[32];
    u64 tiles[NG];
};

#define STAMP(wv, idx) do { } while (0)
#define WAIT_V(n) asm volatile("s_waitcnt vmcnt(" #n ")" ::: "memory")
#define WAIT_L(n) asm volatile("s_waitcnt lgkmcnt(" #n ")" ::: "memory")
#define BAR __builtin_amdgcn_s_barrier()
#define SCHED __builtin_amdgcn_sched_barrier(0)
#define RFL(x) __builtin_amdgcn_readfirstlane(x)

__device__ __forceinline__ unsigned lds_rd32(unsigned a) { unsigned v; asm volatile("ds_read_b32 %0, %1\n\ts_waitcnt lgkmcnt(0)" : "=&v"(v) : "v"(a) : "memory"); return v; }
__device__ __forceinline__ u64 lds_rd64(unsigned a) { u64 v; asm volatile("ds_read_b64 %0, %1\n\ts_waitcnt lgkmcnt(0)" : "=&v"(v) : "v"(a) : "memory"); return v; }
__device__ __forceinline__ void lds_wr32(unsigned a, unsigned v) { asm volatile("ds_write_b32 %0, %1\n\ts_waitcnt lgkmcnt(0)" :: "v"(a), "v"(v) : "memory"); }
__device__ __forceinline__ void lds_wr64(unsigned a, u64 v) { asm volatile("ds_write_b64 %0, %1\n\ts_waitcnt lgkmcnt(0)" :: "v"(a), "v"(v) : "memory"); }
__device__ __forceinline__ unsigned lds_add_rtn(unsigned a, unsigned v) { unsigned o; asm volatile("ds_add_rtn_u32 %0, %1, %2\n\ts_waitcnt lgkmcnt(0)" : "=&v"(o) : "v"(a), "v"(v) : "memory"); return o; }
__device__ __forceinline__ u64 rfl64(u64 v) { return ((u64)(unsigned)RFL((unsigned)(v >> 32)) << 32) | (unsigned)RFL((unsigned)v); }
__device__ __forceinline__ unsigned ld_sys(const char* p) { return __hip_atomic_load((const unsigned*)p, __ATOMIC_RELAXED, __HIP_MEMORY_SCOPE_SYSTEM); }
__device__ __forceinline__ unsigned ld_agent(const char* p) { return __hip_atomic_load((const unsigned*)p, __ATOMIC_RELAXED, __HIP_MEMORY_SCOPE_AGENT); }
__device__ __forceinline__ void st_sys(char* p, unsigned v) { __hip_atomic_store((unsigned*)p, v, __ATOMIC_RELAXED, __HIP_MEMORY_SCOPE_SYSTEM); }
__device__ __forceinline__ void st_agent(char* p, unsigned v) { __hip_atomic_store((unsigned*)p, v, __ATOMIC_RELAXED, __HIP_MEMORY_SCOPE_AGENT); }
__device__ __forceinline__ void add_agent(char* p, unsigned v) { (void)__hip_atomic_fetch_add((unsigned*)p, v, __ATOMIC_RELAXED, __HIP_MEMORY_SCOPE_AGENT); }
__device__ __forceinline__ unsigned cvt_pk_bf16(float lo, float hi) { unsigned r; asm volatile("v_cvt_pk_bf16_f32 %0, %1, %2" : "=v"(r) : "v"(lo), "v"(hi)); return r; }
__device__ __forceinline__ char* uptr(char* p) { return (char*)rfl64((u64)p); }
__device__ __forceinline__ __amdgpu_buffer_rsrc_t mkrsrc(const void* p, unsigned bytes) {
    const u64 a = (u64)p;
    const u64 ua = ((u64)(unsigned)RFL((unsigned)(a >> 32)) << 32) | (unsigned)RFL((unsigned)a);
    return __builtin_amdgcn_make_buffer_rsrc((void*)ua, 0, bytes, 0x00020000);
}
constexpr int AUX_SYS = 17;
constexpr int AUX_AGT = 16;

__device__ __forceinline__ int lds_byte(int r, int c) { const int st = (r >> 4) * 2 + (c >> 5), rr = r & 15, cc = c & 31, ob = rr * 64 + cc * 2; return st * 1024 + (ob ^ (((ob >> 9) & 1) << 5)); }

template <int NSUB>
__device__ __forceinline__ void stage_x_load(const Params& P, u32x4 (&v)[NSUB], int panel, int quarter, int q0) {
    const int lane = threadIdx.x & 63;
    const int rr = lane >> 2, cc = (lane & 3) ^ (((lane >> 5) & 1) << 1);
#pragma unroll
    for (int e = 0; e < NSUB; ++e) {
        const int q = q0 + e, bidx = q >> 4, st = q & 15;
        const int kt = 2 * quarter + (bidx >> 1), h = bidx & 1;
        const int row = 256 * panel + 128 * h + (st >> 1) * 16 + rr, k = kt * 64 + (st & 1) * 32 + cc * 8;
        v[e] = *(const u32x4*)(P.x + (size_t)row * KS + k);
    }
}
template <int NSUB>
__device__ __forceinline__ void stage_x_store(const Params& P, const u32x4 (&v)[NSUB], __amdgpu_buffer_rsrc_t rd, int panel, int quarter, int q0, unsigned call) {
    const int lane = threadIdx.x & 63;
    const unsigned dst0 = xpanel_off(panel, call) + (unsigned)P.rank * SLICE_B + (unsigned)quarter * 64 * KiB + (unsigned)q0 * KiB;
#pragma unroll
    for (int e = 0; e < NSUB; ++e) __builtin_amdgcn_raw_buffer_store_b128(v[e], rd, dst0 + lane * 16 + e * 1024, 0, AUX_SYS);
}

__device__ __forceinline__ void stage_unit(const Params& P, int u, unsigned callv) {
    const int r = P.rank, tid = threadIdx.x, lane = tid & 63, wv = RFL(tid >> 6);
    bool isw; int panel, quarter, li = 0, dstrank = -1;
    if (u < 16) { isw = false; panel = u >> 2; quarter = u & 3; if (panel < NPUSH) dstrank = r; }
    else if (u < 16 + 32 * NA) {
        const int v = u - 16; isw = true; li = v >> 5; dstrank = (v >> 2) & 7; panel = 8 * dstrank + li; quarter = v & 3;
    } else {
        const int u2 = u - 16 - 32 * NA; int idx; pos_item(4 + NA + (u2 >> 2), isw, idx); quarter = u2 & 3;
        if (isw) { li = idx; panel = wpanel(r, idx); } else panel = idx;
    }
    const __amdgpu_buffer_rsrc_t rd = mkrsrc(P.wsr, WS_USED);
    if (!isw) {
        u32x4 v[8];
        stage_x_load<8>(P, v, panel, quarter, wv * 8);
        stage_x_store<8>(P, v, rd, panel, quarter, wv * 8, (unsigned)RFL(callv) + 1u);
    } else {
        const int cc = lane & 3, ng = lane >> 2;
        const int ktl = wv >> 2, chalf = (wv >> 1) & 1, nhalf = wv & 1;
        const int k0 = 128 * quarter + 64 * ktl + 32 * chalf + 8 * cc, c0 = 128 * nhalf + 8 * ng;
        u32x4 in[8];
#pragma unroll
        for (int j = 0; j < 8; ++j) in[j] = *(const u32x4*)(P.w + (size_t)(k0 + j) * NN + 256 * panel + c0);
        const int wc = c0 >> 6, bj = (c0 >> 5) & 1, a = (c0 >> 3) & 3;
        const unsigned dst0 = OFF_WG + (unsigned)panel * PANEL_B + (unsigned)r * SLICE_B + (unsigned)(2 * quarter + ktl) * 32 * KiB;
#pragma unroll
        for (int e = 0; e < 8; ++e) {
            u32x4 o;
#pragma unroll
            for (int d = 0; d < 4; ++d) {
                const unsigned lo = in[2 * d][e >> 1], hi = in[2 * d + 1][e >> 1];
                o[d] = (e & 1) ? __builtin_amdgcn_perm(hi, lo, 0x07060302u) : __builtin_amdgcn_perm(hi, lo, 0x05040100u);
            }
            const int R = 32 * wc + 16 * (e >> 2) + 4 * a + (e & 3), st = (R >> 4) * 2 + chalf, rr = R & 15;
            const unsigned off = (unsigned)(bj * 16 * KiB + st * 1024 + rr * 64 + ((cc ^ ((rr >> 3) << 1)) * 16));
            __builtin_amdgcn_raw_buffer_store_b128(o, rd, dst0 + off, 0, AUX_SYS);
        }
    }
    WAIT_V(0);
    __syncthreads();
    if (wv == 0) {
        const unsigned call = (unsigned)RFL(callv) + 1u;
        const int pos = isw ? posW(li) : posX(panel);
        if (lane < NR) {
            const int q = lane;
            if (q != r && (dstrank < 0 || dstrank == q)) st_sys(P.ws[q] + OFF_INF + ((r * 64 + pos) * 4 + quarter) * 4, call);
            if (q == r && (dstrank < 0 || dstrank == r)) add_agent(P.wsr + OFF_GOT + (pos * 8 + r) * 128, (unsigned)(PUN / 4));
        }
    }
}

__device__ __forceinline__ void gemm_wg(const Params& P, LAS unsigned char* lds, int g, unsigned callv) {
    const int r = P.rank;
    if (threadIdx.x < 64) STAMP(0, 0);
    const int pb = (int)blockIdx.x - 64;
    if (NPUSH && pb >= 4 && pb < 32) {
        const int wv = RFL(threadIdx.x >> 6), sub = P.role[pb] >> 4;
        char* const wsp = P.cws[pb];
        u32x4 xv[8];
        stage_x_load<8>(P, xv, 0, sub, wv * 8);
        const unsigned c1 = (unsigned)RFL(callv) + 1u;
        WAIT_V(0);
        __builtin_amdgcn_s_sleep(PUSH_DELAY);
        stage_x_store<8>(P, xv, mkrsrc(wsp, WS_USED), 0, sub, wv * 8, c1);
        WAIT_V(0);
        __syncthreads();
        if (threadIdx.x == 0)
            (void)__hip_atomic_fetch_add((unsigned*)(wsp + OFF_GOT + (posX(0) * 8 + r) * 128), (unsigned)(PUN / 4), __ATOMIC_RELAXED, __HIP_MEMORY_SCOPE_SYSTEM);
        if (threadIdx.x < 64) STAMP(1, 1);
    } else if (g >= 16) {
        __builtin_amdgcn_s_sleep(STAGE_DELAY); __builtin_amdgcn_s_sleep(STAGE_DELAY);
    }
    for (int u = g; u < NUNITS; u += NG) { stage_unit(P, u, callv); if (u == g && threadIdx.x < 64) STAMP(0, 200); }
    if (threadIdx.x < 64) STAMP(0, 1);
    const unsigned call = (unsigned)RFL(callv) + 1u;
    __syncthreads();
    if (threadIdx.x == 0) st_agent(P.wsr + OFF_CALL + blockIdx.x * 64, call);

    const int tid = threadIdx.x, wid = RFL(tid >> 6), lane = tid & 63, wr = wid >> 2, wc = wid & 3, fr = lane & 15, fq = lane >> 4;
    const unsigned ctl = (unsigned)(size_t)lds + STAGE_BYTES;
    char* const wsr = uptr(P.wsr);
    const unsigned target = (unsigned)PUN * call;
    const unsigned voff0 = (unsigned)tid * 16u;
    constexpr size_t kstep = 32 * KiB, hstep = 16 * KiB;
    const unsigned ldsw = (unsigned)wid * 1024u;
    const int aoff = lds_byte(wr * 64 + fr, fq * 8), boff = lds_byte(wc * 32 + fr, fq * 8);
#define G_SA(b, h) (((b) * 2 + (h)) * HTB)
#define G_SB(b, h) ((4 + (b) * 2 + (h)) * HTB)
#define G_STAGE(bufoff, gbase) do { _Pragma("unroll") for (int _i = 0; _i < 2; ++_i) \
        __builtin_amdgcn_global_load_lds((const unsigned*)((const char*)(gbase) + voff0 + _i * 8192), (LAS unsigned*)(lds + (bufoff) + ldsw + _i * 8192), 16, 0, 0); } while (0)
#define G_LDA(dst, b, h) do { _Pragma("unroll") for (int m = 0; m < 4; ++m) _Pragma("unroll") for (int k = 0; k < 2; ++k) dst[m][k] = *(const LAS bf16x8*)(lds + G_SA(b, h) + aoff + m * 2048 + k * 1024); } while (0)
#define G_LDB(dst, b, h) do { _Pragma("unroll") for (int n = 0; n < 2; ++n) _Pragma("unroll") for (int k = 0; k < 2; ++k) dst[n][k] = *(const LAS bf16x8*)(lds + G_SB(b, h) + boff + n * 2048 + k * 1024); } while (0)
#define G_MMA(ai, bj, At, Bt) do { __builtin_amdgcn_s_setprio(1); _Pragma("unroll") for (int m = 0; m < 4; ++m) _Pragma("unroll") for (int n = 0; n < 2; ++n) _Pragma("unroll") for (int k = 0; k < 2; ++k) \
        acc[ai][bj][m][n] = __builtin_amdgcn_mfma_f32_16x16x32_bf16(Bt[n][k], At[m][k], acc[ai][bj][m][n], 0, 0, 0); __builtin_amdgcn_s_setprio(0); } while (0)

    int ui = 0, nhook = 0;
    const u64 mytiles = rfl64(P.tiles[g]);
    unsigned cur = (unsigned)(mytiles & 0xffffu);
    if (cur == 0xffffu) return;
    bool bad = false;
    auto wait_counters = [&](unsigned off, unsigned want) {
        if (wid == 0) {
            const u64 t0 = __builtin_amdgcn_s_memrealtime();
            unsigned res = 1;
            for (;;) {
                const unsigned v = ld_agent(wsr + off);
                if (__builtin_amdgcn_ballot_w64(v >= want) == ~0ull) break;
                if (RFL(ld_agent(wsr + OFF_TMO)) == call || __builtin_amdgcn_s_memrealtime() - t0 > DEADLINE) {
                    if (lane == 0) st_agent(wsr + OFF_TMO, call);
                    res = 2; break;
                }
                __builtin_amdgcn_s_sleep(POLL_SLEEP);
            }
            lds_wr32(ctl + CTL_HOOK + (nhook & 7) * 4, res);
        }
        asm volatile("" ::: "memory"); BAR; asm volatile("" ::: "memory");
        if (RFL(lds_rd32(ctl + CTL_HOOK + (nhook & 7) * 4)) != 1u) bad = true;
        ++nhook;
    };
    f32x4 acc[2][2][4][2];
    bf16x8 At[4][2], B0[2][2], B1[2][2];
    constexpr int nt = KK / 64;
    const size_t k0off = (size_t)(8 * r) * kstep;
    wait_counters(OFF_GOT + (unsigned)(((lane & 1) ? posX(T_M(cur)) : posW(T_I(cur))) * 8 + r) * 128u, target);
    const char* cA = wsr + xpanel_off(T_M(cur), call);
    const char* cB = wsr + OFF_WG + (size_t)wpanel(r, T_I(cur)) * PANEL_B;
    G_STAGE(G_SB(0, 0), cB + k0off); G_STAGE(G_SA(0, 0), cA + k0off); G_STAGE(G_SB(0, 1), cB + k0off + hstep); G_STAGE(G_SA(0, 1), cA + k0off + hstep);
    if (wr == 1) BAR;
    WAIT_V(4); BAR;
    G_STAGE(G_SB(1, 0), cB + k0off + kstep); G_STAGE(G_SA(1, 0), cA + k0off + kstep); G_STAGE(G_SB(1, 1), cB + k0off + hstep + kstep);
    WAIT_V(6); BAR;
    for (;;) {
#pragma unroll
        for (int a = 0; a < 2; ++a)
#pragma unroll
            for (int b = 0; b < 2; ++b)
#pragma unroll
                for (int m = 0; m < 4; ++m)
#pragma unroll
                    for (int n = 0; n < 2; ++n) acc[a][b][m][n] = (f32x4){0.f, 0.f, 0.f, 0.f};
        const unsigned nxt = (ui + 1 < 4) ? (unsigned)((mytiles >> (16 * (ui + 1))) & 0xffffu) : 0xffffu;
        const bool has_next = nxt != 0xffffu;
        const char* nA = has_next ? wsr + xpanel_off(T_M(nxt), call) : cA;
        const char* nB = has_next ? wsr + OFF_WG + (size_t)wpanel(r, T_I(nxt)) * PANEL_B : cB;
        for (int t = 0; t < nt; t += 2) {
            const bool last = (t == nt - 2);
            if (t == 6) {
                const unsigned nx2 = has_next ? nxt : cur;
                int pos = (lane & 16) ? T_S(cur) : ((lane & 8) ? posX(T_M(cur)) : posW(T_I(cur))), src = lane & 7;
                if (lane >= 24) { pos = (lane & 1) ? posX(T_M(nx2)) : posW(T_I(nx2)); src = r; }
                wait_counters(OFF_GOT + (unsigned)(pos * 8 + src) * 128u, target);
                if (wid == 0) STAMP(0, 2 + 2 * ui);
            }
            const size_t o0 = (size_t)((t + 8 * r) & 63) * kstep, o2 = (size_t)((t + 2 + 8 * r) & 63) * kstep;
            const char* a1 = cA + o0 + kstep;
            const char* a2 = (last ? nA : cA) + o2; const char* b2 = (last ? nB : cB) + o2;
            const char* a3 = a2 + kstep; const char* b3 = b2 + kstep;
            G_LDB(B0, 0, 0); SCHED; G_LDA(At, 0, 0); G_STAGE(G_SA(1, 1), a1 + hstep);
            WAIT_L(8); BAR; WAIT_L(0); G_MMA(0, 0, At, B0); BAR; SCHED;
            G_LDB(B1, 0, 1); G_STAGE(G_SB(0, 0), b2);
            BAR; WAIT_L(0); G_MMA(0, 1, At, B1); BAR;
            G_LDA(At, 0, 1); G_STAGE(G_SA(0, 0), a2);
            BAR; WAIT_L(0); G_MMA(1, 0, At, B0); BAR; SCHED;
            G_STAGE(G_SB(0, 1), b2 + hstep);
            WAIT_V(6); BAR; G_MMA(1, 1, At, B1); BAR;
            G_LDB(B0, 1, 0); SCHED; G_LDA(At, 1, 0); G_STAGE(G_SA(0, 1), a2 + hstep);
            WAIT_L(8); BAR; WAIT_L(0); G_MMA(0, 0, At, B0); BAR; SCHED;
            G_LDB(B1, 1, 1); G_STAGE(G_SB(1, 0), b3);
            BAR; WAIT_L(0); G_MMA(0, 1, At, B1); BAR;
            G_LDA(At, 1, 1); G_STAGE(G_SA(1, 0), a3);
            BAR; WAIT_L(0); G_MMA(1, 0, At, B0); BAR; SCHED;
            G_STAGE(G_SB(1, 1), b3 + hstep);
            WAIT_V(6); BAR; G_MMA(1, 1, At, B1); BAR;
        }
        {
            int tz = threadIdx.x; asm volatile("" : "+v"(tz));
            const int l2 = tz & 63, fr2 = l2 & 15, fq2 = l2 >> 4;
            const int m0 = T_M(cur), li = T_I(cur), pn = wpanel(r, li);
            const __amdgpu_buffer_rsrc_t ro = mkrsrc(P.outr, OUT_B);
            const unsigned voff = (unsigned)(fr2 * (NN * 2) + fq2 * 16) + (unsigned)((m0 * 256 + wr * 64) * (NN * 2) + (pn * 256 + wc * 64) * 2);
#pragma unroll
            for (int ai = 0; ai < 2; ++ai)
#pragma unroll
                for (int m = 0; m < 4; ++m)
#pragma unroll
                    for (int bj = 0; bj < 2; ++bj) {
                        const f32x4 v0 = acc[ai][bj][m][0], v1 = acc[ai][bj][m][1];
                        u32x4 o; o.x = cvt_pk_bf16(v0[0], v0[1]); o.y = cvt_pk_bf16(v0[2], v0[3]); o.z = cvt_pk_bf16(v1[0], v1[1]); o.w = cvt_pk_bf16(v1[2], v1[3]);
                        if (bad) o = (u32x4){0x7fc07fc0u, 0x7fc07fc0u, 0x7fc07fc0u, 0x7fc07fc0u};
                        __builtin_amdgcn_raw_buffer_store_b128(o, ro, voff + (unsigned)((ai * 128 + m * 16) * (NN * 2) + bj * 64), 0, AUX_SYS);
                    }
            if (li < NA) {
                WAIT_V(0);
                unsigned old = 0;
                if (l2 == 0) old = lds_add_rtn(ctl + CTL_ACNT + (ui & 7) * 4, 1u);
                if (RFL(old) == 7u && l2 < NR && l2 != r) st_sys(P.ws[l2] + OFF_YF + (r * 128 + m0 * NA + li) * 4, call);
            }
        }
        if (wid == 0) STAMP(0, 3 + 2 * ui);
        ++ui;
        if (!has_next) break;
        cur = nxt; cA = nA; cB = nB;
    }
    WAIT_V(0);
    if (wr == 0) BAR;
}

__device__ __forceinline__ void dispatch_wave(const Params& P, unsigned ctl, int p, unsigned call) {
    const int r = P.rank, lane = threadIdx.x & 63;
    unsigned* const tkp = (unsigned*)(P.wsr + OFF_TK + p * 128);
    const unsigned tk0 = (call - 1u) * TK_PER_CALL;
    for (unsigned b = 0;; ++b) {
        while ((unsigned)RFL(lds_rd32(ctl + CTL_L)) / BAT + 2u < b) __builtin_amdgcn_s_sleep(2);
        unsigned o = 0;
        if (lane == 0) o = __hip_atomic_fetch_add(tkp, (unsigned)BAT, __ATOMIC_RELAXED, __HIP_MEMORY_SCOPE_AGENT);
        const unsigned base = (unsigned)RFL(o) - tk0 + (unsigned)UT0;
        if (lane == 0) lds_wr64(ctl + CTL_GB + (b & 7) * 8, ((u64)(b + 1u) << 32) | base);
        if (b == 0) STAMP(1, 5);
        if (base >= (unsigned)UTF) break;
    }
}

__device__ __forceinline__ void copy_wave(const Params& P, unsigned ctl, int p, int sub, int w, unsigned call) {
    const int r = P.rank, lane = threadIdx.x & 63;
    char* const wsr = P.wsr;
    char* const wsp = P.cws[blockIdx.x];
    const __amdgpu_buffer_rsrc_t rs_ws = mkrsrc(wsp, WS_USED), rd_ws = mkrsrc(wsr, WS_USED);
    const __amdgpu_buffer_rsrc_t rs_out = mkrsrc(P.cout[blockIdx.x], OUT_B), rd_out = mkrsrc(P.outr, OUT_B);
    const unsigned vo_p = (unsigned)lane * 16u, vo_y = (unsigned)(lane >> 5) * (NN * 2) + (unsigned)(lane & 31) * 16u;
    constexpr unsigned CS_P = KiB, CS_Y = 2u * NN * 2u;
    unsigned tabP, tabY0, tabY1;
    {
        bool isw; int idx; pos_item(lane < NP ? lane : 0, isw, idx);
        tabP = (isw ? OFF_WG + (unsigned)wpanel(r, idx) * PANEL_B : OFF_XG + (unsigned)idx * PANEL_B) + (unsigned)p * SLICE_B;
        const int t0 = lane, t1 = lane + 64;
        tabY0 = (unsigned)(256 * (t0 / NA)) * (unsigned)(NN * 2) + (unsigned)(8 * p + t0 % NA) * 512u;
        tabY1 = (unsigned)(256 * (t1 / NA)) * (unsigned)(NN * 2) + (unsigned)(8 * p + t1 % NA) * 512u;
    }
    u64 rin = 0, ry0 = 0, ry1 = 0;
    const u64 t0 = __builtin_amdgcn_s_memrealtime();
    int prev = -1, prevb = 0;
    auto retire = [&](int pos, int lb) {
        if (pos >= 0) {
            unsigned old = 0;
            if (lane == 0) old = lds_add_rtn(ctl + CTL_CNT + (lb & 15) * 4, 1u);
            if (((unsigned)RFL(old) + 1u) % BAT == 0u && lane == 0) add_agent(wsr + OFF_GOT + (pos * 8 + p) * 128, (unsigned)BAT);
        }
    };
    auto give_up = [&]() -> bool {
        if (RFL(ld_agent(wsr + OFF_TMO)) == call || __builtin_amdgcn_s_memrealtime() - t0 > DEADLINE) { if (lane == 0) st_agent(wsr + OFF_TMO, call); return true; }
        return false;
    };
    bool bad = false;
    STAMP(w, 0);
    for (;;) {
        unsigned o = 0;
        if (lane == 0) o = lds_add_rtn(ctl + CTL_L, 1u);
        const unsigned l = (unsigned)RFL(o), lb = l / BAT;
        u64 gb;
        for (;;) {
            gb = rfl64(lds_rd64(ctl + CTL_GB + (lb & 7) * 8));
            if ((unsigned)(gb >> 32) == lb + 1u) break;
            __builtin_amdgcn_s_sleep(1);
        }
        const unsigned tku = (unsigned)gb + l % BAT;
        if (tku >= (unsigned)UTF) break;
        const int tk = (int)tku;
        u32x4 v[UC];
        if (tk < UPF) {
            const int pos = tk / PUN;
            while (!bad && !((rin >> pos) & 1)) {
                const u32x4 f = __builtin_amdgcn_raw_buffer_load_b128(rd_ws, (unsigned)lane * 16u, OFF_INF + (unsigned)p * 1024u, AUX_SYS);
                rin = __builtin_amdgcn_ballot_w64(f.x >= call && f.y >= call && f.z >= call && f.w >= call);
                if ((rin >> pos) & 1) break;
                WAIT_V(0); retire(prev, prevb); prev = -1;
                bad = give_up();
                __builtin_amdgcn_s_sleep(1);
            }
            if (bad) continue;
            const unsigned off = (unsigned)__builtin_amdgcn_readlane(tabP, pos) + (unsigned)(tk % PUN) * (UC * CS_P);
#pragma unroll
            for (int u = 0; u < UC; ++u) v[u] = __builtin_amdgcn_raw_buffer_load_b128(rs_ws, vo_p + u * CS_P, off, AUX_SYS);
            WAIT_V(0);
#pragma unroll
            for (int u = 0; u < UC; ++u) __builtin_amdgcn_raw_buffer_store_b128(v[u], rd_ws, vo_p + off + u * CS_P, 0, AUX_AGT);
            retire(prev, prevb);
            prev = pos; prevb = (int)lb;
        } else {
            const int t = (tk - UPF) / YUN;
            for (;;) {
                const u64 mk = (NYT <= 64 || t < 64) ? ry0 : ry1;
                if (bad || ((mk >> (t & 63)) & 1)) break;
                const unsigned f0 = ld_sys(wsr + OFF_YF + (p * 128 + lane) * 4), f1 = NYT > 64 ? ld_sys(wsr + OFF_YF + (p * 128 + 64 + lane) * 4) : 0u;
                ry0 = __builtin_amdgcn_ballot_w64(f0 >= call); ry1 = __builtin_amdgcn_ballot_w64(f1 >= call);
                const u64 mk2 = (NYT <= 64 || t < 64) ? ry0 : ry1;
                if ((mk2 >> (t & 63)) & 1) break;
                WAIT_V(0); retire(prev, prevb); prev = -1;
                bad = give_up();
                __builtin_amdgcn_s_sleep(2);
            }
            const unsigned ib = (NYT <= 64 || t < 64) ? (unsigned)__builtin_amdgcn_readlane(tabY0, t & 63) : (unsigned)__builtin_amdgcn_readlane(tabY1, t & 63);
            const unsigned off = ib + (unsigned)((tk - UPF) % YUN) * (UC * CS_Y);
            if (bad) {
#pragma unroll
                for (int u = 0; u < UC; ++u) v[u] = (u32x4){0x7fc07fc0u, 0x7fc07fc0u, 0x7fc07fc0u, 0x7fc07fc0u};
            } else {
#pragma unroll
                for (int u = 0; u < UC; ++u) v[u] = __builtin_amdgcn_raw_buffer_load_b128(rs_out, vo_y + u * CS_Y, off, AUX_SYS);
            }
            WAIT_V(0);
#pragma unroll
            for (int u = 0; u < UC; ++u) __builtin_amdgcn_raw_buffer_store_b128(v[u], rd_out, vo_y + off + u * CS_Y, 0, AUX_AGT);
            retire(prev, prevb);
            prev = -1;
        }
    }
    if (lane == 0) st_sys(wsp + OFF_DONE + (r * 64 + sub * 8 + w) * 4, call);
    WAIT_V(0);
    retire(prev, prevb);
    STAMP(w, 255);
    if (blockIdx.x != 31 || w != 0) return;
    while (!bad) {
        bool ok = true;
#pragma unroll
        for (int k = 0; k < 7; ++k) {
            const int q = (r + 1 + k) & 7;
            const unsigned f = ld_sys(wsr + OFF_DONE + (q * 64 + lane) * 4);
            ok = ok && (lane >= QN * 8 || (lane & 7) >= NWV || f >= call);
        }
        if (__builtin_amdgcn_ballot_w64(ok) == ~0ull) break;
        bad = give_up();
        __builtin_amdgcn_s_sleep(1);
    }
    if (bad && lane == 0) st_sys(P.outr + (size_t)(8 * r * 256) * 2, 0x7fc07fc0u);
    STAMP(7, 1);
}

__global__ __launch_bounds__(512, 2) void gemm_ar_kernel(const Params P) {
    extern __shared__ __attribute__((aligned(16))) unsigned char shm[];
    LAS unsigned char* lds = (LAS unsigned char*)shm;
    const unsigned ctl = (unsigned)(size_t)lds + STAGE_BYTES;
    const int bid = blockIdx.x;
    char* const cptr = P.wsr + OFF_CALL + bid * 64;
    const unsigned callv = ld_agent(cptr);
    if (threadIdx.x < CTL_BYTES / 4) lds_wr32(ctl + threadIdx.x * 4, 0u);
    const int role = bid < 32 ? P.role[bid] : -1;
    if (role >= 0) {
        if (threadIdx.x < 64) STAMP(1, 3);
        const int p = role & 7, sub = role >> 4;
        const int w = RFL(threadIdx.x >> 6);
        const unsigned call = (unsigned)RFL(callv) + 1u;
        WAIT_L(0); BAR;
        if (threadIdx.x == 448) st_agent(cptr, call);
        if (w < NWV) copy_wave(P, ctl, p, sub, w, call);
        else if (w == 7) dispatch_wave(P, ctl, p, call);
    } else {
        gemm_wg(P, lds, bid < 4 ? bid : bid - NC, callv);
    }
}

static inline int gemm_xcd(int g) { return (g < 4 ? g : g + NC) % 8; }

static void build_roles(Params& P) {
    const unsigned quad = 0x4Du;
    const int r = P.rank; int near[3], far[4], nn = 0, nf = 0;
    for (int k = 1; k < NR; ++k) { const int q = (r + k) & 7; if (((quad >> q) & 1) == ((quad >> r) & 1)) near[nn++] = q; else far[nf++] = q; }
    for (int b = 0; b < 32; ++b) {
        const int x = b & 7, k = b >> 3;
        if (b < 4) P.role[b] = -1;
        else if (x < 4) { const int idx = (k - 1) * 4 + x; P.role[b] = near[idx % 3] | ((idx / 3) << 4); }
        else P.role[b] = far[(k + x) & 3] | (k << 4);
        P.cws[b] = P.role[b] >= 0 ? P.ws[P.role[b] & 7] : nullptr;
        P.cout[b] = P.role[b] >= 0 ? P.out[P.role[b] & 7] : nullptr;
    }
}

static void build_tiles(Params& P) {
    static_assert(NG / 8 >= 28, "block size");
    struct Blk { int r0, r1, c0, c1, av; } blk[64]; int nb = 0;
    for (int c0 = 0; c0 < NW; c0 += 7) {
        const int c1 = imin(c0 + 7, NW), rows = imin(16, 4 * (7 / (c1 - c0)));
        for (int r0 = 0; r0 < NXP; r0 += rows) { const int r1 = imin(r0 + rows, NXP); blk[nb++] = Blk{r0, r1, c0, c1, imax(posX(r1 - 1), posW(c1 - 1))}; }
    }
    for (int i = 1; i < nb; ++i) for (int j = i; j > 0 && blk[j].av < blk[j - 1].av; --j) { const Blk x = blk[j]; blk[j] = blk[j - 1]; blk[j - 1] = x; }
    unsigned t[NG][4]; int cnt[NG];
    for (int g = 0; g < NG; ++g) { for (int k = 0; k < 4; ++k) t[g][k] = 0xffffu; cnt[g] = 0; }
    double freeat[8] = {0, 0, 0, 0, 0, 0, 0, 0};
    for (int k = 0; k < nb; ++k) {
        int xg = 0;
        for (int x = 1; x < 8; ++x) if (freeat[x] < freeat[xg]) xg = x;
        const double av = 5.0 + 4.3 * (blk[k].av + 1);
        freeat[xg] = (freeat[xg] + 10.0 > av ? freeat[xg] + 10.0 : av) + 74.0;
        int wgs[NG / 8 + 1], G = 0;
        for (int pass = 3; pass >= 0; --pass)
            for (int g = 0; g < NG; ++g) if (gemm_xcd(g) == xg && cnt[g] == pass) wgs[G++] = g;
        int j = 0;
        for (int i = blk[k].c0; i < blk[k].c1; ++i)
            for (int m = blk[k].r0; m < blk[k].r1; ++m) { const int g = wgs[j++]; t[g][cnt[g]++] = (unsigned)((m << 12) | (i << 6) | blk[k].av); }
    }
    for (int g = 0; g < NG; ++g) P.tiles[g] = (u64)t[g][0] | ((u64)t[g][1] << 16) | ((u64)t[g][2] << 32) | ((u64)t[g][3] << 48);
}

}

extern "C" void kernel_launch(int rank, int nranks, ncclComm_t comm,
                              hipStream_t stream,
                              void* const* d_in, const int* in_sizes, int n_in,
                              void* d_out, int out_size,
                              const int* in_dtypes, int out_dtype,
                              void* const* d_out_all,
                              void* const* d_ws_all, size_t ws_bytes) {
    using namespace gar;
    (void)comm; (void)in_sizes; (void)in_dtypes; (void)out_dtype; (void)d_out;
    if (nranks != NR || n_in < 2 || (unsigned)rank >= (unsigned)NR || ws_bytes < WS_USED || (size_t)out_size != (size_t)MM * NN) {
        fprintf(stderr, "gemm_ar: unexpected configuration\n");
        return;
    }
    static bool attr_set[64] = {};
    int dev = 0;
    HIP_CHECK(hipGetDevice(&dev));
    if (dev >= 0 && dev < 64 && !attr_set[dev]) {
        HIP_CHECK(hipFuncSetAttribute((const void*)gemm_ar_kernel, hipFuncAttributeMaxDynamicSharedMemorySize, LDS_BYTES));
        attr_set[dev] = true;
    }
    Params P;
    P.x = (const unsigned short*)d_in[0];
    P.w = (const unsigned short*)d_in[1];
    for (int p = 0; p < NR; ++p) { P.ws[p] = (char*)d_ws_all[p]; P.out[p] = (char*)d_out_all[p]; }
    P.rank = rank; P.pad = 0;
    P.wsr = P.ws[rank]; P.outr = P.out[rank];
    build_roles(P);
    build_tiles(P);
    hipLaunchKernelGGL(gemm_ar_kernel, dim3(NWG), dim3(512), LDS_BYTES, stream, P);
    HIP_CHECK(hipGetLastError());
}
```

```cpp
#include <hip/hip_runtime.h>
#include <rccl/rccl.h>
#include <cstdio>
#include <cstdint>

#define HIP_CHECK(expr) do { hipError_t _e = (expr); if (_e != hipSuccess) { \
    fprintf(stderr, "HIP error %s at %s:%d\n", hipGetErrorString(_e), __FILE__, __LINE__); \
    return; } } while (0)

#define CFG_NA 4
#define CFG_PUSH 1
#define CFG_UC 4
#define CFG_NWV 6
#define POLL_SLEEP 32
#define STAGE_DELAY 100
#define PUSH_DELAY 32
#define CFG_BAT 8

namespace gar {

#define LAS __attribute__((address_space(3)))
typedef short bf16x8 __attribute__((ext_vector_type(8)));
typedef float f32x4 __attribute__((ext_vector_type(4)));
typedef unsigned u32x4 __attribute__((ext_vector_type(4)));
typedef unsigned long long u64;

constexpr int NR = 8;
constexpr int MM = 4096, NN = 16384, KK = 4096, KS = KK / NR;
constexpr int NXP = MM / 256;
constexpr int NWP = NN / 256;
constexpr int NA = CFG_NA;
constexpr int NB = NWP - NR * NA;
constexpr int NW = NA + NB;
constexpr int NP = NXP + NW;
constexpr int NYT = NXP * NA;
constexpr int QN = 4;
constexpr int NPUSH = CFG_PUSH;
static_assert(NPUSH == 0 || NPUSH == 1, "push");
constexpr int NWG = 256, NC = 7 * QN, NG = NWG - NC;
constexpr int NTILES = NXP * NW;
constexpr int MAXT = 4;
constexpr int NUNITS = (NXP + NWP) * 4;
constexpr int UC = CFG_UC;
constexpr int NWV = CFG_NWV;
static_assert(NP <= 64 && NYT <= 128 && NA >= 1 && NA <= 8, "flag masks");
constexpr int BAT = CFG_BAT;
static_assert(QN >= 1 && QN <= 8 && 64 % UC == 0 && NWV >= 1 && NWV <= 7 && BAT >= NWV && (128 / UC) % BAT == 0, "copy split");
static_assert(8 * (NW / 7) <= 16 * 4, "tile table");

constexpr unsigned KiB = 1024u, MiB = 1024u * 1024u;
constexpr unsigned SLICE_B = 256 * KiB;
constexpr unsigned PANEL_B = 2 * MiB;
constexpr unsigned OFF_CALL = 0;
constexpr unsigned OFF_TMO = 16 * KiB;
constexpr unsigned OFF_TK = 20 * KiB;
constexpr unsigned OFF_INF = 32 * KiB;
constexpr unsigned OFF_YF = 48 * KiB;
constexpr unsigned OFF_DONE = 56 * KiB;
constexpr unsigned OFF_GOT = 64 * KiB;
constexpr unsigned OFF_XG = 1 * MiB;
constexpr unsigned OFF_WG = OFF_XG + (NXP + 1) * PANEL_B;
constexpr unsigned WS_USED = OFF_WG + NWP * PANEL_B;
constexpr unsigned OUT_B = (unsigned)MM * NN * 2;
constexpr int PUN = 256 / UC, YUN = 128 / UC;
constexpr int UPF = NP * PUN, UTF = UPF + NYT * YUN;
constexpr int UT0 = NPUSH * PUN;
constexpr unsigned TK_PER_CALL = UTF - UT0 + QN * BAT;
constexpr u64 DEADLINE = 100000000ull;

constexpr int HTB = 128 * 64 * 2;
constexpr int STAGE_BYTES = 8 * HTB;
constexpr int CTL_BYTES = 256;
constexpr int LDS_BYTES = STAGE_BYTES + CTL_BYTES;
constexpr int CTL_HOOK = 0;
constexpr int CTL_ACNT = 32;
constexpr int CTL_L = 0;
constexpr int CTL_GB = 64;
constexpr int CTL_CNT = 128;

__host__ __device__ constexpr int imin(int a, int b) { return a < b ? a : b; }
__host__ __device__ constexpr int imax(int a, int b) { return a > b ? a : b; }
static_assert(NA <= 7 && NW >= 15 && NW % 7 == 1, "pull order");
__host__ __device__ constexpr int posX(int m) { return m < 4 ? m : (m < 8 ? m + 8 : m + 15); }
__host__ __device__ constexpr int posWj(int j) { return j < 7 ? j + 5 : (j < 14 ? j + 9 : j + 17); }
__host__ __device__ constexpr int posW(int i) { return i == NA ? 4 : posWj(i < NA ? i : i - 1); }
constexpr int NBS = 8 - NA;
__host__ __device__ inline int wpanel(int r, int i) { if (i < NA) return 8 * r + i; const int b = i - NA; return (b / NBS) * 8 + NA + b % NBS; }
__host__ __device__ inline void pos_item(int pos, bool& isw, int& idx) {
    int j;
    if (pos < 4) { isw = false; idx = pos; return; }
    else if (pos == 4) { isw = true; idx = NA; return; }
    else if (pos < 12) j = pos - 5;
    else if (pos < 16) { isw = false; idx = pos - 8; return; }
    else if (pos < 23) j = pos - 9;
    else if (pos < 31) { isw = false; idx = pos - 15; return; }
    else j = pos - 17;
    isw = true; idx = j < NA ? j : j + 1;
}

__device__ __forceinline__ unsigned xpanel_off(int m, unsigned call) { return OFF_XG + (unsigned)((m < NPUSH && (call & 1u)) ? NXP + m : m) * PANEL_B; }
#define T_M(t) ((int)((t) >> 12))
#define T_I(t) ((int)(((t) >> 6) & 63u))
#define T_S(t) ((int)((t) & 63u))
struct Params {
    const unsigned short* x;
    const unsigned short* w;
    char* wsr;
    char* outr;
    int rank, pad;
    char* ws[NR];
    char* out[NR];
    char* cws[32];
    char* cout[32];
    int role[32];
    u64 tiles[NG];
};

#define STAMP(wv, idx) do { } while (0)
#define STAMPC(wv, idx) do { } while (0)
#define WAIT_V(n) asm volatile("s_waitcnt vmcnt(" #n ")" ::: "memory")
#define WAIT_L(n) asm volatile("s_waitcnt lgkmcnt(" #n ")" ::: "memory")
#define BAR __builtin_amdgcn_s_barrier()
#define SCHED __builtin_amdgcn_sched_barrier(0)
#define RFL(x) __builtin_amdgcn_readfirstlane(x)

__device__ __forceinline__ unsigned lds_rd32(unsigned a) { unsigned v; asm volatile("ds_read_b32 %0, %1\n\ts_waitcnt lgkmcnt(0)" : "=&v"(v) : "v"(a) : "memory"); return v; }
__device__ __forceinline__ u64 lds_rd64(unsigned a) { u64 v; asm volatile("ds_read_b64 %0, %1\n\ts_waitcnt lgkmcnt(0)" : "=&v"(v) : "v"(a) : "memory"); return v; }
__device__ __forceinline__ void lds_wr32(unsigned a, unsigned v) { asm volatile("ds_write_b32 %0, %1\n\ts_waitcnt lgkmcnt(0)" :: "v"(a), "v"(v) : "memory"); }
__device__ __forceinline__ void lds_wr64(unsigned a, u64 v) { asm volatile("ds_write_b64 %0, %1\n\ts_waitcnt lgkmcnt(0)" :: "v"(a), "v"(v) : "memory"); }
__device__ __forceinline__ unsigned lds_add_rtn(unsigned a, unsigned v) { unsigned o; asm volatile("ds_add_rtn_u32 %0, %1, %2\n\ts_waitcnt lgkmcnt(0)" : "=&v"(o) : "v"(a), "v"(v) : "memory"); return o; }
__device__ __forceinline__ u64 rfl64(u64 v) { return ((u64)(unsigned)RFL((unsigned)(v >> 32)) << 32) | (unsigned)RFL((unsigned)v); }
__device__ __forceinline__ unsigned ld_sys(const char* p) { return __hip_atomic_load((const unsigned*)p, __ATOMIC_RELAXED, __HIP_MEMORY_SCOPE_SYSTEM); }
__device__ __forceinline__ unsigned ld_agent(const char* p) { return __hip_atomic_load((const unsigned*)p, __ATOMIC_RELAXED, __HIP_MEMORY_SCOPE_AGENT); }
__device__ __forceinline__ void st_sys(char* p, unsigned v) { __hip_atomic_store((unsigned*)p, v, __ATOMIC_RELAXED, __HIP_MEMORY_SCOPE_SYSTEM); }
__device__ __forceinline__ void st_agent(char* p, unsigned v) { __hip_atomic_store((unsigned*)p, v, __ATOMIC_RELAXED, __HIP_MEMORY_SCOPE_AGENT); }
__device__ __forceinline__ void add_agent(char* p, unsigned v) { (void)__hip_atomic_fetch_add((unsigned*)p, v, __ATOMIC_RELAXED, __HIP_MEMORY_SCOPE_AGENT); }
__device__ __forceinline__ unsigned cvt_pk_bf16(float lo, float hi) { unsigned r; asm volatile("v_cvt_pk_bf16_f32 %0, %1, %2" : "=v"(r) : "v"(lo), "v"(hi)); return r; }
__device__ __forceinline__ char* uptr(char* p) { return (char*)rfl64((u64)p); }
__device__ __forceinline__ __amdgpu_buffer_rsrc_t mkrsrc(const void* p, unsigned bytes) {
    const u64 a = (u64)p;
    const u64 ua = ((u64)(unsigned)RFL((unsigned)(a >> 32)) << 32) | (unsigned)RFL((unsigned)a);
    return __builtin_amdgcn_make_buffer_rsrc((void*)ua, 0, bytes, 0x00020000);
}
constexpr int AUX_SYS = 17;
constexpr int AUX_AGT = 16;

__device__ __forceinline__ int lds_byte(int r, int c) { const int st = (r >> 4) * 2 + (c >> 5), rr = r & 15, cc = c & 31, ob = rr * 64 + cc * 2; return st * 1024 + (ob ^ (((ob >> 9) & 1) << 5)); }

template <int NSUB>
__device__ __forceinline__ void stage_x_load(const Params& P, u32x4 (&v)[NSUB], int panel, int quarter, int q0) {
    const int lane = threadIdx.x & 63;
    const int rr = lane >> 2, cc = (lane & 3) ^ (((lane >> 5) & 1) << 1);
#pragma unroll
    for (int e = 0; e < NSUB; ++e) {
        const int q = q0 + e, bidx = q >> 4, st = q & 15;
        const int kt = 2 * quarter + (bidx >> 1), h = bidx & 1;
        const int row = 256 * panel + 128 * h + (st >> 1) * 16 + rr, k = kt * 64 + (st & 1) * 32 + cc * 8;
        v[e] = *(const u32x4*)(P.x + (size_t)row * KS + k);
    }
}
template <int NSUB>
__device__ __forceinline__ void stage_x_store(const Params& P, const u32x4 (&v)[NSUB], __amdgpu_buffer_rsrc_t rd, int panel, int quarter, int q0, unsigned call) {
    const int lane = threadIdx.x & 63;
    const unsigned dst0 = xpanel_off(panel, call) + (unsigned)P.rank * SLICE_B + (unsigned)quarter * 64 * KiB + (unsigned)q0 * KiB;
#pragma unroll
    for (int e = 0; e < NSUB; ++e) __builtin_amdgcn_raw_buffer_store_b128(v[e], rd, dst0 + lane * 16 + e * 1024, 0, AUX_SYS);
}

__device__ __forceinline__ void stage_unit(const Params& P, int u, unsigned callv) {
    const int r = P.rank, tid = threadIdx.x, lane = tid & 63, wv = RFL(tid >> 6);
    bool isw; int panel, quarter, li = 0, dstrank = -1;
    if (u >= 20 && u < 20 + 32 * NA) {
        const int v = u - 20; isw = true; li = v >> 5; dstrank = (v >> 2) & 7; panel = 8 * dstrank + li; quarter = v & 3;
    } else {
        const int u2 = u < 20 ? u : u - 28 * NA; int idx; pos_item(u2 >> 2, isw, idx); quarter = u2 & 3;
        if (isw) { li = idx; panel = wpanel(r, idx); } else { panel = idx; if (panel < NPUSH) dstrank = r; }
    }
    const __amdgpu_buffer_rsrc_t rd = mkrsrc(P.wsr, WS_USED);
    if (!isw) {
        u32x4 v[8];
        stage_x_load<8>(P, v, panel, quarter, wv * 8);
        stage_x_store<8>(P, v, rd, panel, quarter, wv * 8, (unsigned)RFL(callv) + 1u);
    } else {
        const int cc = lane & 3, ng = lane >> 2;
        const int ktl = wv >> 2, chalf = (wv >> 1) & 1, nhalf = wv & 1;
        const int k0 = 128 * quarter + 64 * ktl + 32 * chalf + 8 * cc, c0 = 128 * nhalf + 8 * ng;
        u32x4 in[8];
#pragma unroll
        for (int j = 0; j < 8; ++j) in[j] = *(const u32x4*)(P.w + (size_t)(k0 + j) * NN + 256 * panel + c0);
        const int wc = c0 >> 6, bj = (c0 >> 5) & 1, a = (c0 >> 3) & 3;
        const unsigned dst0 = OFF_WG + (unsigned)panel * PANEL_B + (unsigned)r * SLICE_B + (unsigned)(2 * quarter + ktl) * 32 * KiB;
#pragma unroll
        for (int e = 0; e < 8; ++e) {
            u32x4 o;
#pragma unroll
            for (int d = 0; d < 4; ++d) {
                const unsigned lo = in[2 * d][e >> 1], hi = in[2 * d + 1][e >> 1];
                o[d] = (e & 1) ? __builtin_amdgcn_perm(hi, lo, 0x07060302u) : __builtin_amdgcn_perm(hi, lo, 0x05040100u);
            }
            const int R = 32 * wc + 16 * (e >> 2) + 4 * a + (e & 3), st = (R >> 4) * 2 + chalf, rr = R & 15;
            const unsigned off = (unsigned)(bj * 16 * KiB + st * 1024 + rr * 64 + ((cc ^ ((rr >> 3) << 1)) * 16));
            __builtin_amdgcn_raw_buffer_store_b128(o, rd, dst0 + off, 0, AUX_SYS);
        }
    }
    WAIT_V(0);
    __syncthreads();
    if (wv == 0) {
        const unsigned call = (unsigned)RFL(callv) + 1u;
        const int pos = isw ? posW(li) : posX(panel);
        if (lane < NR) {
            const int q = lane;
            if (q != r && (dstrank < 0 || dstrank == q)) st_sys(P.ws[q] + OFF_INF + ((r * 64 + pos) * 4 + quarter) * 4, call);
            if (q == r && (dstrank < 0 || dstrank == r)) add_agent(P.wsr + OFF_GOT + (pos * 8 + r) * 128, (unsigned)(PUN / 4));
        }
    }
}

__device__ __forceinline__ void gemm_wg(const Params& P, LAS unsigned char* lds, int g, unsigned callv) {
    const int r = P.rank;
    if (threadIdx.x < 64) STAMP(0, 0);
    const int pb = (int)blockIdx.x - 64;
    if (NPUSH && pb >= 4 && pb < 32) {
        const int wv = RFL(threadIdx.x >> 6), sub = P.role[pb] >> 4;
        char* const wsp = P.cws[pb];
        u32x4 xv[8];
        stage_x_load<8>(P, xv, 0, sub, wv * 8);
        const unsigned c1 = (unsigned)RFL(callv) + 1u;
        WAIT_V(0);
        __builtin_amdgcn_s_sleep(PUSH_DELAY);
        stage_x_store<8>(P, xv, mkrsrc(wsp, WS_USED), 0, sub, wv * 8, c1);
        WAIT_V(0);
        __syncthreads();
        if (threadIdx.x == 0)
            (void)__hip_atomic_fetch_add((unsigned*)(wsp + OFF_GOT + (posX(0) * 8 + r) * 128), (unsigned)(PUN / 4), __ATOMIC_RELAXED, __HIP_MEMORY_SCOPE_SYSTEM);
        if (threadIdx.x < 64) STAMP(1, 1);
    } else if (g >= 20) {
        __builtin_amdgcn_s_sleep(STAGE_DELAY); __builtin_amdgcn_s_sleep(STAGE_DELAY);
    }
    for (int u = g; u < NUNITS; u += NG) { stage_unit(P, u, callv); if (u == g && threadIdx.x < 64) STAMP(0, 200); }
    if (threadIdx.x < 64) STAMP(0, 1);
    const unsigned call = (unsigned)RFL(callv) + 1u;
    __syncthreads();
    if (threadIdx.x == 0) st_agent(P.wsr + OFF_CALL + blockIdx.x * 64, call);

    const int tid = threadIdx.x, wid = RFL(tid >> 6), lane = tid & 63, wr = wid >> 2, wc = wid & 3, fr = lane & 15, fq = lane >> 4;
    const unsigned ctl = (unsigned)(size_t)lds + STAGE_BYTES;
    char* const wsr = uptr(P.wsr);
    const unsigned target = (unsigned)PUN * call;
    const unsigned voff0 = (unsigned)tid * 16u;
    constexpr size_t kstep = 32 * KiB, hstep = 16 * KiB;
    const unsigned ldsw = (unsigned)wid * 1024u;
    const int aoff = lds_byte(wr * 64 + fr, fq * 8), boff = lds_byte(wc * 32 + fr, fq * 8);
#define G_SA(b, h) (((b) * 2 + (h)) * HTB)
#define G_SB(b, h) ((4 + (b) * 2 + (h)) * HTB)
#define G_STAGE(bufoff, gbase) do { _Pragma("unroll") for (int _i = 0; _i < 2; ++_i) \
        __builtin_amdgcn_global_load_lds((const unsigned*)((const char*)(gbase) + voff0 + _i * 8192), (LAS unsigned*)(lds + (bufoff) + ldsw + _i * 8192), 16, 0, 0); } while (0)
#define G_LDA(dst, b, h) do { _Pragma("unroll") for (int m = 0; m < 4; ++m) _Pragma("unroll") for (int k = 0; k < 2; ++k) dst[m][k] = *(const LAS bf16x8*)(lds + G_SA(b, h) + aoff + m * 2048 + k * 1024); } while (0)
#define G_LDB(dst, b, h) do { _Pragma("unroll") for (int n = 0; n < 2; ++n) _Pragma("unroll") for (int k = 0; k < 2; ++k) dst[n][k] = *(const LAS bf16x8*)(lds + G_SB(b, h) + boff + n * 2048 + k * 1024); } while (0)
#define G_MMA(ai, bj, At, Bt) do { __builtin_amdgcn_s_setprio(1); _Pragma("unroll") for (int m = 0; m < 4; ++m) _Pragma("unroll") for (int n = 0; n < 2; ++n) _Pragma("unroll") for (int k = 0; k < 2; ++k) \
        acc[ai][bj][m][n] = __builtin_amdgcn_mfma_f32_16x16x32_bf16(Bt[n][k], At[m][k], acc[ai][bj][m][n], 0, 0, 0); __builtin_amdgcn_s_setprio(0); } while (0)

    int ui = 0, nhook = 0;
    const u64 mytiles = rfl64(P.tiles[g]);
    unsigned cur = (unsigned)(mytiles & 0xffffu);
    if (cur == 0xffffu) return;
    bool bad = false;
    auto wait_counters = [&](unsigned off, unsigned want) {
        if (wid == 0) {
            const u64 t0 = __builtin_amdgcn_s_memrealtime();
            unsigned res = 1;
            for (;;) {
                const unsigned v = ld_agent(wsr + off);
                if (__builtin_amdgcn_ballot_w64(v >= want) == ~0ull) break;
                if (RFL(ld_agent(wsr + OFF_TMO)) == call || __builtin_amdgcn_s_memrealtime() - t0 > DEADLINE) {
                    if (lane == 0) st_agent(wsr + OFF_TMO, call);
                    res = 2; break;
                }
                __builtin_amdgcn_s_sleep(POLL_SLEEP);
            }
            lds_wr32(ctl + CTL_HOOK + (nhook & 7) * 4, res);
        }
        asm volatile("" ::: "memory"); BAR; asm volatile("" ::: "memory");
        if (RFL(lds_rd32(ctl + CTL_HOOK + (nhook & 7) * 4)) != 1u) bad = true;
        ++nhook;
    };
    f32x4 acc[2][2][4][2];
    bf16x8 At[4][2], B0[2][2], B1[2][2];
    constexpr int nt = KK / 64;
    const size_t k0off = (size_t)(8 * r) * kstep;
    wait_counters(OFF_GOT + (unsigned)(((lane & 1) ? posX(T_M(cur)) : posW(T_I(cur))) * 8 + r) * 128u, target);
    const char* cA = wsr + xpanel_off(T_M(cur), call);
    const char* cB = wsr + OFF_WG + (size_t)wpanel(r, T_I(cur)) * PANEL_B;
    G_STAGE(G_SB(0, 0), cB + k0off); G_STAGE(G_SA(0, 0), cA + k0off); G_STAGE(G_SB(0, 1), cB + k0off + hstep); G_STAGE(G_SA(0, 1), cA + k0off + hstep);
    if (wr == 1) BAR;
    WAIT_V(4); BAR;
    G_STAGE(G_SB(1, 0), cB + k0off + kstep); G_STAGE(G_SA(1, 0), cA + k0off + kstep); G_STAGE(G_SB(1, 1), cB + k0off + hstep + kstep);
    WAIT_V(6); BAR;
    for (;;) {
#pragma unroll
        for (int a = 0; a < 2; ++a)
#pragma unroll
            for (int b = 0; b < 2; ++b)
#pragma unroll
                for (int m = 0; m < 4; ++m)
#pragma unroll
                    for (int n = 0; n < 2; ++n) acc[a][b][m][n] = (f32x4){0.f, 0.f, 0.f, 0.f};
        const unsigned nxt = (ui + 1 < 4) ? (unsigned)((mytiles >> (16 * (ui + 1))) & 0xffffu) : 0xffffu;
        const bool has_next = nxt != 0xffffu;
        const char* nA = has_next ? wsr + xpanel_off(T_M(nxt), call) : cA;
        const char* nB = has_next ? wsr + OFF_WG + (size_t)wpanel(r, T_I(nxt)) * PANEL_B : cB;
        for (int t = 0; t < nt; t += 2) {
            const bool last = (t == nt - 2);
            if (t == 6) {
                const unsigned nx2 = has_next ? nxt : cur;
                int pos = (lane & 16) ? T_S(cur) : ((lane & 8) ? posX(T_M(cur)) : posW(T_I(cur))), src = lane & 7;
                if (lane >= 24) { pos = (lane & 1) ? posX(T_M(nx2)) : posW(T_I(nx2)); src = r; }
                wait_counters(OFF_GOT + (unsigned)(pos * 8 + src) * 128u, target);
                if (wid == 0) { STAMP(0, 2 + 2 * ui); STAMPC(2, 2 + 2 * ui); }
            }
            const size_t o0 = (size_t)((t + 8 * r) & 63) * kstep, o2 = (size_t)((t + 2 + 8 * r) & 63) * kstep;
            const char* a1 = cA + o0 + kstep;
            const char* a2 = (last ? nA : cA) + o2; const char* b2 = (last ? nB : cB) + o2;
            const char* a3 = a2 + kstep; const char* b3 = b2 + kstep;
            G_LDB(B0, 0, 0); SCHED; G_LDA(At, 0, 0); G_STAGE(G_SA(1, 1), a1 + hstep);
            WAIT_L(8); BAR; WAIT_L(0); G_MMA(0, 0, At, B0); BAR; SCHED;
            G_LDB(B1, 0, 1); G_STAGE(G_SB(0, 0), b2);
            BAR; WAIT_L(0); G_MMA(0, 1, At, B1); BAR;
            G_LDA(At, 0, 1); G_STAGE(G_SA(0, 0), a2);
            BAR; WAIT_L(0); G_MMA(1, 0, At, B0); BAR; SCHED;
            G_STAGE(G_SB(0, 1), b2 + hstep);
            WAIT_V(6); BAR; G_MMA(1, 1, At, B1); BAR;
            G_LDB(B0, 1, 0); SCHED; G_LDA(At, 1, 0); G_STAGE(G_SA(0, 1), a2 + hstep);
            WAIT_L(8); BAR; WAIT_L(0); G_MMA(0, 0, At, B0); BAR; SCHED;
            G_LDB(B1, 1, 1); G_STAGE(G_SB(1, 0), b3);
            BAR; WAIT_L(0); G_MMA(0, 1, At, B1); BAR;
            G_LDA(At, 1, 1); G_STAGE(G_SA(1, 0), a3);
            BAR; WAIT_L(0); G_MMA(1, 0, At, B0); BAR; SCHED;
            G_STAGE(G_SB(1, 1), b3 + hstep);
            WAIT_V(6); BAR; G_MMA(1, 1, At, B1); BAR;
        }
        {
            int tz = threadIdx.x; asm volatile("" : "+v"(tz));
            const int l2 = tz & 63, fr2 = l2 & 15, fq2 = l2 >> 4;
            const int m0 = T_M(cur), li = T_I(cur), pn = wpanel(r, li);
            const __amdgpu_buffer_rsrc_t ro = mkrsrc(P.outr, OUT_B);
            const unsigned voff = (unsigned)(fr2 * (NN * 2) + fq2 * 16) + (unsigned)((m0 * 256 + wr * 64) * (NN * 2) + (pn * 256 + wc * 64) * 2);
#pragma unroll
            for (int ai = 0; ai < 2; ++ai)
#pragma unroll
                for (int m = 0; m < 4; ++m)
#pragma unroll
                    for (int bj = 0; bj < 2; ++bj) {
                        const f32x4 v0 = acc[ai][bj][m][0], v1 = acc[ai][bj][m][1];
                        u32x4 o; o.x = cvt_pk_bf16(v0[0], v0[1]); o.y = cvt_pk_bf16(v0[2], v0[3]); o.z = cvt_pk_bf16(v1[0], v1[1]); o.w = cvt_pk_bf16(v1[2], v1[3]);
                        if (bad) o = (u32x4){0x7fc07fc0u, 0x7fc07fc0u, 0x7fc07fc0u, 0x7fc07fc0u};
                        __builtin_amdgcn_raw_buffer_store_b128(o, ro, voff + (unsigned)((ai * 128 + m * 16) * (NN * 2) + bj * 64), 0, AUX_SYS);
                    }
            if (li < NA) {
                WAIT_V(0);
                unsigned old = 0;
                if (l2 == 0) old = lds_add_rtn(ctl + CTL_ACNT + (ui & 7) * 4, 1u);
                if (RFL(old) == 7u && l2 < NR && l2 != r) st_sys(P.ws[l2] + OFF_YF + (r * 128 + m0 * NA + li) * 4, call);
            }
        }
        if (wid == 0) { STAMP(0, 3 + 2 * ui); STAMPC(2, 3 + 2 * ui); }
        ++ui;
        if (!has_next) break;
        cur = nxt; cA = nA; cB = nB;
    }
    WAIT_V(0);
    if (wr == 0) BAR;
}

__device__ __forceinline__ void dispatch_wave(const Params& P, unsigned ctl, int p, unsigned call) {
    const int r = P.rank, lane = threadIdx.x & 63;
    unsigned* const tkp = (unsigned*)(P.wsr + OFF_TK + p * 128);
    const unsigned tk0 = (call - 1u) * TK_PER_CALL;
    for (unsigned b = 0;; ++b) {
        while ((unsigned)RFL(lds_rd32(ctl + CTL_L)) / BAT + 2u < b) __builtin_amdgcn_s_sleep(2);
        unsigned o = 0;
        if (lane == 0) o = __hip_atomic_fetch_add(tkp, (unsigned)BAT, __ATOMIC_RELAXED, __HIP_MEMORY_SCOPE_AGENT);
        const unsigned base = (unsigned)RFL(o) - tk0 + (unsigned)UT0;
        if (lane == 0) lds_wr64(ctl + CTL_GB + (b & 7) * 8, ((u64)(b + 1u) << 32) | base);
        if (b == 0) STAMP(1, 5);
        if (base >= (unsigned)UTF) break;
    }
}

__device__ __forceinline__ void copy_wave(const Params& P, unsigned ctl, int p, int sub, int w, unsigned call) {
    const int r = P.rank, lane = threadIdx.x & 63;
    char* const wsr = P.wsr;
    char* const wsp = P.cws[blockIdx.x];
    const __amdgpu_buffer_rsrc_t rs_ws = mkrsrc(wsp, WS_USED), rd_ws = mkrsrc(wsr, WS_USED);
    const __amdgpu_buffer_rsrc_t rs_out = mkrsrc(P.cout[blockIdx.x], OUT_B), rd_out = mkrsrc(P.outr, OUT_B);
    const unsigned vo_p = (unsigned)lane * 16u, vo_y = (unsigned)(lane >> 5) * (NN * 2) + (unsigned)(lane & 31) * 16u;
    constexpr unsigned CS_P = KiB, CS_Y = 2u * NN * 2u;
    unsigned tabP, tabY0, tabY1;
    {
        bool isw; int idx; pos_item(lane < NP ? lane : 0, isw, idx);
        tabP = (isw ? OFF_WG + (unsigned)wpanel(r, idx) * PANEL_B : OFF_XG + (unsigned)idx * PANEL_B) + (unsigned)p * SLICE_B;
        const int t0 = lane, t1 = lane + 64;
        tabY0 = (unsigned)(256 * (t0 / NA)) * (unsigned)(NN * 2) + (unsigned)(8 * p + t0 % NA) * 512u;
        tabY1 = (unsigned)(256 * (t1 / NA)) * (unsigned)(NN * 2) + (unsigned)(8 * p + t1 % NA) * 512u;
    }
    u64 rin = 0, ry0 = 0, ry1 = 0;
    const u64 t0 = __builtin_amdgcn_s_memrealtime();
    int prev = -1, prevb = 0;
    auto retire = [&](int pos, int lb) {
        if (pos >= 0) {
            unsigned old = 0;
            if (lane == 0) old = lds_add_rtn(ctl + CTL_CNT + (lb & 15) * 4, 1u);
            if (((unsigned)RFL(old) + 1u) % BAT == 0u && lane == 0) add_agent(wsr + OFF_GOT + (pos * 8 + p) * 128, (unsigned)BAT);
        }
    };
    auto give_up = [&]() -> bool {
        if (RFL(ld_agent(wsr + OFF_TMO)) == call || __builtin_amdgcn_s_memrealtime() - t0 > DEADLINE) { if (lane == 0) st_agent(wsr + OFF_TMO, call); return true; }
        return false;
    };
    bool bad = false;
    STAMP(w, 0);
    for (;;) {
        unsigned o = 0;
        if (lane == 0) o = lds_add_rtn(ctl + CTL_L, 1u);
        const unsigned l = (unsigned)RFL(o), lb = l / BAT;
        u64 gb;
        for (;;) {
            gb = rfl64(lds_rd64(ctl + CTL_GB + (lb & 7) * 8));
            if ((unsigned)(gb >> 32) == lb + 1u) break;
            __builtin_amdgcn_s_sleep(1);
        }
        const unsigned tku = (unsigned)gb + l % BAT;
        if (tku >= (unsigned)UTF) break;
        const int tk = (int)tku;
        u32x4 v[UC];
        if (tk < UPF) {
            const int pos = tk / PUN;
            while (!bad && !((rin >> pos) & 1)) {
                const u32x4 f = __builtin_amdgcn_raw_buffer_load_b128(rd_ws, (unsigned)lane * 16u, OFF_INF + (unsigned)p * 1024u, AUX_SYS);
                rin = __builtin_amdgcn_ballot_w64(f.x >= call && f.y >= call && f.z >= call && f.w >= call);
                if ((rin >> pos) & 1) break;
                WAIT_V(0); retire(prev, prevb); prev = -1;
                bad = give_up();
                __builtin_amdgcn_s_sleep(1);
            }
            if (bad) continue;
            const unsigned off = (unsigned)__builtin_amdgcn_readlane(tabP, pos) + (unsigned)(tk % PUN) * (UC * CS_P);
#pragma unroll
            for (int u = 0; u < UC; ++u) v[u] = __builtin_amdgcn_raw_buffer_load_b128(rs_ws, vo_p + u * CS_P, off, AUX_SYS);
            WAIT_V(0);
#pragma unroll
            for (int u = 0; u < UC; ++u) __builtin_amdgcn_raw_buffer_store_b128(v[u], rd_ws, vo_p + off + u * CS_P, 0, AUX_AGT);
            retire(prev, prevb);
            prev = pos; prevb = (int)lb;
        } else {
            const int t = (tk - UPF) / YUN;
            for (;;) {
                const u64 mk = (NYT <= 64 || t < 64) ? ry0 : ry1;
                if (bad || ((mk >> (t & 63)) & 1)) break;
                const unsigned f0 = ld_sys(wsr + OFF_YF + (p * 128 + lane) * 4), f1 = NYT > 64 ? ld_sys(wsr + OFF_YF + (p * 128 + 64 + lane) * 4) : 0u;
                ry0 = __builtin_amdgcn_ballot_w64(f0 >= call); ry1 = __builtin_amdgcn_ballot_w64(f1 >= call);
                const u64 mk2 = (NYT <= 64 || t < 64) ? ry0 : ry1;
                if ((mk2 >> (t & 63)) & 1) break;
                WAIT_V(0); retire(prev, prevb); prev = -1;
                bad = give_up();
                __builtin_amdgcn_s_sleep(2);
            }
            const unsigned ib = (NYT <= 64 || t < 64) ? (unsigned)__builtin_amdgcn_readlane(tabY0, t & 63) : (unsigned)__builtin_amdgcn_readlane(tabY1, t & 63);
            const unsigned off = ib + (unsigned)((tk - UPF) % YUN) * (UC * CS_Y);
            if (bad) {
#pragma unroll
                for (int u = 0; u < UC; ++u) v[u] = (u32x4){0x7fc07fc0u, 0x7fc07fc0u, 0x7fc07fc0u, 0x7fc07fc0u};
            } else {
#pragma unroll
                for (int u = 0; u < UC; ++u) v[u] = __builtin_amdgcn_raw_buffer_load_b128(rs_out, vo_y + u * CS_Y, off, AUX_SYS);
            }
            WAIT_V(0);
#pragma unroll
            for (int u = 0; u < UC; ++u) __builtin_amdgcn_raw_buffer_store_b128(v[u], rd_out, vo_y + off + u * CS_Y, 0, AUX_AGT);
            retire(prev, prevb);
            prev = -1;
        }
    }
    if (lane == 0) st_sys(wsp + OFF_DONE + (r * 64 + sub * 8 + w) * 4, call);
    WAIT_V(0);
    retire(prev, prevb);
    STAMP(w, 255);
    if (blockIdx.x != 31 || w != 0) return;
    while (!bad) {
        bool ok = true;
#pragma unroll
        for (int k = 0; k < 7; ++k) {
            const int q = (r + 1 + k) & 7;
            const unsigned f = ld_sys(wsr + OFF_DONE + (q * 64 + lane) * 4);
            ok = ok && (lane >= QN * 8 || (lane & 7) >= NWV || f >= call);
        }
        if (__builtin_amdgcn_ballot_w64(ok) == ~0ull) break;
        bad = give_up();
        __builtin_amdgcn_s_sleep(1);
    }
    if (bad && lane == 0) st_sys(P.outr + (size_t)(8 * r * 256) * 2, 0x7fc07fc0u);
    STAMP(7, 1);
}

__global__ __launch_bounds__(512, 2) void gemm_ar_kernel(const Params P) {
    extern __shared__ __attribute__((aligned(16))) unsigned char shm[];
    LAS unsigned char* lds = (LAS unsigned char*)shm;
    const unsigned ctl = (unsigned)(size_t)lds + STAGE_BYTES;
    const int bid = blockIdx.x;
    char* const cptr = P.wsr + OFF_CALL + bid * 64;
    const unsigned callv = ld_agent(cptr);
    if (threadIdx.x < CTL_BYTES / 4) lds_wr32(ctl + threadIdx.x * 4, 0u);
    const int role = bid < 32 ? P.role[bid] : -1;
    if (role >= 0) {
        if (threadIdx.x < 64) STAMP(1, 3);
        const int p = role & 7, sub = role >> 4;
        const int w = RFL(threadIdx.x >> 6);
        const unsigned call = (unsigned)RFL(callv) + 1u;
        WAIT_L(0); BAR;
        if (threadIdx.x == 448) st_agent(cptr, call);
        if (w < NWV) copy_wave(P, ctl, p, sub, w, call);
        else if (w == 7) dispatch_wave(P, ctl, p, call);
    } else {
        gemm_wg(P, lds, bid < 4 ? bid : bid - NC, callv);
    }
}

static inline int gemm_xcd(int g) { return (g < 4 ? g : g + NC) % 8; }

static void build_roles(Params& P) {
    const unsigned quad = 0x4Du;
    const int r = P.rank; int near[3], far[4], nn = 0, nf = 0;
    for (int k = 1; k < NR; ++k) { const int q = (r + k) & 7; if (((quad >> q) & 1) == ((quad >> r) & 1)) near[nn++] = q; else far[nf++] = q; }
    for (int b = 0; b < 32; ++b) {
        const int x = b & 7, k = b >> 3;
        if (b < 4) P.role[b] = -1;
        else if (x < 4) { const int idx = (k - 1) * 4 + x; P.role[b] = near[idx % 3] | ((idx / 3) << 4); }
        else P.role[b] = far[(k + x) & 3] | (k << 4);
        P.cws[b] = P.role[b] >= 0 ? P.ws[P.role[b] & 7] : nullptr;
        P.cout[b] = P.role[b] >= 0 ? P.out[P.role[b] & 7] : nullptr;
    }
}

static void build_tiles(Params& P) {
    static_assert(NG == 8 * 28 + 4, "groups");
    struct Blk { int r0, j0, av; } blk[8 * (NW / 7)]; int nb = 0;
    for (int j0 = 0; j0 + 7 <= NW - 1; j0 += 7)
        for (int r0 = 0; r0 < NXP; r0 += 2) blk[nb++] = Blk{r0, j0, imax(posX(r0 + 1), posWj(j0 + 6))};
    for (int i = 1; i < nb; ++i) for (int j = i; j > 0 && blk[j].av < blk[j - 1].av; --j) { const Blk x = blk[j]; blk[j] = blk[j - 1]; blk[j - 1] = x; }
    unsigned t[NG][4]; int cnt[NG];
    for (int g = 0; g < NG; ++g) { for (int k = 0; k < 4; ++k) t[g][k] = 0xffffu; cnt[g] = 0; }
    int wgs[8][29], nw[8] = {0, 0, 0, 0, 0, 0, 0, 0};
    for (int g = 0; g < NG; ++g) { const int x = gemm_xcd(g); wgs[x][nw[x]++] = g; }
    double freeat[16];
    for (int k = 0; k < 16; ++k) freeat[k] = 0;
    for (int k = 0; k < nb; ++k) {
        int gr = 0;
        for (int x = 1; x < 16; ++x) if (freeat[x] < freeat[gr]) gr = x;
        const double av = 9.0 + 4.33 * (blk[k].av + 1);
        freeat[gr] = (freeat[gr] + 10.0 > av ? freeat[gr] + 10.0 : av) + 77.0;
        int n = 0;
        for (int j = blk[k].j0; j < blk[k].j0 + 7; ++j)
            for (int m = blk[k].r0; m < blk[k].r0 + 2; ++m) {
                const int g = wgs[gr & 7][(gr >> 3) * 14 + n++], i = j < NA ? j : j + 1;
                if (cnt[g] < 4) t[g][cnt[g]++] = (unsigned)((m << 12) | (i << 6) | blk[k].av);
            }
    }
    int sp = 0;
    for (int x = 0; x < 8; ++x)
        if (nw[x] == 29) { const int g = wgs[x][28]; for (int k = 0; k < 4; ++k) { const int m = sp + 4 * k; t[g][k] = (unsigned)((m << 12) | (NA << 6) | imax(posX(m), posW(NA))); } ++sp; }
    for (int g = 0; g < NG; ++g) P.tiles[g] = (u64)t[g][0] | ((u64)t[g][1] << 16) | ((u64)t[g][2] << 32) | ((u64)t[g][3] << 48);
}

}

extern "C" void kernel_launch(int rank, int nranks, ncclComm_t comm,
                              hipStream_t stream,
                              void* const* d_in, const int* in_sizes, int n_in,
                              void* d_out, int out_size,
                              const int* in_dtypes, int out_dtype,
                              void* const* d_out_all,
                              void* const* d_ws_all, size_t ws_bytes) {
    using namespace gar;
    (void)comm; (void)in_sizes; (void)in_dtypes; (void)out_dtype; (void)d_out;
    if (nranks != NR || n_in < 2 || (unsigned)rank >= (unsigned)NR || ws_bytes < WS_USED || (size_t)out_size != (size_t)MM * NN) {
        fprintf(stderr, "gemm_ar: unexpected configuration\n");
        return;
    }
    static bool attr_set[64] = {};
    int dev = 0;
    HIP_CHECK(hipGetDevice(&dev));
    if (dev >= 0 && dev < 64 && !attr_set[dev]) {
        HIP_CHECK(hipFuncSetAttribute((const void*)gemm_ar_kernel, hipFuncAttributeMaxDynamicSharedMemorySize, LDS_BYTES));
        attr_set[dev] = true;
    }
    Params P;
    P.x = (const unsigned short*)d_in[0];
    P.w = (const unsigned short*)d_in[1];
    for (int p = 0; p < NR; ++p) { P.ws[p] = (char*)d_ws_all[p]; P.out[p] = (char*)d_out_all[p]; }
    P.rank = rank; P.pad = 0;
    P.wsr = P.ws[rank]; P.outr = P.out[rank];
    build_roles(P);
    build_tiles(P);
    hipLaunchKernelGGL(gemm_ar_kernel, dim3(NWG), dim3(512), LDS_BYTES, stream, P);
    HIP_CHECK(hipGetLastError());
}
```

```cpp
#include <hip/hip_runtime.h>
#include <rccl/rccl.h>
#include <cstdio>
#include <cstdint>

#define HIP_CHECK(expr) do { hipError_t _e = (expr); if (_e != hipSuccess) { \
    fprintf(stderr, "HIP error %s at %s:%d\n", hipGetErrorString(_e), __FILE__, __LINE__); \
    return; } } while (0)

#define CFG_NA 3
#define CFG_PUSH 1
#define CFG_UC 4
#define CFG_NWV 6
#define POLL_SLEEP 32
#define STAGE_DELAY 100
#define PUSH_DELAY 32
#define CFG_BAT 8

namespace gar {

#define LAS __attribute__((address_space(3)))
typedef short bf16x8 __attribute__((ext_vector_type(8)));
typedef float f32x4 __attribute__((ext_vector_type(4)));
typedef unsigned u32x4 __attribute__((ext_vector_type(4)));
typedef unsigned long long u64;

constexpr int NR = 8;
constexpr int MM = 4096, NN = 16384, KK = 4096, KS = KK / NR;
constexpr int NXP = MM / 256;
constexpr int NWP = NN / 256;
constexpr int NA = CFG_NA;
constexpr int NB = NWP - NR * NA;
constexpr int NW = NA + NB;
constexpr int NP = NXP + NW;
constexpr int NYT = NXP * NA;
constexpr int QN = 4;
constexpr int NPUSH = CFG_PUSH;
static_assert(NPUSH == 0 || NPUSH == 1, "push");
constexpr int NWG = 256, NC = 7 * QN, NG = NWG - NC;
constexpr int NTILES = NXP * NW;
constexpr int MAXT = 4;
constexpr int NUNITS = (NXP + NWP) * 4;
constexpr int UC = CFG_UC;
constexpr int NWV = CFG_NWV;
static_assert(NP <= 64 && NYT <= 128 && NA >= 1 && NA <= 8, "flag masks");
constexpr int BAT = CFG_BAT;
static_assert(QN >= 1 && QN <= 8 && 64 % UC == 0 && NWV >= 1 && NWV <= 7 && BAT >= NWV && (64 / UC) % BAT == 0, "copy split");
static_assert(NW / 7 <= 8, "tile table");

constexpr unsigned KiB = 1024u, MiB = 1024u * 1024u;
constexpr unsigned SLICE_B = 256 * KiB;
constexpr unsigned PANEL_B = 2 * MiB;
constexpr unsigned OFF_CALL = 0;
constexpr unsigned OFF_TMO = 16 * KiB;
constexpr unsigned OFF_TK = 20 * KiB;
constexpr unsigned OFF_INF = 32 * KiB;
constexpr unsigned OFF_YF = 48 * KiB;
constexpr unsigned OFF_DONE = 56 * KiB;
constexpr unsigned OFF_GOT = 64 * KiB;
constexpr unsigned OFF_XG = 1 * MiB;
constexpr unsigned OFF_WG = OFF_XG + (NXP + 1) * PANEL_B;
constexpr unsigned WS_USED = OFF_WG + NWP * PANEL_B;
constexpr unsigned OUT_B = (unsigned)MM * NN * 2;
constexpr int QUN = 64 / UC;
constexpr int PUN = 256 / UC, YUN = 128 / UC;
constexpr int UPF = NP * PUN, UTF = UPF + NYT * YUN;
constexpr int UT0 = NPUSH * PUN;
constexpr unsigned TK_PER_CALL = UTF - UT0 + QN * BAT;
constexpr u64 DEADLINE = 100000000ull;

constexpr int HTB = 128 * 64 * 2;
constexpr int STAGE_BYTES = 8 * HTB;
constexpr int CTL_BYTES = 256;
constexpr int LDS_BYTES = STAGE_BYTES + CTL_BYTES;
constexpr int CTL_HOOK = 0;
constexpr int CTL_ACNT = 32;
constexpr int CTL_L = 0;
constexpr int CTL_GB = 64;
constexpr int CTL_CNT = 128;

__host__ __device__ constexpr int imin(int a, int b) { return a < b ? a : b; }
__host__ __device__ constexpr int imax(int a, int b) { return a > b ? a : b; }
static_assert(NA <= 7 && NW >= 15 && NW % 7 == 1, "pull order");
__host__ __device__ constexpr int posX(int m) { return m < 4 ? m : m + 1; }
__host__ __device__ constexpr int posWj(int j) { return 17 + j; }
__host__ __device__ constexpr int posW(int i) { return i == NA ? 4 : posWj(i < NA ? i : i - 1); }
constexpr int NBS = 8 - NA;
__host__ __device__ inline int wpanel(int r, int i) { if (i < NA) return 8 * r + i; const int b = i - NA; return (b / NBS) * 8 + NA + b % NBS; }
__host__ __device__ inline void pos_item(int pos, bool& isw, int& idx) {
    if (pos >= 17) { const int j = pos - 17; isw = true; idx = j < NA ? j : j + 1; }
    else if (pos == 4) { isw = true; idx = NA; }
    else { isw = false; idx = pos < 4 ? pos : pos - 1; }
}
__device__ __forceinline__ void quarter_item(int qi, int& pos, int& q) {
    if (qi < 20) { pos = qi >> 2; q = qi & 3; }
    else if (qi < 124) { const int k = qi - 20; q = k / 26; pos = 5 + k % 26; }
    else {
        const int k = qi - 124, ph = k / 56, k2 = k % 56, base = 31 + 14 * ph;
        if ((NP - 31) % 14 == 0 || base + 14 <= NP) { q = k2 / 14; pos = base + k2 % 14; }
        else { q = k2 / 7; pos = base + k2 % 7; }
    }
}

__device__ __forceinline__ unsigned xpanel_off(int m, unsigned call) { return OFF_XG + (unsigned)((m < NPUSH && (call & 1u)) ? NXP + m : m) * PANEL_B; }
#define T_M(t) ((int)((t) >> 12))
#define T_I(t) ((int)(((t) >> 6) & 63u))
#define T_S(t) ((int)((t) & 63u))
struct Params {
    const unsigned short* x;
    const unsigned short* w;
    char* wsr;
    char* outr;
    int rank, pad;
    char* ws[NR];
    char* out[NR];
    char* cws[32];
    char* cout[32];
    int role[32];
    u64 tiles[NG];
};

#define STAMP(wv, idx) do { } while (0)
#define STAMPC(wv, idx) do { } while (0)
#define WAIT_V(n) asm volatile("s_waitcnt vmcnt(" #n ")" ::: "memory")
#define WAIT_L(n) asm volatile("s_waitcnt lgkmcnt(" #n ")" ::: "memory")
#define BAR __builtin_amdgcn_s_barrier()
#define SCHED __builtin_amdgcn_sched_barrier(0)
#define RFL(x) __builtin_amdgcn_readfirstlane(x)

__device__ __forceinline__ unsigned lds_rd32(unsigned a) { unsigned v; asm volatile("ds_read_b32 %0, %1\n\ts_waitcnt lgkmcnt(0)" : "=&v"(v) : "v"(a) : "memory"); return v; }
__device__ __forceinline__ u64 lds_rd64(unsigned a) { u64 v; asm volatile("ds_read_b64 %0, %1\n\ts_waitcnt lgkmcnt(0)" : "=&v"(v) : "v"(a) : "memory"); return v; }
__device__ __forceinline__ void lds_wr32(unsigned a, unsigned v) { asm volatile("ds_write_b32 %0, %1\n\ts_waitcnt lgkmcnt(0)" :: "v"(a), "v"(v) : "memory"); }
__device__ __forceinline__ void lds_wr64(unsigned a, u64 v) { asm volatile("ds_write_b64 %0, %1\n\ts_waitcnt lgkmcnt(0)" :: "v"(a), "v"(v) : "memory"); }
__device__ __forceinline__ unsigned lds_add_rtn(unsigned a, unsigned v) { unsigned o; asm volatile("ds_add_rtn_u32 %0, %1, %2\n\ts_waitcnt lgkmcnt(0)" : "=&v"(o) : "v"(a), "v"(v) : "memory"); return o; }
__device__ __forceinline__ u64 rfl64(u64 v) { return ((u64)(unsigned)RFL((unsigned)(v >> 32)) << 32) | (unsigned)RFL((unsigned)v); }
__device__ __forceinline__ unsigned ld_sys(const char* p) { return __hip_atomic_load((const unsigned*)p, __ATOMIC_RELAXED, __HIP_MEMORY_SCOPE_SYSTEM); }
__device__ __forceinline__ unsigned ld_agent(const char* p) { return __hip_atomic_load((const unsigned*)p, __ATOMIC_RELAXED, __HIP_MEMORY_SCOPE_AGENT); }
__device__ __forceinline__ void st_sys(char* p, unsigned v) { __hip_atomic_store((unsigned*)p, v, __ATOMIC_RELAXED, __HIP_MEMORY_SCOPE_SYSTEM); }
__device__ __forceinline__ void st_agent(char* p, unsigned v) { __hip_atomic_store((unsigned*)p, v, __ATOMIC_RELAXED, __HIP_MEMORY_SCOPE_AGENT); }
__device__ __forceinline__ void add_agent(char* p, unsigned v) { (void)__hip_atomic_fetch_add((unsigned*)p, v, __ATOMIC_RELAXED, __HIP_MEMORY_SCOPE_AGENT); }
__device__ __forceinline__ unsigned cvt_pk_bf16(float lo, float hi) { unsigned r; asm volatile("v_cvt_pk_bf16_f32 %0, %1, %2" : "=v"(r) : "v"(lo), "v"(hi)); return r; }
__device__ __forceinline__ char* uptr(char* p) { return (char*)rfl64((u64)p); }
__device__ __forceinline__ __amdgpu_buffer_rsrc_t mkrsrc(const void* p, unsigned bytes) {
    const u64 a = (u64)p;
    const u64 ua = ((u64)(unsigned)RFL((unsigned)(a >> 32)) << 32) | (unsigned)RFL((unsigned)a);
    return __builtin_amdgcn_make_buffer_rsrc((void*)ua, 0, bytes, 0x00020000);
}
constexpr int AUX_SYS = 17;
constexpr int AUX_AGT = 16;

__device__ __forceinline__ int lds_byte(int r, int c) { const int st = (r >> 4) * 2 + (c >> 5), rr = r & 15, cc = c & 31, ob = rr * 64 + cc * 2; return st * 1024 + (ob ^ (((ob >> 9) & 1) << 5)); }

template <int NSUB>
__device__ __forceinline__ void stage_x_load(const Params& P, u32x4 (&v)[NSUB], int panel, int quarter, int q0) {
    const int lane = threadIdx.x & 63;
    const int rr = lane >> 2, cc = (lane & 3) ^ (((lane >> 5) & 1) << 1);
#pragma unroll
    for (int e = 0; e < NSUB; ++e) {
        const int q = q0 + e, bidx = q >> 4, st = q & 15;
        const int kt = 2 * quarter + (bidx >> 1), h = bidx & 1;
        const int row = 256 * panel + 128 * h + (st >> 1) * 16 + rr, k = kt * 64 + (st & 1) * 32 + cc * 8;
        v[e] = *(const u32x4*)(P.x + (size_t)row * KS + k);
    }
}
template <int NSUB>
__device__ __forceinline__ void stage_x_store(const Params& P, const u32x4 (&v)[NSUB], __amdgpu_buffer_rsrc_t rd, int panel, int quarter, int q0, unsigned call) {
    const int lane = threadIdx.x & 63;
    const unsigned dst0 = xpanel_off(panel, call) + (unsigned)P.rank * SLICE_B + (unsigned)quarter * 64 * KiB + (unsigned)q0 * KiB;
#pragma unroll
    for (int e = 0; e < NSUB; ++e) __builtin_amdgcn_raw_buffer_store_b128(v[e], rd, dst0 + lane * 16 + e * 1024, 0, AUX_SYS);
}

__device__ __forceinline__ void stage_unit(const Params& P, int u, unsigned callv) {
    const int r = P.rank, tid = threadIdx.x, lane = tid & 63, wv = RFL(tid >> 6);
    bool isw; int panel, quarter, li = 0, dstrank = -1;
    constexpr int S1 = 26 + 7 * NA;
    if (u < 20) { int idx; pos_item(u >> 2, isw, idx); quarter = u & 3; if (isw) li = idx; panel = isw ? wpanel(r, idx) : idx; if (!isw && panel < NPUSH) dstrank = r; }
    else if (u < 20 + 4 * S1) {
        const int k = u - 20, e = k % S1; quarter = k / S1;
        if (e < 12) { isw = false; panel = 4 + e; }
        else if (e < 12 + 8 * NA) { const int v = e - 12; isw = true; li = v >> 3; dstrank = v & 7; panel = 8 * dstrank + li; }
        else { isw = true; li = e - 12 - 7 * NA + 1; panel = wpanel(r, li); }
    } else { const int k = u - 20 - 4 * S1; isw = true; li = 15 + (k >> 2); quarter = k & 3; panel = wpanel(r, li); }
    const __amdgpu_buffer_rsrc_t rd = mkrsrc(P.wsr, WS_USED);
    if (!isw) {
        u32x4 v[8];
        stage_x_load<8>(P, v, panel, quarter, wv * 8);
        stage_x_store<8>(P, v, rd, panel, quarter, wv * 8, (unsigned)RFL(callv) + 1u);
    } else {
        const int cc = lane & 3, ng = lane >> 2;
        const int ktl = wv >> 2, chalf = (wv >> 1) & 1, nhalf = wv & 1;
        const int k0 = 128 * quarter + 64 * ktl + 32 * chalf + 8 * cc, c0 = 128 * nhalf + 8 * ng;
        u32x4 in[8];
#pragma unroll
        for (int j = 0; j < 8; ++j) in[j] = *(const u32x4*)(P.w + (size_t)(k0 + j) * NN + 256 * panel + c0);
        const int wc = c0 >> 6, bj = (c0 >> 5) & 1, a = (c0 >> 3) & 3;
        const unsigned dst0 = OFF_WG + (unsigned)panel * PANEL_B + (unsigned)r * SLICE_B + (unsigned)(2 * quarter + ktl) * 32 * KiB;
#pragma unroll
        for (int e = 0; e < 8; ++e) {
            u32x4 o;
#pragma unroll
            for (int d = 0; d < 4; ++d) {
                const unsigned lo = in[2 * d][e >> 1], hi = in[2 * d + 1][e >> 1];
                o[d] = (e & 1) ? __builtin_amdgcn_perm(hi, lo, 0x07060302u) : __builtin_amdgcn_perm(hi, lo, 0x05040100u);
            }
            const int R = 32 * wc + 16 * (e >> 2) + 4 * a + (e & 3), st = (R >> 4) * 2 + chalf, rr = R & 15;
            const unsigned off = (unsigned)(bj * 16 * KiB + st * 1024 + rr * 64 + ((cc ^ ((rr >> 3) << 1)) * 16));
            __builtin_amdgcn_raw_buffer_store_b128(o, rd, dst0 + off, 0, AUX_SYS);
        }
    }
    WAIT_V(0);
    __syncthreads();
    if (wv == 0) {
        const unsigned call = (unsigned)RFL(callv) + 1u;
        const int pos = isw ? posW(li) : posX(panel);
        if (lane < NR) {
            const int q = lane;
            if (q != r && (dstrank < 0 || dstrank == q)) st_sys(P.ws[q] + OFF_INF + ((r * 64 + pos) * 4 + quarter) * 4, call);
            if (q == r && (dstrank < 0 || dstrank == r)) add_agent(P.wsr + OFF_GOT + (pos * 8 + r) * 128 + quarter * 4, (unsigned)QUN);
        }
    }
}

__device__ __forceinline__ void gemm_wg(const Params& P, LAS unsigned char* lds, int g, unsigned callv) {
    const int r = P.rank;
    if (threadIdx.x < 64) STAMP(0, 0);
    const int pb = (int)blockIdx.x - 64;
    if (NPUSH && pb >= 4 && pb < 32) {
        const int wv = RFL(threadIdx.x >> 6), sub = P.role[pb] >> 4;
        char* const wsp = P.cws[pb];
        u32x4 xv[8];
        stage_x_load<8>(P, xv, 0, sub, wv * 8);
        const unsigned c1 = (unsigned)RFL(callv) + 1u;
        WAIT_V(0);
        __builtin_amdgcn_s_sleep(PUSH_DELAY);
        stage_x_store<8>(P, xv, mkrsrc(wsp, WS_USED), 0, sub, wv * 8, c1);
        WAIT_V(0);
        __syncthreads();
        if (threadIdx.x == 0)
            (void)__hip_atomic_fetch_add((unsigned*)(wsp + OFF_GOT + (posX(0) * 8 + r) * 128 + sub * 4), (unsigned)QUN, __ATOMIC_RELAXED, __HIP_MEMORY_SCOPE_SYSTEM);
        if (threadIdx.x < 64) STAMP(1, 1);
    } else if (g >= 20) {
        __builtin_amdgcn_s_sleep(STAGE_DELAY); __builtin_amdgcn_s_sleep(STAGE_DELAY);
    }
    for (int u = g; u < NUNITS; u += NG) { stage_unit(P, u, callv); if (u == g && threadIdx.x < 64) STAMP(0, 200); }
    if (threadIdx.x < 64) STAMP(0, 1);
    const unsigned call = (unsigned)RFL(callv) + 1u;
    __syncthreads();
    if (threadIdx.x == 0) st_agent(P.wsr + OFF_CALL + blockIdx.x * 64, call);

    const int tid = threadIdx.x, wid = RFL(tid >> 6), lane = tid & 63, wr = wid >> 2, wc = wid & 3, fr = lane & 15, fq = lane >> 4;
    const unsigned ctl = (unsigned)(size_t)lds + STAGE_BYTES;
    char* const wsr = uptr(P.wsr);
    const unsigned target = (unsigned)QUN * call;
    const unsigned voff0 = (unsigned)tid * 16u;
    constexpr size_t kstep = 32 * KiB, hstep = 16 * KiB;
    const unsigned ldsw = (unsigned)wid * 1024u;
    const int aoff = lds_byte(wr * 64 + fr, fq * 8), boff = lds_byte(wc * 32 + fr, fq * 8);
#define G_SA(b, h) (((b) * 2 + (h)) * HTB)
#define G_SB(b, h) ((4 + (b) * 2 + (h)) * HTB)
#define G_STAGE(bufoff, gbase) do { _Pragma("unroll") for (int _i = 0; _i < 2; ++_i) \
        __builtin_amdgcn_global_load_lds((const unsigned*)((const char*)(gbase) + voff0 + _i * 8192), (LAS unsigned*)(lds + (bufoff) + ldsw + _i * 8192), 16, 0, 0); } while (0)
#define G_LDA(dst, b, h) do { _Pragma("unroll") for (int m = 0; m < 4; ++m) _Pragma("unroll") for (int k = 0; k < 2; ++k) dst[m][k] = *(const LAS bf16x8*)(lds + G_SA(b, h) + aoff + m * 2048 + k * 1024); } while (0)
#define G_LDB(dst, b, h) do { _Pragma("unroll") for (int n = 0; n < 2; ++n) _Pragma("unroll") for (int k = 0; k < 2; ++k) dst[n][k] = *(const LAS bf16x8*)(lds + G_SB(b, h) + boff + n * 2048 + k * 1024); } while (0)
#define G_MMA(ai, bj, At, Bt) do { __builtin_amdgcn_s_setprio(1); _Pragma("unroll") for (int m = 0; m < 4; ++m) _Pragma("unroll") for (int n = 0; n < 2; ++n) _Pragma("unroll") for (int k = 0; k < 2; ++k) \
        acc[ai][bj][m][n] = __builtin_amdgcn_mfma_f32_16x16x32_bf16(Bt[n][k], At[m][k], acc[ai][bj][m][n], 0, 0, 0); __builtin_amdgcn_s_setprio(0); } while (0)

    int ui = 0, nhook = 0;
    const u64 mytiles = rfl64(P.tiles[g]);
    unsigned cur = (unsigned)(mytiles & 0xffffu);
    if (cur == 0xffffu) return;
    bool bad = false;
    const __amdgpu_buffer_rsrc_t rws = mkrsrc(wsr, WS_USED);
    auto wait_q = [&](unsigned off, bool whole, unsigned dflt, int qneed) -> int {
        if (wid == 0) {
            const u64 t0 = __builtin_amdgcn_s_memrealtime();
            unsigned res;
            for (;;) {
                const u32x4 v = __builtin_amdgcn_raw_buffer_load_b128(rws, off, 0, AUX_AGT);
                unsigned n = v.x >= target ? (v.y >= target ? (v.z >= target ? (v.w >= target ? 4u : 3u) : 2u) : 1u) : 0u;
                if (whole) n = n == 4u ? 4u : dflt;
                res = (unsigned)(__builtin_amdgcn_ballot_w64(n >= 1u) == ~0ull) + (unsigned)(__builtin_amdgcn_ballot_w64(n >= 2u) == ~0ull)
                    + (unsigned)(__builtin_amdgcn_ballot_w64(n >= 3u) == ~0ull) + (unsigned)(__builtin_amdgcn_ballot_w64(n >= 4u) == ~0ull);
                if ((int)res > qneed) break;
                if (RFL(ld_agent(wsr + OFF_TMO)) == call || __builtin_amdgcn_s_memrealtime() - t0 > DEADLINE) {
                    if (lane == 0) st_agent(wsr + OFF_TMO, call);
                    res = 0x84u; break;
                }
                __builtin_amdgcn_s_sleep(POLL_SLEEP);
            }
            lds_wr32(ctl + CTL_HOOK + (nhook & 7) * 4, res);
        }
        asm volatile("" ::: "memory"); BAR; asm volatile("" ::: "memory");
        const unsigned res = (unsigned)RFL(lds_rd32(ctl + CTL_HOOK + (nhook & 7) * 4));
        ++nhook;
        if (res & 0x80u) bad = true;
        return (int)(res & 7u);
    };
    f32x4 acc[2][2][4][2];
    bf16x8 At[4][2], B0[2][2], B1[2][2];
    const size_t own0 = (size_t)r * SLICE_B;
    unsigned ktab;
    {
        const int T1 = (lane + 1) & 31, k1 = T1 - 4, q1 = (k1 * 37) >> 8;
        ktab = T1 < 4 ? (unsigned)r * SLICE_B + (unsigned)T1 * (64 * KiB) : (unsigned)((r + 1 + k1 - 7 * q1) & 7) * SLICE_B + (unsigned)q1 * (64 * KiB);
    }
    (void)wait_q(OFF_GOT + (unsigned)(((lane & 1) ? posX(T_M(cur)) : posW(T_I(cur))) * 8 + r) * 128u, true, 0u, 3);
    const char* cA = wsr + xpanel_off(T_M(cur), call);
    const char* cB = wsr + OFF_WG + (size_t)wpanel(r, T_I(cur)) * PANEL_B;
    G_STAGE(G_SB(0, 0), cB + own0); G_STAGE(G_SA(0, 0), cA + own0); G_STAGE(G_SB(0, 1), cB + own0 + hstep); G_STAGE(G_SA(0, 1), cA + own0 + hstep);
    if (wr == 1) BAR;
    WAIT_V(4); BAR;
    G_STAGE(G_SB(1, 0), cB + own0 + kstep); G_STAGE(G_SA(1, 0), cA + own0 + kstep); G_STAGE(G_SB(1, 1), cB + own0 + hstep + kstep);
    WAIT_V(6); BAR;
    for (;;) {
#pragma unroll
        for (int a = 0; a < 2; ++a)
#pragma unroll
            for (int b = 0; b < 2; ++b)
#pragma unroll
                for (int m = 0; m < 4; ++m)
#pragma unroll
                    for (int n = 0; n < 2; ++n) acc[a][b][m][n] = (f32x4){0.f, 0.f, 0.f, 0.f};
        const unsigned nxt = (ui + 1 < 4) ? (unsigned)((mytiles >> (16 * (ui + 1))) & 0xffffu) : 0xffffu;
        const bool has_next = nxt != 0xffffu;
        const char* nA = has_next ? wsr + xpanel_off(T_M(nxt), call) : cA;
        const char* nB = has_next ? wsr + OFF_WG + (size_t)wpanel(r, T_I(nxt)) * PANEL_B : cB;
        int have = 0, qn = 0, hookT = 3;
        size_t o0 = own0;
        if (wid == 0) { STAMP(0, 2 + 2 * ui); STAMPC(2, 2 + 2 * ui); }
        for (int T = 0; T < 32; ++T) {
            const bool last = (T == 31);
            if (T == hookT) {
                if (have <= qn) {
                    const unsigned nx2 = has_next ? nxt : cur;
                    int pos = (lane & 16) ? T_S(cur) : ((lane & 8) ? posX(T_M(cur)) : posW(T_I(cur))), src = lane & 7;
                    if (lane >= 24) { pos = (lane & 1) ? posX(T_M(nx2)) : posW(T_I(nx2)); src = r; }
                    have = wait_q(OFF_GOT + (unsigned)(pos * 8 + src) * 128u, lane >= 24, 3u, qn);
                    if (wid == 0 && ui == 0) STAMP(3, qn);
                }
                ++qn; hookT = have >= 4 ? 99 : hookT + 7;
            }
            const size_t o2 = (unsigned)__builtin_amdgcn_readlane(ktab, T);
            const char* a1 = cA + o0 + kstep;
            const char* a2 = (last ? nA : cA) + o2; const char* b2 = (last ? nB : cB) + o2;
            const char* a3 = a2 + kstep; const char* b3 = b2 + kstep;
            o0 = o2;
            G_LDB(B0, 0, 0); SCHED; G_LDA(At, 0, 0); G_STAGE(G_SA(1, 1), a1 + hstep);
            WAIT_L(8); BAR; WAIT_L(0); G_MMA(0, 0, At, B0); BAR; SCHED;
            G_LDB(B1, 0, 1); G_STAGE(G_SB(0, 0), b2);
            BAR; WAIT_L(0); G_MMA(0, 1, At, B1); BAR;
            G_LDA(At, 0, 1); G_STAGE(G_SA(0, 0), a2);
            BAR; WAIT_L(0); G_MMA(1, 0, At, B0); BAR; SCHED;
            G_STAGE(G_SB(0, 1), b2 + hstep);
            WAIT_V(6); BAR; G_MMA(1, 1, At, B1); BAR;
            G_LDB(B0, 1, 0); SCHED; G_LDA(At, 1, 0); G_STAGE(G_SA(0, 1), a2 + hstep);
            WAIT_L(8); BAR; WAIT_L(0); G_MMA(0, 0, At, B0); BAR; SCHED;
            G_LDB(B1, 1, 1); G_STAGE(G_SB(1, 0), b3);
            BAR; WAIT_L(0); G_MMA(0, 1, At, B1); BAR;
            G_LDA(At, 1, 1); G_STAGE(G_SA(1, 0), a3);
            BAR; WAIT_L(0); G_MMA(1, 0, At, B0); BAR; SCHED;
            G_STAGE(G_SB(1, 1), b3 + hstep);
            WAIT_V(6); BAR; G_MMA(1, 1, At, B1); BAR;
        }
        {
            int tz = threadIdx.x; asm volatile("" : "+v"(tz));
            const int l2 = tz & 63, fr2 = l2 & 15, fq2 = l2 >> 4;
            const int m0 = T_M(cur), li = T_I(cur), pn = wpanel(r, li);
            const __amdgpu_buffer_rsrc_t ro = mkrsrc(P.outr, OUT_B);
            const unsigned voff = (unsigned)(fr2 * (NN * 2) + fq2 * 16) + (unsigned)((m0 * 256 + wr * 64) * (NN * 2) + (pn * 256 + wc * 64) * 2);
#pragma unroll
            for (int ai = 0; ai < 2; ++ai)
#pragma unroll
                for (int m = 0; m < 4; ++m)
#pragma unroll
                    for (int bj = 0; bj < 2; ++bj) {
                        const f32x4 v0 = acc[ai][bj][m][0], v1 = acc[ai][bj][m][1];
                        u32x4 o; o.x = cvt_pk_bf16(v0[0], v0[1]); o.y = cvt_pk_bf16(v0[2], v0[3]); o.z = cvt_pk_bf16(v1[0], v1[1]); o.w = cvt_pk_bf16(v1[2], v1[3]);
                        if (bad) o = (u32x4){0x7fc07fc0u, 0x7fc07fc0u, 0x7fc07fc0u, 0x7fc07fc0u};
                        __builtin_amdgcn_raw_buffer_store_b128(o, ro, voff + (unsigned)((ai * 128 + m * 16) * (NN * 2) + bj * 64), 0, AUX_SYS);
                    }
            if (li < NA) {
                WAIT_V(0);
                unsigned old = 0;
                if (l2 == 0) old = lds_add_rtn(ctl + CTL_ACNT + (ui & 7) * 4, 1u);
                if (RFL(old) == 7u && l2 < NR && l2 != r) st_sys(P.ws[l2] + OFF_YF + (r * 128 + m0 * NA + li) * 4, call);
            }
        }
        if (wid == 0) { STAMP(0, 3 + 2 * ui); STAMPC(2, 3 + 2 * ui); }
        ++ui;
        if (!has_next) break;
        cur = nxt; cA = nA; cB = nB;
    }
    WAIT_V(0);
    if (wr == 0) BAR;
}

__device__ __forceinline__ void dispatch_wave(const Params& P, unsigned ctl, int p, unsigned call) {
    const int r = P.rank, lane = threadIdx.x & 63;
    unsigned* const tkp = (unsigned*)(P.wsr + OFF_TK + p * 128);
    const unsigned tk0 = (call - 1u) * TK_PER_CALL;
    for (unsigned b = 0;; ++b) {
        while ((unsigned)RFL(lds_rd32(ctl + CTL_L)) / BAT + 2u < b) __builtin_amdgcn_s_sleep(2);
        unsigned o = 0;
        if (lane == 0) o = __hip_atomic_fetch_add(tkp, (unsigned)BAT, __ATOMIC_RELAXED, __HIP_MEMORY_SCOPE_AGENT);
        const unsigned base = (unsigned)RFL(o) - tk0 + (unsigned)UT0;
        if (lane == 0) lds_wr64(ctl + CTL_GB + (b & 7) * 8, ((u64)(b + 1u) << 32) | base);
        if (b == 0) STAMP(1, 5);
        if (base >= (unsigned)UTF) break;
    }
}

__device__ __forceinline__ void copy_wave(const Params& P, unsigned ctl, int p, int sub, int w, unsigned call) {
    const int r = P.rank, lane = threadIdx.x & 63;
    char* const wsr = P.wsr;
    char* const wsp = P.cws[blockIdx.x];
    const __amdgpu_buffer_rsrc_t rs_ws = mkrsrc(wsp, WS_USED), rd_ws = mkrsrc(wsr, WS_USED);
    const __amdgpu_buffer_rsrc_t rs_out = mkrsrc(P.cout[blockIdx.x], OUT_B), rd_out = mkrsrc(P.outr, OUT_B);
    const unsigned vo_p = (unsigned)lane * 16u, vo_y = (unsigned)(lane >> 5) * (NN * 2) + (unsigned)(lane & 31) * 16u;
    constexpr unsigned CS_P = KiB, CS_Y = 2u * NN * 2u;
    unsigned tabP, tabY0, tabY1;
    {
        bool isw; int idx; pos_item(lane < NP ? lane : 0, isw, idx);
        tabP = (isw ? OFF_WG + (unsigned)wpanel(r, idx) * PANEL_B : OFF_XG + (unsigned)idx * PANEL_B) + (unsigned)p * SLICE_B;
        const int t0 = lane, t1 = lane + 64;
        tabY0 = (unsigned)(256 * (t0 / NA)) * (unsigned)(NN * 2) + (unsigned)(8 * p + t0 % NA) * 512u;
        tabY1 = (unsigned)(256 * (t1 / NA)) * (unsigned)(NN * 2) + (unsigned)(8 * p + t1 % NA) * 512u;
    }
    u64 rin0 = 0, rin1 = 0, rin2 = 0, rin3 = 0, ry0 = 0, ry1 = 0;
    const u64 t0 = __builtin_amdgcn_s_memrealtime();
    int prev = -1, prevb = 0;
    auto retire = [&](int pos, int lb) {
        if (pos >= 0) {
            unsigned old = 0;
            if (lane == 0) old = lds_add_rtn(ctl + CTL_CNT + (lb & 15) * 4, 1u);
            if (((unsigned)RFL(old) + 1u) % BAT == 0u && lane == 0) add_agent(wsr + OFF_GOT + p * 128 + (pos >> 5) * 1024 + (pos & 31) * 4, (unsigned)BAT);
        }
    };
    auto give_up = [&]() -> bool {
        if (RFL(ld_agent(wsr + OFF_TMO)) == call || __builtin_amdgcn_s_memrealtime() - t0 > DEADLINE) { if (lane == 0) st_agent(wsr + OFF_TMO, call); return true; }
        return false;
    };
    bool bad = false;
    STAMP(w, 0);
    for (;;) {
        unsigned o = 0;
        if (lane == 0) o = lds_add_rtn(ctl + CTL_L, 1u);
        const unsigned l = (unsigned)RFL(o), lb = l / BAT;
        u64 gb;
        for (;;) {
            gb = rfl64(lds_rd64(ctl + CTL_GB + (lb & 7) * 8));
            if ((unsigned)(gb >> 32) == lb + 1u) break;
            __builtin_amdgcn_s_sleep(1);
        }
        const unsigned tku = (unsigned)gb + l % BAT;
        if (tku >= (unsigned)UTF) break;
        const int tk = (int)tku;
        u32x4 v[UC];
        if (tk < UPF) {
            int pos, q; quarter_item(tk / QUN, pos, q);
            for (;;) {
                const u64 mk = q == 0 ? rin0 : (q == 1 ? rin1 : (q == 2 ? rin2 : rin3));
                if (bad || ((mk >> pos) & 1)) break;
                const u32x4 f = __builtin_amdgcn_raw_buffer_load_b128(rd_ws, (unsigned)lane * 16u, OFF_INF + (unsigned)p * 1024u, AUX_SYS);
                rin0 = __builtin_amdgcn_ballot_w64(f.x >= call); rin1 = __builtin_amdgcn_ballot_w64(f.y >= call);
                rin2 = __builtin_amdgcn_ballot_w64(f.z >= call); rin3 = __builtin_amdgcn_ballot_w64(f.w >= call);
                const u64 mk2 = q == 0 ? rin0 : (q == 1 ? rin1 : (q == 2 ? rin2 : rin3));
                if ((mk2 >> pos) & 1) break;
                WAIT_V(0); retire(prev, prevb); prev = -1;
                bad = give_up();
                __builtin_amdgcn_s_sleep(1);
            }
            if (bad) continue;
            const unsigned off = (unsigned)__builtin_amdgcn_readlane(tabP, pos) + (unsigned)q * (64 * KiB) + (unsigned)(tk % QUN) * (UC * CS_P);
#pragma unroll
            for (int u = 0; u < UC; ++u) v[u] = __builtin_amdgcn_raw_buffer_load_b128(rs_ws, vo_p + u * CS_P, off, AUX_SYS);
            WAIT_V(0);
#pragma unroll
            for (int u = 0; u < UC; ++u) __builtin_amdgcn_raw_buffer_store_b128(v[u], rd_ws, vo_p + off + u * CS_P, 0, AUX_AGT);
            retire(prev, prevb);
            prev = pos * 32 + q; prevb = (int)lb;
        } else {
            const int t = (tk - UPF) / YUN;
            for (;;) {
                const u64 mk = (NYT <= 64 || t < 64) ? ry0 : ry1;
                if (bad || ((mk >> (t & 63)) & 1)) break;
                const unsigned f0 = ld_sys(wsr + OFF_YF + (p * 128 + lane) * 4), f1 = NYT > 64 ? ld_sys(wsr + OFF_YF + (p * 128 + 64 + lane) * 4) : 0u;
                ry0 = __builtin_amdgcn_ballot_w64(f0 >= call); ry1 = __builtin_amdgcn_ballot_w64(f1 >= call);
                const u64 mk2 = (NYT <= 64 || t < 64) ? ry0 : ry1;
                if ((mk2 >> (t & 63)) & 1) break;
                WAIT_V(0); retire(prev, prevb); prev = -1;
                bad = give_up();
                __builtin_amdgcn_s_sleep(2);
            }
            const unsigned ib = (NYT <= 64 || t < 64) ? (unsigned)__builtin_amdgcn_readlane(tabY0, t & 63) : (unsigned)__builtin_amdgcn_readlane(tabY1, t & 63);
            const unsigned off = ib + (unsigned)((tk - UPF) % YUN) * (UC * CS_Y);
            if (bad) {
#pragma unroll
                for (int u = 0; u < UC; ++u) v[u] = (u32x4){0x7fc07fc0u, 0x7fc07fc0u, 0x7fc07fc0u, 0x7fc07fc0u};
            } else {
#pragma unroll
                for (int u = 0; u < UC; ++u) v[u] = __builtin_amdgcn_raw_buffer_load_b128(rs_out, vo_y + u * CS_Y, off, AUX_SYS);
            }
            WAIT_V(0);
#pragma unroll
            for (int u = 0; u < UC; ++u) __builtin_amdgcn_raw_buffer_store_b128(v[u], rd_out, vo_y + off + u * CS_Y, 0, AUX_AGT);
            retire(prev, prevb);
            prev = -1;
        }
    }
    if (lane == 0) st_sys(wsp + OFF_DONE + (r * 64 + sub * 8 + w) * 4, call);
    WAIT_V(0);
    retire(prev, prevb);
    STAMP(w, 255);
    if (blockIdx.x != 31 || w != 0) return;
    while (!bad) {
        bool ok = true;
#pragma unroll
        for (int k = 0; k < 7; ++k) {
            const int q = (r + 1 + k) & 7;
            const unsigned f = ld_sys(wsr + OFF_DONE + (q * 64 + lane) * 4);
            ok = ok && (lane >= QN * 8 || (lane & 7) >= NWV || f >= call);
        }
        if (__builtin_amdgcn_ballot_w64(ok) == ~0ull) break;
        bad = give_up();
        __builtin_amdgcn_s_sleep(1);
    }
    if (bad && lane == 0) st_sys(P.outr + (size_t)(8 * r * 256) * 2, 0x7fc07fc0u);
    STAMP(7, 1);
}

__global__ __launch_bounds__(512, 2) void gemm_ar_kernel(const Params P) {
    extern __shared__ __attribute__((aligned(16))) unsigned char shm[];
    LAS unsigned char* lds = (LAS unsigned char*)shm;
    const unsigned ctl = (unsigned)(size_t)lds + STAGE_BYTES;
    const int bid = blockIdx.x;
    char* const cptr = P.wsr + OFF_CALL + bid * 64;
    const unsigned callv = ld_agent(cptr);
    if (threadIdx.x < CTL_BYTES / 4) lds_wr32(ctl + threadIdx.x * 4, 0u);
    const int role = bid < 32 ? P.role[bid] : -1;
    if (role >= 0) {
        if (threadIdx.x < 64) STAMP(1, 3);
        const int p = role & 7, sub = role >> 4;
        const int w = RFL(threadIdx.x >> 6);
        const unsigned call = (unsigned)RFL(callv) + 1u;
        WAIT_L(0); BAR;
        if (threadIdx.x == 448) st_agent(cptr, call);
        if (w < NWV) copy_wave(P, ctl, p, sub, w, call);
        else if (w == 7) dispatch_wave(P, ctl, p, call);
    } else {
        gemm_wg(P, lds, bid < 4 ? bid : bid - NC, callv);
    }
}

static inline int gemm_xcd(int g) { return (g < 4 ? g : g + NC) % 8; }

static void build_roles(Params& P) {
    const unsigned quad = 0x4Du;
    const int r = P.rank; int near[3], far[4], nn = 0, nf = 0;
    for (int k = 1; k < NR; ++k) { const int q = (r + k) & 7; if (((quad >> q) & 1) == ((quad >> r) & 1)) near[nn++] = q; else far[nf++] = q; }
    for (int b = 0; b < 32; ++b) {
        const int x = b & 7, k = b >> 3;
        if (b < 4) P.role[b] = -1;
        else if (x < 4) { const int idx = (k - 1) * 4 + x; P.role[b] = near[idx % 3] | ((idx / 3) << 4); }
        else P.role[b] = far[(k + x) & 3] | (k << 4);
        P.cws[b] = P.role[b] >= 0 ? P.ws[P.role[b] & 7] : nullptr;
        P.cout[b] = P.role[b] >= 0 ? P.out[P.role[b] & 7] : nullptr;
    }
}

static void build_tiles(Params& P) {
    static_assert(NG == 8 * 28 + 4, "groups");
    constexpr int NCG = (NW - 1) / 7;
    unsigned t[NG][4]; int cnt[NG];
    for (int g = 0; g < NG; ++g) { for (int k = 0; k < 4; ++k) t[g][k] = 0xffffu; cnt[g] = 0; }
    int wgs[8][29], nw[8] = {0, 0, 0, 0, 0, 0, 0, 0};
    for (int g = 0; g < NG; ++g) { const int x = gemm_xcd(g); wgs[x][nw[x]++] = g; }
    for (int c = 0; c < NCG; c += 2)
        for (int x = 0; x < 8; ++x) {
            const bool two = c + 1 < NCG;
            const int j0 = 7 * (two ? c + (x & 1) : c), rows = two ? 4 : 2, r0 = two ? 4 * (x >> 1) : 2 * x;
            const int sync = imax(posX(r0 + rows - 1), posWj(j0 + 6));
            int n = 0;
            for (int j = j0; j < j0 + 7; ++j)
                for (int m = r0; m < r0 + rows; ++m) { const int g = wgs[x][n++], i = j < NA ? j : j + 1; t[g][cnt[g]++] = (unsigned)((m << 12) | (i << 6) | sync); }
        }
    int sp = 0;
    for (int x = 0; x < 8; ++x)
        if (nw[x] == 29) { const int g = wgs[x][28]; for (int k = 0; k < 4; ++k) { const int m = sp + 4 * k; t[g][k] = (unsigned)((m << 12) | (NA << 6) | imax(posX(m), posW(NA))); } ++sp; }
    for (int g = 0; g < NG; ++g) P.tiles[g] = (u64)t[g][0] | ((u64)t[g][1] << 16) | ((u64)t[g][2] << 32) | ((u64)t[g][3] << 48);
}

}

extern "C" void kernel_launch(int rank, int nranks, ncclComm_t comm,
                              hipStream_t stream,
                              void* const* d_in, const int* in_sizes, int n_in,
                              void* d_out, int out_size,
                              const int* in_dtypes, int out_dtype,
                              void* const* d_out_all,
                              void* const* d_ws_all, size_t ws_bytes) {
    using namespace gar;
    (void)comm; (void)in_sizes; (void)in_dtypes; (void)out_dtype; (void)d_out;
    if (nranks != NR || n_in < 2 || (unsigned)rank >= (unsigned)NR || ws_bytes < WS_USED || (size_t)out_size != (size_t)MM * NN) {
        fprintf(stderr, "gemm_ar: unexpected configuration\n");
        return;
    }
    static bool attr_set[64] = {};
    int dev = 0;
    HIP_CHECK(hipGetDevice(&dev));
    if (dev >= 0 && dev < 64 && !attr_set[dev]) {
        HIP_CHECK(hipFuncSetAttribute((const void*)gemm_ar_kernel, hipFuncAttributeMaxDynamicSharedMemorySize, LDS_BYTES));
        attr_set[dev] = true;
    }
    Params P;
    P.x = (const unsigned short*)d_in[0];
    P.w = (const unsigned short*)d_in[1];
    for (int p = 0; p < NR; ++p) { P.ws[p] = (char*)d_ws_all[p]; P.out[p] = (char*)d_out_all[p]; }
    P.rank = rank; P.pad = 0;
    P.wsr = P.ws[rank]; P.outr = P.out[rank];
    build_roles(P);
    build_tiles(P);
    hipLaunchKernelGGL(gemm_ar_kernel, dim3(NWG), dim3(512), LDS_BYTES, stream, P);
    HIP_CHECK(hipGetLastError());
}
```

```cpp
#include <hip/hip_runtime.h>
#include <rccl/rccl.h>
#include <cstdio>
#include <cstdint>

#define HIP_CHECK(expr) do { hipError_t _e = (expr); if (_e != hipSuccess) { \
    fprintf(stderr, "HIP error %s at %s:%d\n", hipGetErrorString(_e), __FILE__, __LINE__); \
    return; } } while (0)

namespace gar {

#define LAS __attribute__((address_space(3)))
typedef short bf16x8 __attribute__((ext_vector_type(8)));
typedef float f32x4 __attribute__((ext_vector_type(4)));
typedef unsigned u32x4 __attribute__((ext_vector_type(4)));
typedef unsigned long long u64;

constexpr int NR = 8;
constexpr int MM = 4096, NN = 16384, KK = 4096, KS = KK / NR;
constexpr int NXP = MM / 256;
constexpr int NWP = NN / 256;
constexpr int NA = 3;
constexpr int NB = NWP - NR * NA;
constexpr int NW = NA + NB;
constexpr int NP = NXP + NW;
constexpr int NYT = NXP * NA;
constexpr int QN = 4;
constexpr int NPUSH = 1;
static_assert(NPUSH == 0 || NPUSH == 1, "push");
constexpr int NWG = 256, NC = 7 * QN, NG = NWG - NC;
constexpr int NUNITS = (NXP + NWP) * 4;
constexpr int UC = 4;
constexpr int NWT = 23;
constexpr int NWV = (NWT + 3) / 4;
__host__ __device__ constexpr int nwv_of(int sub) { return NWT / 4 + (sub < NWT % 4 ? 1 : 0); }
static_assert(NP <= 64 && NYT <= 128 && NA >= 1 && NA <= 8, "flag masks");
constexpr int BAT = 8;
static_assert(QN >= 1 && QN <= 8 && 64 % UC == 0 && NWV >= 1 && NWV <= 7 && BAT >= NWV && (64 / UC) % BAT == 0, "copy split");
static_assert(NW / 7 <= 8, "tile table");

constexpr unsigned KiB = 1024u, MiB = 1024u * 1024u;
constexpr unsigned SLICE_B = 256 * KiB;
constexpr unsigned PANEL_B = 2 * MiB;
constexpr unsigned OFF_CALL = 0;
constexpr unsigned OFF_TMO = 16 * KiB;
constexpr unsigned OFF_TK = 20 * KiB;
constexpr unsigned OFF_INF = 32 * KiB;
constexpr unsigned OFF_YF = 48 * KiB;
constexpr unsigned OFF_DONE = 56 * KiB;
constexpr unsigned OFF_GOT = 64 * KiB;
constexpr unsigned OFF_XG = 1 * MiB;
constexpr unsigned OFF_WG = OFF_XG + (NXP + 1) * PANEL_B;
constexpr unsigned WS_USED = OFF_WG + NWP * PANEL_B;
constexpr unsigned OUT_B = (unsigned)MM * NN * 2;
constexpr int QUN = 64 / UC;
constexpr int PUN = 256 / UC, YUN = 128 / UC;
constexpr int UPF = NP * PUN, UTF = UPF + NYT * YUN;
constexpr int UT0 = NPUSH * PUN;
constexpr unsigned TK_PER_CALL = UTF - UT0 + QN * BAT;
constexpr u64 DEADLINE = 100000000ull;
constexpr int POLL_SLEEP = 32, STAGE_DELAY = 100, PUSH_EARLY = 2, PUSH_DELAY = 32;

constexpr int HTB = 128 * 64 * 2;
constexpr int STAGE_BYTES = 8 * HTB;
constexpr int CTL_BYTES = 1024;
constexpr int LDS_BYTES = STAGE_BYTES + CTL_BYTES;
constexpr int CTL_HOOK = 0;
constexpr int CTL_ACNT = 32;
constexpr int CTL_L = 0;
constexpr int CTL_GB = 256;
constexpr int CTL_CNT = 512;

__host__ __device__ constexpr int imin(int a, int b) { return a < b ? a : b; }
__host__ __device__ constexpr int imax(int a, int b) { return a > b ? a : b; }
static_assert(NA <= 7 && NW >= 15 && NW % 7 == 1, "pull order");
__host__ __device__ constexpr int posX(int m) { return m < 4 ? m : m + 1; }
__host__ __device__ constexpr int posWj(int j) { return 17 + j; }
__host__ __device__ constexpr int posW(int i) { return i == NA ? 4 : posWj(i < NA ? i : i - 1); }
constexpr int NBS = 8 - NA;
__host__ __device__ inline int wpanel(int r, int i) { if (i < NA) return 8 * r + i; const int b = i - NA; return (b / NBS) * 8 + NA + b % NBS; }
__host__ __device__ inline void pos_item(int pos, bool& isw, int& idx) {
    if (pos >= 17) { const int j = pos - 17; isw = true; idx = j < NA ? j : j + 1; }
    else if (pos == 4) { isw = true; idx = NA; }
    else { isw = false; idx = pos < 4 ? pos : pos - 1; }
}
__device__ __forceinline__ void quarter_item(int qi, int& pos, int& q) {
    if (qi < 4) { pos = 0; q = qi; }
    else if (qi < 20) { q = (qi - 4) >> 2; pos = 1 + ((qi - 4) & 3); }
    else if (qi < 124) { const int k = qi - 20; q = k / 26; pos = 5 + k % 26; }
    else {
        const int k = qi - 124, ph = k / 56, k2 = k % 56, base = 31 + 14 * ph;
        if ((NP - 31) % 14 == 0 || base + 14 <= NP) { q = k2 / 14; pos = base + k2 % 14; }
        else { q = k2 / 7; pos = base + k2 % 7; }
    }
}

__device__ __forceinline__ unsigned xpanel_off(int m, unsigned call) { return OFF_XG + (unsigned)((m < NPUSH && (call & 1u)) ? NXP + m : m) * PANEL_B; }
#define T_M(t) ((int)((t) >> 12))
#define T_I(t) ((int)(((t) >> 6) & 63u))
#define T_S(t) ((int)((t) & 63u))
struct Params {
    const unsigned short* x;
    const unsigned short* w;
    char* wsr;
    char* outr;
    int rank, pad;
    char* ws[NR];
    char* out[NR];
    char* cws[32];
    char* cout[32];
    int role[32];
    u64 tiles[NG];
};

#define WAIT_V(n) asm volatile("s_waitcnt vmcnt(" #n ")" ::: "memory")
#define WAIT_L(n) asm volatile("s_waitcnt lgkmcnt(" #n ")" ::: "memory")
#define BAR __builtin_amdgcn_s_barrier()
#define SCHED __builtin_amdgcn_sched_barrier(0)
#define RFL(x) __builtin_amdgcn_readfirstlane(x)

__device__ __forceinline__ unsigned lds_rd32(unsigned a) { unsigned v; asm volatile("ds_read_b32 %0, %1\n\ts_waitcnt lgkmcnt(0)" : "=&v"(v) : "v"(a) : "memory"); return v; }
__device__ __forceinline__ u64 lds_rd64(unsigned a) { u64 v; asm volatile("ds_read_b64 %0, %1\n\ts_waitcnt lgkmcnt(0)" : "=&v"(v) : "v"(a) : "memory"); return v; }
__device__ __forceinline__ void lds_wr32(unsigned a, unsigned v) { asm volatile("ds_write_b32 %0, %1\n\ts_waitcnt lgkmcnt(0)" :: "v"(a), "v"(v) : "memory"); }
__device__ __forceinline__ void lds_wr64(unsigned a, u64 v) { asm volatile("ds_write_b64 %0, %1\n\ts_waitcnt lgkmcnt(0)" :: "v"(a), "v"(v) : "memory"); }
__device__ __forceinline__ unsigned lds_add_rtn(unsigned a, unsigned v) { unsigned o; asm volatile("ds_add_rtn_u32 %0, %1, %2\n\ts_waitcnt lgkmcnt(0)" : "=&v"(o) : "v"(a), "v"(v) : "memory"); return o; }
__device__ __forceinline__ u64 rfl64(u64 v) { return ((u64)(unsigned)RFL((unsigned)(v >> 32)) << 32) | (unsigned)RFL((unsigned)v); }
__device__ __forceinline__ unsigned ld_sys(const char* p) { return __hip_atomic_load((const unsigned*)p, __ATOMIC_RELAXED, __HIP_MEMORY_SCOPE_SYSTEM); }
__device__ __forceinline__ unsigned ld_agent(const char* p) { return __hip_atomic_load((const unsigned*)p, __ATOMIC_RELAXED, __HIP_MEMORY_SCOPE_AGENT); }
__device__ __forceinline__ void st_sys(char* p, unsigned v) { __hip_atomic_store((unsigned*)p, v, __ATOMIC_RELAXED, __HIP_MEMORY_SCOPE_SYSTEM); }
__device__ __forceinline__ void st_agent(char* p, unsigned v) { __hip_atomic_store((unsigned*)p, v, __ATOMIC_RELAXED, __HIP_MEMORY_SCOPE_AGENT); }
__device__ __forceinline__ void add_agent(char* p, unsigned v) { (void)__hip_atomic_fetch_add((unsigned*)p, v, __ATOMIC_RELAXED, __HIP_MEMORY_SCOPE_AGENT); }
__device__ __forceinline__ unsigned cvt_pk_bf16(float lo, float hi) { unsigned r; asm volatile("v_cvt_pk_bf16_f32 %0, %1, %2" : "=v"(r) : "v"(lo), "v"(hi)); return r; }
__device__ __forceinline__ char* uptr(char* p) { return (char*)rfl64((u64)p); }
__device__ __forceinline__ __amdgpu_buffer_rsrc_t mkrsrc(const void* p, unsigned bytes) {
    const u64 a = (u64)p;
    const u64 ua = ((u64)(unsigned)RFL((unsigned)(a >> 32)) << 32) | (unsigned)RFL((unsigned)a);
    return __builtin_amdgcn_make_buffer_rsrc((void*)ua, 0, bytes, 0x00020000);
}
constexpr int AUX_SYS = 17;
constexpr int AUX_AGT = 16;

__device__ __forceinline__ int lds_byte(int r, int c) { const int st = (r >> 4) * 2 + (c >> 5), rr = r & 15, cc = c & 31, ob = rr * 64 + cc * 2; return st * 1024 + (ob ^ (((ob >> 9) & 1) << 5)); }

template <int NSUB>
__device__ __forceinline__ void stage_x_load(const Params& P, u32x4 (&v)[NSUB], int panel, int quarter, int q0) {
    const int lane = threadIdx.x & 63;
    const int rr = lane >> 2, cc = (lane & 3) ^ (((lane >> 5) & 1) << 1);
#pragma unroll
    for (int e = 0; e < NSUB; ++e) {
        const int q = q0 + e, bidx = q >> 4, st = q & 15;
        const int kt = 2 * quarter + (bidx >> 1), h = bidx & 1;
        const int row = 256 * panel + 128 * h + (st >> 1) * 16 + rr, k = kt * 64 + (st & 1) * 32 + cc * 8;
        v[e] = *(const u32x4*)(P.x + (size_t)row * KS + k);
    }
}
template <int NSUB, int E0 = 0, int E1 = NSUB>
__device__ __forceinline__ void stage_x_store(const Params& P, const u32x4 (&v)[NSUB], __amdgpu_buffer_rsrc_t rd, int panel, int quarter, int q0, unsigned call) {
    const int lane = threadIdx.x & 63;
    const unsigned dst0 = xpanel_off(panel, call) + (unsigned)P.rank * SLICE_B + (unsigned)quarter * 64 * KiB + (unsigned)q0 * KiB;
#pragma unroll
    for (int e = E0; e < E1; ++e) __builtin_amdgcn_raw_buffer_store_b128(v[e], rd, dst0 + lane * 16 + e * 1024, 0, AUX_SYS);
}

__device__ __forceinline__ void stage_unit(const Params& P, int u, unsigned callv) {
    const int r = P.rank, tid = threadIdx.x, lane = tid & 63, wv = RFL(tid >> 6);
    bool isw; int panel, quarter, li = 0, dstrank = -1;
    constexpr int S1 = 26 + 7 * NA;
    if (u < 20) { int idx; pos_item(u >> 2, isw, idx); quarter = u & 3; if (isw) li = idx; panel = isw ? wpanel(r, idx) : idx; if (!isw && panel < NPUSH) dstrank = r; }
    else if (u < 20 + 4 * S1) {
        const int k = u - 20, e = k % S1; quarter = k / S1;
        if (e < 12) { isw = false; panel = 4 + e; }
        else if (e < 12 + 8 * NA) { const int v = e - 12; isw = true; li = v >> 3; dstrank = v & 7; panel = 8 * dstrank + li; }
        else { isw = true; li = e - 12 - 7 * NA + 1; panel = wpanel(r, li); }
    } else { const int k = u - 20 - 4 * S1; isw = true; li = 15 + (k >> 2); quarter = k & 3; panel = wpanel(r, li); }
    const __amdgpu_buffer_rsrc_t rd = mkrsrc(P.wsr, WS_USED);
    if (!isw) {
        u32x4 v[8];
        stage_x_load<8>(P, v, panel, quarter, wv * 8);
        stage_x_store<8>(P, v, rd, panel, quarter, wv * 8, (unsigned)RFL(callv) + 1u);
    } else {
        const int cc = lane & 3, ng = lane >> 2;
        const int ktl = wv >> 2, chalf = (wv >> 1) & 1, nhalf = wv & 1;
        const int k0 = 128 * quarter + 64 * ktl + 32 * chalf + 8 * cc, c0 = 128 * nhalf + 8 * ng;
        u32x4 in[8];
#pragma unroll
        for (int j = 0; j < 8; ++j) in[j] = *(const u32x4*)(P.w + (size_t)(k0 + j) * NN + 256 * panel + c0);
        const int wc = c0 >> 6, bj = (c0 >> 5) & 1, a = (c0 >> 3) & 3;
        const unsigned dst0 = OFF_WG + (unsigned)panel * PANEL_B + (unsigned)r * SLICE_B + (unsigned)(2 * quarter + ktl) * 32 * KiB;
#pragma unroll
        for (int e = 0; e < 8; ++e) {
            u32x4 o;
#pragma unroll
            for (int d = 0; d < 4; ++d) {
                const unsigned lo = in[2 * d][e >> 1], hi = in[2 * d + 1][e >> 1];
                o[d] = (e & 1) ? __builtin_amdgcn_perm(hi, lo, 0x07060302u) : __builtin_amdgcn_perm(hi, lo, 0x05040100u);
            }
            const int R = 32 * wc + 16 * (e >> 2) + 4 * a + (e & 3), st = (R >> 4) * 2 + chalf, rr = R & 15;
            const unsigned off = (unsigned)(bj * 16 * KiB + st * 1024 + rr * 64 + ((cc ^ ((rr >> 3) << 1)) * 16));
            __builtin_amdgcn_raw_buffer_store_b128(o, rd, dst0 + off, 0, AUX_SYS);
        }
    }
    WAIT_V(0);
    __syncthreads();
    if (wv == 0) {
        const unsigned call = (unsigned)RFL(callv) + 1u;
        const int pos = isw ? posW(li) : posX(panel);
        if (lane < NR) {
            const int q = lane;
            if (q != r && (dstrank < 0 || dstrank == q)) st_sys(P.ws[q] + OFF_INF + ((r * 64 + pos) * 4 + quarter) * 4, call);
            if (q == r && (dstrank < 0 || dstrank == r)) add_agent(P.wsr + OFF_GOT + (pos * 8 + r) * 128 + quarter * 4, (unsigned)QUN);
        }
    }
}

__device__ __forceinline__ void gemm_wg(const Params& P, LAS unsigned char* lds, int g, unsigned callv) {
    const int r = P.rank;
    const int pb = (int)blockIdx.x - 64;
    const bool spare = g >= NG - 8 && g < NG - 4;
    if (NPUSH && pb >= 4 && pb < 32) {
        const int wv = RFL(threadIdx.x >> 6), sub = P.role[pb] >> 4;
        char* const wsp = P.cws[pb];
        u32x4 xv[8];
        stage_x_load<8>(P, xv, 0, sub, wv * 8);
        const unsigned c1 = (unsigned)RFL(callv) + 1u;
        const __amdgpu_buffer_rsrc_t rp = mkrsrc(wsp, WS_USED);
        stage_x_store<8, 0, PUSH_EARLY>(P, xv, rp, 0, sub, wv * 8, c1);
        asm volatile("s_waitcnt vmcnt(%0)" :: "n"(PUSH_EARLY) : "memory");
        __builtin_amdgcn_s_sleep(PUSH_DELAY);
        stage_x_store<8, PUSH_EARLY, 8>(P, xv, rp, 0, sub, wv * 8, c1);
        WAIT_V(0);
        __syncthreads();
        if (threadIdx.x == 0)
            (void)__hip_atomic_fetch_add((unsigned*)(wsp + OFF_GOT + (posX(0) * 8 + r) * 128 + sub * 4), (unsigned)QUN, __ATOMIC_RELAXED, __HIP_MEMORY_SCOPE_SYSTEM);
    } else if (g >= 20 && !spare) {
        __builtin_amdgcn_s_sleep(STAGE_DELAY); __builtin_amdgcn_s_sleep(STAGE_DELAY);
    }
    if (!spare)
        for (int u = g < NG - 8 ? g : g - 4; u < NUNITS; u += NG - 4) stage_unit(P, u, callv);
    const unsigned call = (unsigned)RFL(callv) + 1u;
    __syncthreads();
    if (threadIdx.x == 0) st_agent(P.wsr + OFF_CALL + blockIdx.x * 64, call);

    const int tid = threadIdx.x, wid = RFL(tid >> 6), lane = tid & 63, wr = wid >> 2, wc = wid & 3, fr = lane & 15, fq = lane >> 4;
    const unsigned ctl = (unsigned)(size_t)lds + STAGE_BYTES;
    char* const wsr = uptr(P.wsr);
    const unsigned target = (unsigned)QUN * call;
    const unsigned voff0 = (unsigned)tid * 16u;
    constexpr size_t kstep = 32 * KiB, hstep = 16 * KiB;
    const unsigned ldsw = (unsigned)wid * 1024u;
    const int aoff = lds_byte(wr * 64 + fr, fq * 8), boff = lds_byte(wc * 32 + fr, fq * 8);
#define G_SA(b, h) (((b) * 2 + (h)) * HTB)
#define G_SB(b, h) ((4 + (b) * 2 + (h)) * HTB)
#define G_STAGE(bufoff, gbase) do { _Pragma("unroll") for (int _i = 0; _i < 2; ++_i) \
        __builtin_amdgcn_global_load_lds((const unsigned*)((const char*)(gbase) + voff0 + _i * 8192), (LAS unsigned*)(lds + (bufoff) + ldsw + _i * 8192), 16, 0, 0); } while (0)
#define G_LDA(dst, b, h) do { _Pragma("unroll") for (int m = 0; m < 4; ++m) _Pragma("unroll") for (int k = 0; k < 2; ++k) dst[m][k] = *(const LAS bf16x8*)(lds + G_SA(b, h) + aoff + m * 2048 + k * 1024); } while (0)
#define G_LDB(dst, b, h) do { _Pragma("unroll") for (int n = 0; n < 2; ++n) _Pragma("unroll") for (int k = 0; k < 2; ++k) dst[n][k] = *(const LAS bf16x8*)(lds + G_SB(b, h) + boff + n * 2048 + k * 1024); } while (0)
#define G_MMA(ai, bj, At, Bt) do { __builtin_amdgcn_s_setprio(1); _Pragma("unroll") for (int m = 0; m < 4; ++m) _Pragma("unroll") for (int n = 0; n < 2; ++n) _Pragma("unroll") for (int k = 0; k < 2; ++k) \
        acc[ai][bj][m][n] = __builtin_amdgcn_mfma_f32_16x16x32_bf16(Bt[n][k], At[m][k], acc[ai][bj][m][n], 0, 0, 0); __builtin_amdgcn_s_setprio(0); } while (0)

    int ui = 0, nhook = 0;
    const u64 mytiles = rfl64(P.tiles[g]);
    unsigned cur = (unsigned)(mytiles & 0xffffu);
    if (cur == 0xffffu) return;
    bool bad = false;
    const __amdgpu_buffer_rsrc_t rws = mkrsrc(wsr, WS_USED);
    auto wait_q = [&](unsigned off, bool whole, unsigned dflt, int qneed) -> int {
        if (wid == 0) {
            const u64 t0 = __builtin_amdgcn_s_memrealtime();
            unsigned res;
            for (;;) {
                const u32x4 v = __builtin_amdgcn_raw_buffer_load_b128(rws, off, 0, AUX_AGT);
                unsigned n = v.x >= target ? (v.y >= target ? (v.z >= target ? (v.w >= target ? 4u : 3u) : 2u) : 1u) : 0u;
                if (whole) n = n == 4u ? 4u : dflt;
                res = (unsigned)(__builtin_amdgcn_ballot_w64(n >= 1u) == ~0ull) + (unsigned)(__builtin_amdgcn_ballot_w64(n >= 2u) == ~0ull)
                    + (unsigned)(__builtin_amdgcn_ballot_w64(n >= 3u) == ~0ull) + (unsigned)(__builtin_amdgcn_ballot_w64(n >= 4u) == ~0ull);
                if ((int)res > qneed) break;
                if (RFL(ld_agent(wsr + OFF_TMO)) == call || __builtin_amdgcn_s_memrealtime() - t0 > DEADLINE) {
                    if (lane == 0) st_agent(wsr + OFF_TMO, call);
                    res = 0x84u; break;
                }
                __builtin_amdgcn_s_sleep(POLL_SLEEP);
            }
            if (nhook == 0) { __builtin_amdgcn_fence(__ATOMIC_ACQUIRE, "agent"); WAIT_V(0); }
            lds_wr32(ctl + CTL_HOOK + (nhook & 7) * 4, res);
        }
        asm volatile("" ::: "memory"); BAR; asm volatile("" ::: "memory");
        const unsigned res = (unsigned)RFL(lds_rd32(ctl + CTL_HOOK + (nhook & 7) * 4));
        ++nhook;
        if (res & 0x80u) bad = true;
        return (int)(res & 7u);
    };
    f32x4 acc[2][2][4][2];
    bf16x8 At[4][2], B0[2][2], B1[2][2];
    const size_t own0 = (size_t)r * SLICE_B;
    unsigned ktab;
    {
        const int T1 = (lane + 1) & 31, k1 = T1 - 4, q1 = (k1 * 37) >> 8;
        ktab = T1 < 4 ? (unsigned)r * SLICE_B + (unsigned)T1 * (64 * KiB) : (unsigned)((r + 1 + k1 - 7 * q1) & 7) * SLICE_B + (unsigned)q1 * (64 * KiB);
    }
    (void)wait_q(OFF_GOT + (unsigned)(((lane & 1) ? posX(T_M(cur)) : posW(T_I(cur))) * 8 + r) * 128u, true, 0u, 3);
    const char* cA = wsr + xpanel_off(T_M(cur), call);
    const char* cB = wsr + OFF_WG + (size_t)wpanel(r, T_I(cur)) * PANEL_B;
    G_STAGE(G_SB(0, 0), cB + own0); G_STAGE(G_SA(0, 0), cA + own0); G_STAGE(G_SB(0, 1), cB + own0 + hstep); G_STAGE(G_SA(0, 1), cA + own0 + hstep);
    if (wr == 1) BAR;
    WAIT_V(4); BAR;
    G_STAGE(G_SB(1, 0), cB + own0 + kstep); G_STAGE(G_SA(1, 0), cA + own0 + kstep); G_STAGE(G_SB(1, 1), cB + own0 + hstep + kstep);
    WAIT_V(6); BAR;
    for (;;) {
#pragma unroll
        for (int a = 0; a < 2; ++a)
#pragma unroll
            for (int b = 0; b < 2; ++b)
#pragma unroll
                for (int m = 0; m < 4; ++m)
#pragma unroll
                    for (int n = 0; n < 2; ++n) acc[a][b][m][n] = (f32x4){0.f, 0.f, 0.f, 0.f};
        const unsigned nxt = (ui + 1 < 4) ? (unsigned)((mytiles >> (16 * (ui + 1))) & 0xffffu) : 0xffffu;
        const bool has_next = nxt != 0xffffu;
        const char* nA = has_next ? wsr + xpanel_off(T_M(nxt), call) : cA;
        const char* nB = has_next ? wsr + OFF_WG + (size_t)wpanel(r, T_I(nxt)) * PANEL_B : cB;
        int have = 0, qn = 0, hookT = 3;
        size_t o0 = own0;
        for (int T = 0; T < 32; ++T) {
            const bool last = (T == 31);
            if (T == hookT) {
                if (have <= qn) {
                    const unsigned nx2 = has_next ? nxt : cur;
                    int pos = (lane & 16) ? T_S(cur) : ((lane & 8) ? posX(T_M(cur)) : posW(T_I(cur))), src = lane & 7;
                    if (lane >= 24) { pos = (lane & 1) ? posX(T_M(nx2)) : posW(T_I(nx2)); src = r; }
                    have = wait_q(OFF_GOT + (unsigned)(pos * 8 + src) * 128u, lane >= 24, 3u, qn);
                }
                ++qn; hookT = have >= 4 ? 99 : hookT + 7;
            }
            const size_t o2 = (unsigned)__builtin_amdgcn_readlane(ktab, T);
            const char* a1 = cA + o0 + kstep;
            const char* a2 = (last ? nA : cA) + o2; const char* b2 = (last ? nB : cB) + o2;
            const char* a3 = a2 + kstep; const char* b3 = b2 + kstep;
            o0 = o2;
            G_LDB(B0, 0, 0); SCHED; G_LDA(At, 0, 0); G_STAGE(G_SA(1, 1), a1 + hstep);
            WAIT_L(8); BAR; WAIT_L(0); G_MMA(0, 0, At, B0); BAR; SCHED;
            G_LDB(B1, 0, 1); G_STAGE(G_SB(0, 0), b2);
            BAR; WAIT_L(0); G_MMA(0, 1, At, B1); BAR;
            G_LDA(At, 0, 1); G_STAGE(G_SA(0, 0), a2);
            BAR; WAIT_L(0); G_MMA(1, 0, At, B0); BAR; SCHED;
            G_STAGE(G_SB(0, 1), b2 + hstep);
            WAIT_V(6); BAR; G_MMA(1, 1, At, B1); BAR;
            G_LDB(B0, 1, 0); SCHED; G_LDA(At, 1, 0); G_STAGE(G_SA(0, 1), a2 + hstep);
            WAIT_L(8); BAR; WAIT_L(0); G_MMA(0, 0, At, B0); BAR; SCHED;
            G_LDB(B1, 1, 1); G_STAGE(G_SB(1, 0), b3);
            BAR; WAIT_L(0); G_MMA(0, 1, At, B1); BAR;
            G_LDA(At, 1, 1); G_STAGE(G_SA(1, 0), a3);
            BAR; WAIT_L(0); G_MMA(1, 0, At, B0); BAR; SCHED;
            G_STAGE(G_SB(1, 1), b3 + hstep);
            WAIT_V(6); BAR; G_MMA(1, 1, At, B1); BAR;
        }
        {
            int tz = threadIdx.x; asm volatile("" : "+v"(tz));
            const int l2 = tz & 63, fr2 = l2 & 15, fq2 = l2 >> 4;
            const int m0 = T_M(cur), li = T_I(cur), pn = wpanel(r, li);
            const __amdgpu_buffer_rsrc_t ro = mkrsrc(P.outr, OUT_B);
            const unsigned voff = (unsigned)(fr2 * (NN * 2) + fq2 * 16) + (unsigned)((m0 * 256 + wr * 64) * (NN * 2) + (pn * 256 + wc * 64) * 2);
#pragma unroll
            for (int ai = 0; ai < 2; ++ai)
#pragma unroll
                for (int m = 0; m < 4; ++m)
#pragma unroll
                    for (int bj = 0; bj < 2; ++bj) {
                        const f32x4 v0 = acc[ai][bj][m][0], v1 = acc[ai][bj][m][1];
                        u32x4 o; o.x = cvt_pk_bf16(v0[0], v0[1]); o.y = cvt_pk_bf16(v0[2], v0[3]); o.z = cvt_pk_bf16(v1[0], v1[1]); o.w = cvt_pk_bf16(v1[2], v1[3]);
                        if (bad) o = (u32x4){0x7fc07fc0u, 0x7fc07fc0u, 0x7fc07fc0u, 0x7fc07fc0u};
                        __builtin_amdgcn_raw_buffer_store_b128(o, ro, voff + (unsigned)((ai * 128 + m * 16) * (NN * 2) + bj * 64), 0, AUX_SYS);
                    }
            if (li < NA) {
                WAIT_V(0);
                unsigned old = 0;
                if (l2 == 0) old = lds_add_rtn(ctl + CTL_ACNT + (ui & 7) * 4, 1u);
                if (RFL(old) == 7u && l2 < NR && l2 != r) st_sys(P.ws[l2] + OFF_YF + (r * 128 + m0 * NA + li) * 4, call);
            }
        }
        ++ui;
        if (!has_next) break;
        cur = nxt; cA = nA; cB = nB;
    }
    WAIT_V(0);
    if (wr == 0) BAR;
}

__device__ __forceinline__ void dispatch_wave(const Params& P, unsigned ctl, int p, unsigned call) {
    const int lane = threadIdx.x & 63;
    unsigned* const tkp = (unsigned*)(P.wsr + OFF_TK + p * 128);
    const unsigned tk0 = (call - 1u) * TK_PER_CALL;
    const u64 t0 = __builtin_amdgcn_s_memrealtime();
    for (unsigned b = 0;; ++b) {
        while ((unsigned)RFL(lds_rd32(ctl + CTL_L)) / BAT + 2u < b && __builtin_amdgcn_s_memrealtime() - t0 < 2 * DEADLINE - DEADLINE / 4) __builtin_amdgcn_s_sleep(2);
        unsigned o = 0;
        if (lane == 0) o = __hip_atomic_fetch_add(tkp, (unsigned)BAT, __ATOMIC_RELAXED, __HIP_MEMORY_SCOPE_AGENT);
        const unsigned base = (unsigned)RFL(o) - tk0 + (unsigned)UT0;
        if (lane == 0) lds_wr64(ctl + CTL_GB + (b & 31) * 8, ((u64)(b + 1u) << 32) | base);
        if (base >= (unsigned)UTF) break;
    }
}

__device__ __forceinline__ void copy_wave(const Params& P, unsigned ctl, int p, int sub, int w, unsigned call) {
    const int r = P.rank, lane = threadIdx.x & 63;
    char* const wsr = P.wsr;
    char* const wsp = P.cws[blockIdx.x];
    const __amdgpu_buffer_rsrc_t rs_ws = mkrsrc(wsp, WS_USED), rd_ws = mkrsrc(wsr, WS_USED);
    const __amdgpu_buffer_rsrc_t rs_out = mkrsrc(P.cout[blockIdx.x], OUT_B), rd_out = mkrsrc(P.outr, OUT_B);
    const unsigned vo_p = (unsigned)lane * 16u, vo_y = (unsigned)(lane >> 5) * (NN * 2) + (unsigned)(lane & 31) * 16u;
    constexpr unsigned CS_P = KiB, CS_Y = 2u * NN * 2u;
    unsigned tabP, tabY0, tabY1;
    {
        bool isw; int idx; pos_item(lane < NP ? lane : 0, isw, idx);
        tabP = (isw ? OFF_WG + (unsigned)wpanel(r, idx) * PANEL_B : OFF_XG + (unsigned)idx * PANEL_B) + (unsigned)p * SLICE_B;
        const int t0 = lane, t1 = lane + 64;
        tabY0 = (unsigned)(256 * (t0 / NA)) * (unsigned)(NN * 2) + (unsigned)(8 * p + t0 % NA) * 512u;
        tabY1 = (unsigned)(256 * (t1 / NA)) * (unsigned)(NN * 2) + (unsigned)(8 * p + t1 % NA) * 512u;
    }
    u64 rin0 = 0, rin1 = 0, rin2 = 0, rin3 = 0, ry0 = 0, ry1 = 0;
    const u64 t0 = __builtin_amdgcn_s_memrealtime();
    int prev = -1, prevb = 0;
    auto retire = [&](int pos, int lb) {
        if (pos >= 0) {
            unsigned old = 0;
            if (lane == 0) old = lds_add_rtn(ctl + CTL_CNT + (lb & 63) * 4, 1u);
            if (((unsigned)RFL(old) + 1u) % BAT == 0u && lane == 0) add_agent(wsr + OFF_GOT + p * 128 + (pos >> 5) * 1024 + (pos & 31) * 4, (unsigned)BAT);
        }
    };
    unsigned npoll = 0;
    auto give_up = [&]() -> bool {
        if ((++npoll & 7u) != 0u) return false;
        if (RFL(ld_agent(wsr + OFF_TMO)) == call || __builtin_amdgcn_s_memrealtime() - t0 > DEADLINE) { if (lane == 0) st_agent(wsr + OFF_TMO, call); return true; }
        return false;
    };
    bool bad = false;
    for (;;) {
        unsigned o = 0;
        if (lane == 0) o = lds_add_rtn(ctl + CTL_L, 1u);
        const unsigned l = (unsigned)RFL(o), lb = l / BAT;
        u64 gb;
        for (;;) {
            gb = rfl64(lds_rd64(ctl + CTL_GB + (lb & 31) * 8));
            if ((unsigned)(gb >> 32) == lb + 1u) break;
            if (__builtin_amdgcn_s_memrealtime() - t0 > DEADLINE + DEADLINE / 2) {
                if (lane == 0) { st_agent(wsr + OFF_TMO, call); st_sys(P.outr + (size_t)(8 * p * 256) * 2, 0x7fc07fc0u); }
                bad = true; gb = (u64)(unsigned)UTF; break;
            }
            __builtin_amdgcn_s_sleep(1);
        }
        const unsigned tku = (unsigned)gb + l % BAT;
        if (tku >= (unsigned)UTF) break;
        const int tk = (int)tku;
        u32x4 v[UC];
        if (tk < UPF) {
            int pos, q; quarter_item(tk / QUN, pos, q);
            for (;;) {
                const u64 mk = q == 0 ? rin0 : (q == 1 ? rin1 : (q == 2 ? rin2 : rin3));
                if (bad || ((mk >> pos) & 1)) break;
                const u32x4 f = __builtin_amdgcn_raw_buffer_load_b128(rd_ws, (unsigned)lane * 16u, OFF_INF + (unsigned)p * 1024u, AUX_SYS);
                rin0 = __builtin_amdgcn_ballot_w64(f.x >= call); rin1 = __builtin_amdgcn_ballot_w64(f.y >= call);
                rin2 = __builtin_amdgcn_ballot_w64(f.z >= call); rin3 = __builtin_amdgcn_ballot_w64(f.w >= call);
                const u64 mk2 = q == 0 ? rin0 : (q == 1 ? rin1 : (q == 2 ? rin2 : rin3));
                if ((mk2 >> pos) & 1) break;
                WAIT_V(0); retire(prev, prevb); prev = -1;
                bad = give_up();
                __builtin_amdgcn_s_sleep(1);
            }
            if (bad) continue;
            const unsigned off = (unsigned)__builtin_amdgcn_readlane(tabP, pos) + (unsigned)q * (64 * KiB) + (unsigned)(tk % QUN) * (UC * CS_P);
#pragma unroll
            for (int u = 0; u < UC; ++u) v[u] = __builtin_amdgcn_raw_buffer_load_b128(rs_ws, vo_p + u * CS_P, off, AUX_SYS);
            WAIT_V(0);
#pragma unroll
            for (int u = 0; u < UC; ++u) __builtin_amdgcn_raw_buffer_store_b128(v[u], rd_ws, vo_p + off + u * CS_P, 0, AUX_AGT);
            retire(prev, prevb);
            prev = pos * 32 + q; prevb = (int)lb;
        } else {
            const int t = (tk - UPF) / YUN;
            for (;;) {
                const u64 mk = (NYT <= 64 || t < 64) ? ry0 : ry1;
                if (bad || ((mk >> (t & 63)) & 1)) break;
                const unsigned f0 = ld_sys(wsr + OFF_YF + (p * 128 + lane) * 4), f1 = NYT > 64 ? ld_sys(wsr + OFF_YF + (p * 128 + 64 + lane) * 4) : 0u;
                ry0 = __builtin_amdgcn_ballot_w64(f0 >= call); ry1 = __builtin_amdgcn_ballot_w64(f1 >= call);
                const u64 mk2 = (NYT <= 64 || t < 64) ? ry0 : ry1;
                if ((mk2 >> (t & 63)) & 1) break;
                WAIT_V(0); retire(prev, prevb); prev = -1;
                bad = give_up();
                __builtin_amdgcn_s_sleep(2);
            }
            const unsigned ib = (NYT <= 64 || t < 64) ? (unsigned)__builtin_amdgcn_readlane(tabY0, t & 63) : (unsigned)__builtin_amdgcn_readlane(tabY1, t & 63);
            const unsigned off = ib + (unsigned)((tk - UPF) % YUN) * (UC * CS_Y);
            if (bad) {
#pragma unroll
                for (int u = 0; u < UC; ++u) v[u] = (u32x4){0x7fc07fc0u, 0x7fc07fc0u, 0x7fc07fc0u, 0x7fc07fc0u};
            } else {
#pragma unroll
                for (int u = 0; u < UC; ++u) v[u] = __builtin_amdgcn_raw_buffer_load_b128(rs_out, vo_y + u * CS_Y, off, AUX_SYS);
            }
            WAIT_V(0);
#pragma unroll
            for (int u = 0; u < UC; ++u) __builtin_amdgcn_raw_buffer_store_b128(v[u], rd_out, vo_y + off + u * CS_Y, 0, AUX_AGT);
            retire(prev, prevb);
            prev = -1;
        }
    }
    if (lane == 0) st_sys(wsp + OFF_DONE + (r * 64 + sub * 8 + w) * 4, call);
    WAIT_V(0);
    retire(prev, prevb);
    if (blockIdx.x != 31 || w != 0) return;
    while (!bad) {
        bool ok = true;
#pragma unroll
        for (int k = 0; k < 7; ++k) {
            const int q = (r + 1 + k) & 7;
            const unsigned f = ld_sys(wsr + OFF_DONE + (q * 64 + lane) * 4);
            ok = ok && (lane >= QN * 8 || (lane & 7) >= nwv_of(lane >> 3) || f >= call);
        }
        if (__builtin_amdgcn_ballot_w64(ok) == ~0ull) break;
        bad = give_up();
        __builtin_amdgcn_s_sleep(1);
    }
    if (bad && lane == 0) st_sys(P.outr + (size_t)(8 * r * 256) * 2, 0x7fc07fc0u);
}

__global__ __launch_bounds__(512, 2) void gemm_ar_kernel(const Params P) {
    extern __shared__ __attribute__((aligned(16))) unsigned char shm[];
    LAS unsigned char* lds = (LAS unsigned char*)shm;
    const unsigned ctl = (unsigned)(size_t)lds + STAGE_BYTES;
    const int bid = blockIdx.x;
    char* const cptr = P.wsr + OFF_CALL + bid * 64;
    const unsigned callv = ld_agent(cptr);
    if (threadIdx.x < CTL_BYTES / 4) lds_wr32(ctl + threadIdx.x * 4, 0u);
    const int role = bid < 32 ? P.role[bid] : -1;
    if (role >= 0) {
        const int p = role & 7, sub = role >> 4;
        const int w = RFL(threadIdx.x >> 6);
        const unsigned call = (unsigned)RFL(callv) + 1u;
        WAIT_L(0); BAR;
        if (threadIdx.x == 448) st_agent(cptr, call);
        if (w < nwv_of(sub)) copy_wave(P, ctl, p, sub, w, call);
        else if (w == 7) dispatch_wave(P, ctl, p, call);
    } else {
        gemm_wg(P, lds, bid < 4 ? bid : bid - NC, callv);
    }
}

static inline int gemm_xcd(int g) { return (g < 4 ? g : g + NC) % 8; }

static void build_roles(Params& P) {
    const unsigned quad = 0x4Du;
    const int r = P.rank; int near[3], far[4], nn = 0, nf = 0;
    for (int k = 1; k < NR; ++k) { const int q = (r + k) & 7; if (((quad >> q) & 1) == ((quad >> r) & 1)) near[nn++] = q; else far[nf++] = q; }
    for (int b = 0; b < 32; ++b) {
        const int x = b & 7, k = b >> 3;
        if (b < 4) P.role[b] = -1;
        else if (x < 4) { const int idx = (k - 1) * 4 + x; P.role[b] = near[idx % 3] | ((idx / 3) << 4); }
        else P.role[b] = far[(k + x) & 3] | (k << 4);
        P.cws[b] = P.role[b] >= 0 ? P.ws[P.role[b] & 7] : nullptr;
        P.cout[b] = P.role[b] >= 0 ? P.out[P.role[b] & 7] : nullptr;
    }
}

static void build_tiles(Params& P) {
    static_assert(NG == 8 * 28 + 4, "groups");
    constexpr int NCG = (NW - 1) / 7;
    unsigned t[NG][4]; int cnt[NG];
    for (int g = 0; g < NG; ++g) { for (int k = 0; k < 4; ++k) t[g][k] = 0xffffu; cnt[g] = 0; }
    int wgs[8][29], nw[8] = {0, 0, 0, 0, 0, 0, 0, 0};
    for (int g = 0; g < NG; ++g) { const int x = gemm_xcd(g); wgs[x][nw[x]++] = g; }
    for (int c = 0; c < NCG; c += 2)
        for (int x = 0; x < 8; ++x) {
            const bool two = c + 1 < NCG;
            const int j0 = 7 * (two ? c + (x & 1) : c), rows = two ? 4 : 2, r0 = two ? 4 * (x >> 1) : 2 * x;
            const int sync = imax(posX(r0 + rows - 1), posWj(j0 + 6));
            int n = 0;
            for (int j = j0; j < j0 + 7; ++j)
                for (int m = r0; m < r0 + rows; ++m) { const int g = wgs[x][n++], i = j < NA ? j : j + 1; t[g][cnt[g]++] = (unsigned)((m << 12) | (i << 6) | sync); }
        }
    int sp = 0;
    for (int x = 0; x < 8; ++x)
        if (nw[x] == 29) { const int g = wgs[x][28];   for (int k = 0; k < 4; ++k) { const int m = sp + 4 * k; t[g][k] = (unsigned)((m << 12) | (NA << 6) | imax(posX(m), posW(NA))); } ++sp; }
    for (int g = 0; g < NG; ++g) P.tiles[g] = (u64)t[g][0] | ((u64)t[g][1] << 16) | ((u64)t[g][2] << 32) | ((u64)t[g][3] << 48);
}

}


extern "C" void kernel_launch(int rank, int nranks, ncclComm_t comm,
                              hipStream_t stream,
                              void* const* d_in, const int* in_sizes, int n_in,
                              void* d_out, int out_size,
                              const int* in_dtypes, int out_dtype,
                              void* const* d_out_all,
                              void* const* d_ws_all, size_t ws_bytes) {
    using namespace gar;
    (void)comm; (void)in_sizes; (void)in_dtypes; (void)out_dtype; (void)d_out;
    if (nranks != NR || n_in < 2 || (unsigned)rank >= (unsigned)NR || ws_bytes < WS_USED || (size_t)out_size != (size_t)MM * NN) {
        fprintf(stderr, "gemm_ar: unexpected configuration\n");
        return;
    }
    static bool attr_set[64] = {};
    int dev = 0;
    HIP_CHECK(hipGetDevice(&dev));
    if (dev >= 0 && dev < 64 && !attr_set[dev]) {
        HIP_CHECK(hipFuncSetAttribute((const void*)gemm_ar_kernel, hipFuncAttributeMaxDynamicSharedMemorySize, LDS_BYTES));
        attr_set[dev] = true;
    }
    Params P;
    P.x = (const unsigned short*)d_in[0];
    P.w = (const unsigned short*)d_in[1];
    for (int p = 0; p < NR; ++p) { P.ws[p] = (char*)d_ws_all[p]; P.out[p] = (char*)d_out_all[p]; }
    P.rank = rank; P.pad = 0;
    P.wsr = P.ws[rank]; P.outr = P.out[rank];
    build_roles(P);
    build_tiles(P);
    hipLaunchKernelGGL(gemm_ar_kernel, dim3(NWG), dim3(512), LDS_BYTES, stream, P);
    HIP_CHECK(hipGetLastError());
}
```
